```python
import numpy as np

kernel_name = "gpu_gemm_a2a_m4096_k4096_n4096_e4m3e4m3_dqsilu"


ABSMAX_RTOL = 0.03

COMPUTE_DTYPE = "float32"
EPILOGUE = "dequant_silu"
IN_QUANT = None
OUT_QUANT = None
IN_FP8 = "e4m3e4m3"
IN_DTYPES = {"x": "e4m3", "w_mat": "e4m3",
             "scale_x": "float32", "scale_w": "float32"}
WORLD_SIZE = 8

def _snap_fp8(x, mant_bits, max_abs, min_normal):
    a = np.abs(x.astype(np.float64))
    _m, _e = np.frexp(a)
    q = np.where(a >= min_normal, np.ldexp(1.0, _e - 1 - mant_bits),
                 min_normal * 2.0 ** -mant_bits)
    snapped = np.minimum(np.round(a / q) * q, max_abs)
    return (np.sign(x) * snapped).astype(np.float32)

def setup_inputs():

    x = _snap_fp8(np.random.randn(4096, 4096).astype(np.float32), 3, 448.0, 2.0 ** -6)
    w_mat = _snap_fp8(np.random.randn(4096, 4096).astype(np.float32), 3, 448.0, 2.0 ** -6)
    scale_x = (0.02 * (1.0 + np.random.rand(1))).astype(np.float32)
    scale_w = (0.02 * (1.0 + np.random.rand(1))).astype(np.float32)
    return {"x": x, "w_mat": w_mat, "scale_x": scale_x, "scale_w": scale_w}

def reference(x, w_mat, scale_x, scale_w):
    acc = (x @ w_mat).astype(np.float32)
    y = (acc * (scale_x * scale_w)).astype(np.float32)
    return (y / (1.0 + np.exp(-np.clip(y, -60.0, 60.0)))).astype(np.float32)
```

```cpp
#include <hip/hip_runtime.h>
#include <rccl/rccl.h>
#include <cstdio>
#include <cstring>

#define HIP_CHECK(expr) do { hipError_t _e = (expr); if (_e != hipSuccess) { \
    fprintf(stderr, "HIP error %s at %s:%d\n", hipGetErrorString(_e), __FILE__, __LINE__); \
    return; } } while (0)

namespace {

typedef int v2i __attribute__((ext_vector_type(2)));
typedef int v4i __attribute__((ext_vector_type(4)));
typedef int v8i __attribute__((ext_vector_type(8)));
typedef float v4f __attribute__((ext_vector_type(4)));
typedef unsigned v4u __attribute__((ext_vector_type(4)));
#define LDS_AS __attribute__((address_space(3)))

constexpr int NR = 8;
constexpr int NWG = 256;
constexpr int NT = 512;
constexpr int KD = 4096;
constexpr int MSH = 512;
constexpr int NSH = 512;
#define XM_ROWS 32
constexpr int XM = XM_ROWS;
constexpr int MAXJ = 3;
constexpr int WAVE_LDS = 20480;
constexpr unsigned long long DEADLINE_TICKS = 100000000ull;

constexpr size_t WS_CNT = 0;
constexpr size_t WS_TMO = 2048;
constexpr size_t WS_OFLAG = 4096;
constexpr size_t WS_XFLAG = 8192;
constexpr size_t WS_XST = 65536;
constexpr size_t WS_XST_PAR = (size_t)NR * 64 * KD;
constexpr size_t WS_DBG = WS_XST + 2 * WS_XST_PAR;
constexpr size_t WS_NEED = WS_DBG + 65536;

struct Args {
    const unsigned char* x;
    const unsigned char* w;
    const float* sx;
    const float* sw;
    float* out[NR];
    unsigned char* ws[NR];
    int rank;
    int nflags;
    unsigned jobs[NWG][MAXJ];
};

__device__ __forceinline__ void glds_pair(unsigned lds, unsigned v0, unsigned v1, const unsigned char* base) {
    unsigned keep;
    asm volatile("s_mov_b32 %0, m0\n\t"
                 "s_mov_b32 m0, %1\n\t"
                 "s_nop 2\n\t"
                 "global_load_lds_dwordx4 %2, %4 sc0 sc1\n\t"
                 "global_load_lds_dwordx4 %3, %4 offset:1024 sc0 sc1\n\t"
                 "s_mov_b32 m0, %0"
                 : "=&s"(keep) : "s"(lds), "v"(v0), "v"(v1), "s"(base) : "memory");
}

__device__ __forceinline__ const unsigned char* uniform_ptr(const unsigned char* p) {
    const unsigned long long v = (unsigned long long)p;
    const unsigned lo = __builtin_amdgcn_readfirstlane((unsigned)v), hi = __builtin_amdgcn_readfirstlane((unsigned)(v >> 32));
    return (const unsigned char*)(((unsigned long long)hi << 32) | lo);
}

#define VMW(n) asm volatile("s_waitcnt vmcnt(" #n ")" ::: "memory")
__device__ __forceinline__ void wait_pairs(int n) {
    if (n >= 9) VMW(18);
    else if (n >= 6) { if (n == 6) VMW(12); else if (n == 7) VMW(14); else VMW(16); }
    else if (n >= 3) { if (n == 3) VMW(6); else if (n == 4) VMW(8); else VMW(10); }
    else { if (n == 2) VMW(4); else if (n == 1) VMW(2); else VMW(0); }
}

__device__ __forceinline__ void store16_sys(void* p, v4f v) {
    asm volatile("global_store_dwordx4 %0, %1, off sc0 sc1\n\ts_nop 1" :: "v"(p), "v"(v) : "memory");
}
__device__ __forceinline__ void store16_sys_u(void* p, v4u v) {
    asm volatile("global_store_dwordx4 %0, %1, off sc0 sc1\n\ts_nop 1" :: "v"(p), "v"(v) : "memory");
}
__device__ __forceinline__ void barrier() {
    asm volatile("s_waitcnt lgkmcnt(0)\n\ts_barrier" ::: "memory");
}
__device__ __forceinline__ float act(float acc, float s) {
    float y = acc * s;
    float c = fminf(fmaxf(y, -60.0f), 60.0f);
    return y / (1.0f + expf(-c));
}

#define STAMP(i) do {} while (0)

__global__ __launch_bounds__(NT) void fused_kernel(const Args a) {
    __shared__ __attribute__((aligned(1024))) unsigned char smem[8 * WAVE_LDS];
    const unsigned long long t_start = __builtin_amdgcn_s_memrealtime();
    const int tid = threadIdx.x;
    const int lane = tid & 63;
    const int wave = __builtin_amdgcn_readfirstlane(tid >> 6);
    const int b = blockIdx.x;
    const int rank = a.rank & (NR - 1);
    unsigned char* const myws = a.ws[rank];
    unsigned* const tmo = (unsigned*)(myws + WS_TMO);
    STAMP(0);

    const unsigned callv = __hip_atomic_load((unsigned*)(myws + WS_CNT) + b, __ATOMIC_RELAXED, __HIP_MEMORY_SCOPE_SYSTEM);
    auto get_call = [&]() -> unsigned {
        unsigned t = callv;
        asm volatile("" : "+v"(t));
        return (unsigned)__builtin_amdgcn_readfirstlane(t) + 1u;
    };

    if (XM > 0 && b < XM) {
        const unsigned char* src = a.x + (size_t)(MSH - XM + b) * KD + (wave & 3) * 1024 + lane * 16;
        const v4u v = *(const v4u*)src;
        const unsigned call = get_call();
        const size_t off = WS_XST + (size_t)(call & 1u) * WS_XST_PAR + ((size_t)rank * 64 + b) * KD + (wave & 3) * 1024 + lane * 16;
        const int r0 = (wave >> 2) ? 5 : 1, r1 = (wave >> 2) ? 8 : 5;
        for (int r = r0; r < r1; ++r) store16_sys_u(a.ws[(rank + r) & (NR - 1)] + off, v);
        VMW(0);
        barrier();
        if (tid >= 1 && tid < NR)
            __hip_atomic_store((unsigned*)(a.ws[(rank + tid) & (NR - 1)] + WS_XFLAG) + rank * 64 + b, call, __ATOMIC_RELAXED, __HIP_MEMORY_SCOPE_SYSTEM);
        STAMP(1);
    }

    const unsigned lds0 = (unsigned)(size_t)(LDS_AS unsigned char*)smem;
    const unsigned ldsw = lds0 + wave * WAVE_LDS;
    unsigned wo[2][2], xo[2];
    {
        const int row = lane >> 2, pos = lane & 3;
        for (int par = 0; par < 2; ++par) {
            const unsigned o = row * KD + ((pos ^ (((row >> 2) & 1) | (par << 1))) << 4);
            wo[par][0] = o;
            wo[par][1] = o + 16 * KD - 1024;
        }
        for (int q = 0; q < 2; ++q) {
            const int m = 8 * q + (lane >> 3), p8 = lane & 7;
            const int sg = (((m >> 1) & 1) << 1) | ((m >> 2) & 1) | ((m >> 3) << 2);
            xo[q] = m * KD + ((p8 ^ sg) << 4) - q * 1024;
        }
    }
    const int g = lane >> 4, i16 = lane & 15;
    unsigned wr[4], xr[2];
    {
        const int q = i16 >> 1, p = i16 & 1;
        const int f = ((q >> 2) & 1) | ((g & 1) << 1);
        for (int j = 0; j < 4; ++j) wr[j] = q * 64 + ((j ^ f) << 4) + 8 * p;
        const int m = i16;
        const int sg = (((m >> 1) & 1) << 1) | ((m >> 2) & 1) | ((m >> 3) << 2);
        for (int h = 0; h < 2; ++h) xr[h] = m * 128 + (((2 * g + h) ^ sg) << 4);
    }

    for (int jn = 0; jn < MAXJ; ++jn) {
        const unsigned job = a.jobs[b][jn];
        const int nmt = (job & 15u) > 8u ? 8 : (int)(job & 15u);
        if (nmt == 0) break;
        const int staged = (job >> 15) & 1;
        const int rel = (job >> 12) & 7;
        const int strip = (job >> 9) & 7;
        int row0 = ((job >> 4) & 31) * 16;
        const int other = (rank + rel) & (NR - 1);
        const int dest = staged ? rank : other;
        bool bad = false;
        const unsigned char* xbase;
        int orow;
        if (staged) {
            const unsigned call = get_call();
            if (row0 + nmt * 16 > XM) row0 = 0;
            xbase = myws + WS_XST + (size_t)(call & 1u) * WS_XST_PAR + ((size_t)other * 64 + row0) * KD;
            orow = other * MSH + (MSH - XM) + row0;
            const unsigned* fl = (const unsigned*)(myws + WS_XFLAG) + other * 64 + (lane < XM ? lane : 0);
            const unsigned long long t0 = __builtin_amdgcn_s_memrealtime();
            while (true) {
                const unsigned v = __hip_atomic_load(fl, __ATOMIC_RELAXED, __HIP_MEMORY_SCOPE_SYSTEM);
                if (__builtin_amdgcn_ballot_w64(v < call) == 0ull) break;
                if (__builtin_amdgcn_readfirstlane(__hip_atomic_load(tmo, __ATOMIC_RELAXED, __HIP_MEMORY_SCOPE_AGENT)) == call
                    || __builtin_amdgcn_s_memrealtime() - t0 > DEADLINE_TICKS) {
                    if (lane == 0) __hip_atomic_store(tmo, call, __ATOMIC_RELAXED, __HIP_MEMORY_SCOPE_AGENT);
                    bad = true;
                    break;
                }
                __builtin_amdgcn_s_sleep(2);
            }
            VMW(0);
        } else {
            if (row0 + nmt * 16 > MSH) row0 = MSH - nmt * 16;
            xbase = a.x + (size_t)row0 * KD;
            orow = rank * MSH + row0;
        }
        const unsigned delay = (job >> 24) * 5u;
        while (__builtin_amdgcn_s_memrealtime() - t_start < delay) __builtin_amdgcn_s_sleep(1);

        const unsigned char* wsrc = a.w + (size_t)(wave * 512) * KD + dest * NSH + strip * 64;
        const unsigned char* xsrc = uniform_ptr(xbase + wave * 512);

        const int per = 4 + nmt, total = 4 * per;
        int issued = 0, pstep = 0, pitem = 0, pslot = 0;
        auto issue = [&]() {
            if (issued < total) {
                const unsigned dst = ldsw + pslot * 2048;
                if (pitem < 4) {
                    const unsigned char* src = wsrc + (size_t)(pstep * 128 + pitem * 32) * KD;
                    if (pitem & 1) glds_pair(dst, wo[1][0], wo[1][1], src);
                    else glds_pair(dst, wo[0][0], wo[0][1], src);
                } else {
                    const unsigned char* src = xsrc + (size_t)((pitem - 4) * 16) * KD + pstep * 128;
                    glds_pair(dst, xo[0], xo[1], src);
                }
                ++issued;
                if (++pitem == per) { pitem = 0; ++pstep; }
                if (++pslot == 10) pslot = 0;
            }
        };
        #pragma unroll 1
        for (int i = 0; i < 10; ++i) issue();

        v4f acc[8][4];
        #pragma unroll
        for (int i = 0; i < 8; ++i)
            #pragma unroll
            for (int j = 0; j < 4; ++j) acc[i][j] = (v4f){0.f, 0.f, 0.f, 0.f};

        int done = 0, cslot = 0;
        #pragma unroll 1
        for (int s = 0; s < 4; ++s) {
            wait_pairs(issued - done - 4);
            int sl = cslot + g;
            if (sl >= 10) sl -= 10;
            const unsigned wb = ldsw + sl * 2048;
            v8i wf[4];
            #pragma unroll
            for (int j = 0; j < 4; ++j) {
                #pragma unroll
                for (int r = 0; r < 4; ++r) {
                    const v2i t = __builtin_amdgcn_ds_read_tr8_b64_v2i32((LDS_AS v2i*)(size_t)(wb + wr[j] + r * 512));
                    wf[j][2 * r] = t[0];
                    wf[j][2 * r + 1] = t[1];
                }
            }
            asm volatile("s_waitcnt lgkmcnt(0)" ::: "memory");
            done += 4;
            cslot += 4;
            if (cslot >= 10) cslot -= 10;
            #pragma unroll 1
            for (int i = 0; i < 4; ++i) issue();
            #pragma unroll
            for (int i = 0; i < 8; ++i) {
                if (i < nmt) {
                    wait_pairs(issued - done - 1);
                    const unsigned xb = ldsw + cslot * 2048;
                    const v4i x0 = *(const LDS_AS v4i*)(size_t)(xb + xr[0]);
                    const v4i x1 = *(const LDS_AS v4i*)(size_t)(xb + xr[1]);
                    asm volatile("s_waitcnt lgkmcnt(0)" ::: "memory");
                    ++done;
                    if (++cslot == 10) cslot = 0;
                    issue();
                    const v8i xf = {x0[0], x0[1], x0[2], x0[3], x1[0], x1[1], x1[2], x1[3]};
                    #pragma unroll
                    for (int j = 0; j < 4; ++j)
                        acc[i][j] = __builtin_amdgcn_mfma_scale_f32_16x16x128_f8f6f4(wf[j], xf, acc[i][j], 0, 0, 0, 127, 0, 127);
                }
            }
        }
        STAMP(2 + 4 * jn);

        const unsigned call = get_call();
        const float scale = a.sx[0] * a.sw[0];
        if (staged) bad = bad || __builtin_amdgcn_readfirstlane(__hip_atomic_load(tmo, __ATOMIC_RELAXED, __HIP_MEMORY_SCOPE_AGENT)) == call;
        v4f res[4];
        #pragma unroll
        for (int rd = 0; rd < 2; ++rd) {
            if (rd * 4 < nmt) {
                if (rd) barrier();
                #pragma unroll
                for (int i = 0; i < 4; ++i) {
                    if (rd * 4 + i < nmt) {
                        #pragma unroll
                        for (int j = 0; j < 4; ++j)
                            *(LDS_AS v4f*)(size_t)(ldsw + i * 4096 + i16 * 256 + (((4 * j + g) ^ i16) << 4)) = acc[rd * 4 + i][j];
                    }
                }
                barrier();
                #pragma unroll
                for (int u = 0; u < 2; ++u) {
                    const int un = wave + 8 * u;
                    if (rd * 16 + un < nmt * 4) {
                        const int m = (un & 3) * 4 + g;
                        const unsigned ad = lds0 + (un >> 2) * 4096 + m * 256 + ((i16 ^ m) << 4);
                        v4f sum = *(const LDS_AS v4f*)(size_t)ad;
                        #pragma unroll
                        for (int k = 1; k < 8; ++k) sum += *(const LDS_AS v4f*)(size_t)(ad + k * WAVE_LDS);
                        v4f o;
                        #pragma unroll
                        for (int e = 0; e < 4; ++e) o[e] = bad ? __builtin_nanf("") : act(sum[e], scale);
                        res[rd * 2 + u] = o;
                    }
                }
            }
        }
        STAMP(3 + 4 * jn);

        unsigned char* const obase = (unsigned char*)a.out[dest] + (size_t)(orow + g) * (NSH * 4) + strip * 256 + i16 * 16;
        const int lph = (job >> 16) & 3;
        const int myph = wave >> (3 - lph);
        #pragma unroll 1
        for (int ph = 0; ph < (1 << lph); ++ph) {
            if (ph == myph) {
                #pragma unroll
                for (int k = 0; k < 4; ++k) {
                    const int un = (k >> 1) * 16 + wave + 8 * (k & 1);
                    if (un < nmt * 4) {
                        store16_sys(obase + (size_t)un * 4 * (NSH * 4), res[k]);
                        if (lph) VMW(0);
                    }
                }
                VMW(0);
            }
            barrier();
        }
        if (dest != rank && tid == 0)
            __hip_atomic_store((unsigned*)(a.ws[dest] + WS_OFLAG) + rank * 64 + ((job >> 18) & 63u), call, __ATOMIC_RELAXED, __HIP_MEMORY_SCOPE_SYSTEM);
        STAMP(4 + 4 * jn);
    }

    const unsigned call = get_call();
    if (tid == 0) __hip_atomic_store((unsigned*)(myws + WS_CNT) + b, call, __ATOMIC_RELAXED, __HIP_MEMORY_SCOPE_SYSTEM);
    if (b == 0 && wave == 0) {
        const int srcr = lane >> 3, f0 = (lane & 7) * 8;
        const unsigned char* fp = myws + WS_OFLAG + lane * 32;
        const unsigned long long t0 = __builtin_amdgcn_s_memrealtime();
        while (true) {
            v4u f0v, f1v;
            asm volatile("global_load_dwordx4 %0, %2, off sc0 sc1\n\t"
                         "global_load_dwordx4 %1, %2, off offset:16 sc0 sc1\n\t"
                         "s_waitcnt vmcnt(0)" : "=&v"(f0v), "=&v"(f1v) : "v"(fp) : "memory");
            bool miss = false;
            #pragma unroll
            for (int e = 0; e < 4; ++e) {
                miss = miss || (f0 + e < a.nflags && f0v[e] < call);
                miss = miss || (f0 + 4 + e < a.nflags && f1v[e] < call);
            }
            if (srcr == rank) miss = false;
            if (__builtin_amdgcn_ballot_w64(miss) == 0ull) break;
            if (__builtin_amdgcn_readfirstlane(__hip_atomic_load(tmo, __ATOMIC_RELAXED, __HIP_MEMORY_SCOPE_AGENT)) == call
                || __builtin_amdgcn_s_memrealtime() - t0 > DEADLINE_TICKS) {
                if (lane == 0) {
                    __hip_atomic_store(tmo, call, __ATOMIC_RELAXED, __HIP_MEMORY_SCOPE_AGENT);
                    a.out[rank][(size_t)rank * MSH * NSH] = __builtin_nanf("");
                }
                break;
            }
            __builtin_amdgcn_s_sleep(1);
        }
        __builtin_amdgcn_fence(__ATOMIC_ACQUIRE, "");
        STAMP(15);
    }
}

inline unsigned mkjob(int nmt, int row16, int strip, int rel, int staged, int lph, int flag, int delay50) {
    return (unsigned)nmt | (unsigned)row16 << 4 | (unsigned)strip << 9 | (unsigned)rel << 12 | (unsigned)staged << 15 |
           (unsigned)lph << 16 | (unsigned)flag << 18 | (unsigned)delay50 << 24;
}

#define DLY2 0
#define DLY3 0

void build_jobs(Args& a) {
    memset(a.jobs, 0, sizeof(a.jobs));
    constexpr int XT = XM / 16;
    constexpr int CT = 32 - XT;
    constexpr int rest = CT - 15;
    constexpr int s0 = rest > 8 ? (rest + 1) / 2 : rest, s1 = rest - s0;
    static_assert(s0 <= 8 && s1 <= 8 && s1 >= 0, "tiles");
    for (int b = 0; b < NWG; ++b) {
        const int k = b & 7, idx = b >> 3;
        if (idx < 4) {
            a.jobs[b][0] = mkjob(8, idx * 8, k, 0, 0, 0, 0, 0);
            continue;
        }
        const int gs = k * 7 + (idx - 4) % 7, gen = (idx - 4) / 7;
        const int rel = 1 + gs / 8, strip = gs % 8;
        if (gen == 0) {
            a.jobs[b][0] = mkjob(1, 0, strip, rel, 0, 0, strip * 6 + 0, 0);
            a.jobs[b][1] = mkjob(s0, 15, strip, rel, 0, 1, strip * 6 + 4, 0);
        } else if (gen == 1) {
            a.jobs[b][0] = mkjob(2, 1, strip, rel, 0, 0, strip * 6 + 1, 0);
            a.jobs[b][1] = mkjob(s1, 15 + s0, strip, rel, 0, 1, strip * 6 + 5, 0);
        } else if (gen == 2) {
            a.jobs[b][0] = mkjob(4, 3, strip, rel, 0, 0, strip * 6 + 2, DLY2);
            if (XT > 0) a.jobs[b][1] = mkjob(XT, 0, strip, rel, 1, 0, 0, 0);
        } else {
            a.jobs[b][0] = mkjob(8, 7, strip, rel, 0, 1, strip * 6 + 3, DLY3);
        }
    }
    a.nflags = 48;
}

}

extern "C" void kernel_launch(int rank, int nranks, ncclComm_t comm,
                              hipStream_t stream,
                              void* const* d_in, const int* in_sizes, int n_in,
                              void* d_out, int out_size,
                              const int* in_dtypes, int out_dtype,
                              void* const* d_out_all,
                              void* const* d_ws_all, size_t ws_bytes) {
    (void)comm; (void)in_dtypes; (void)out_dtype;
    if (nranks != NR || n_in != 4 || (unsigned)rank >= (unsigned)NR || ws_bytes < WS_NEED ||
        in_sizes[0] != MSH * KD || in_sizes[1] != KD * NR * NSH || out_size != NR * MSH * NSH) {
        fprintf(stderr, "kernel_launch: unexpected problem shape\n");
        return;
    }
    Args a;
    a.x = (const unsigned char*)d_in[0];
    a.w = (const unsigned char*)d_in[1];
    a.sx = (const float*)d_in[2];
    a.sw = (const float*)d_in[3];
    for (int p = 0; p < NR; ++p) {
        a.out[p] = (float*)d_out_all[p];
        a.ws[p] = (unsigned char*)d_ws_all[p];
    }
    a.out[rank] = (float*)d_out;
    a.rank = rank;
    build_jobs(a);
    hipLaunchKernelGGL(fused_kernel, dim3(NWG), dim3(NT), 0, stream, a);
    HIP_CHECK(hipGetLastError());
}
```

```cpp
#include <hip/hip_runtime.h>
#include <rccl/rccl.h>
#include <cstdio>
#include <cstring>
#include <type_traits>

#define HIP_CHECK(expr) do { hipError_t _e = (expr); if (_e != hipSuccess) { \
    fprintf(stderr, "HIP error %s at %s:%d\n", hipGetErrorString(_e), __FILE__, __LINE__); \
    return; } } while (0)

namespace {

typedef int v2i __attribute__((ext_vector_type(2)));
typedef int v4i __attribute__((ext_vector_type(4)));
typedef int v8i __attribute__((ext_vector_type(8)));
typedef float v2f __attribute__((ext_vector_type(2)));
typedef float v4f __attribute__((ext_vector_type(4)));
typedef unsigned v4u __attribute__((ext_vector_type(4)));
typedef unsigned v16u __attribute__((ext_vector_type(16)));
typedef unsigned char u8;
#define LDS_AS __attribute__((address_space(3)))
#define CONST_AS __attribute__((address_space(4)))

constexpr int NR = 8;
constexpr int NWG = 256;
constexpr int NT = 512;
constexpr int KD = 4096;
constexpr int MSH = 512;
constexpr int NSH = 512;
#define ABL 0
#define XM_ROWS 32
constexpr int XM = XM_ROWS;
constexpr int XT = XM / 16;
constexpr int XPW = XM / 16;
constexpr unsigned long long DEADLINE_TICKS = 100000000ull;

constexpr int LDS_BUF = 32768;
constexpr int LDS_BYTES = LDS_BUF + 8 * 16384;

constexpr size_t WS_CNT = 0;
constexpr size_t WS_TMO = 2048;
constexpr size_t WS_OFLAG = 4096;
constexpr size_t WS_XFLAG = 8192;
constexpr size_t WS_XST = 65536;
constexpr size_t WS_XST_PAR = (size_t)7 * 64 * KD;
constexpr size_t WS_SCR = WS_XST + 2 * WS_XST_PAR;
constexpr size_t WS_DBG = WS_SCR + (1u << 20);
constexpr size_t WS_NEED = WS_DBG + 131072;

struct Rec {
    const u8* x;
    const u8* w;
    u8* out;
    u8* dws;
    u8* myws;
    const float* sx;
    const float* sw;
    unsigned rank;
    unsigned first4;
};
struct alignas(64) Args {
    Rec rec[NR];
    unsigned jobs[2][32];
};
static_assert(sizeof(Rec) == 64, "layout");

#define GLDS4(POL) \
    asm volatile("s_mov_b32 m0, %0\n\t" \
                 "s_nop 0\n\t" \
                 "global_load_lds_dwordx4 %1, %5" POL "\n\t" \
                 "global_load_lds_dwordx4 %2, %5 offset:1024" POL "\n\t" \
                 "global_load_lds_dwordx4 %3, %5 offset:2048" POL "\n\t" \
                 "global_load_lds_dwordx4 %4, %5 offset:3072" POL \
                 :: "s"(lds), "v"(v0), "v"(v1), "v"(v2), "v"(v3), "s"(base) : "memory", "m0")
__device__ __forceinline__ void glds4(unsigned lds, unsigned v0, unsigned v1, unsigned v2, unsigned v3, const u8* base) {
    GLDS4("");
}
#define GLDS2(POL) \
    asm volatile("s_mov_b32 m0, %0\n\t" \
                 "s_mov_b64 exec, %4\n\t" \
                 "global_load_lds_dwordx4 %1, %3 offset:%5" POL "\n\t" \
                 "global_load_lds_dwordx4 %2, %3 offset:%6" POL "\n\t" \
                 "s_mov_b64 exec, -1" \
                 :: "s"(lds), "v"(v0), "v"(v1), "s"(base), "s"(mask), "n"(O), "n"(O + 1024) : "memory", "m0")
template <int O>
__device__ __forceinline__ void glds2(unsigned lds, unsigned v0, unsigned v1, const u8* base, unsigned long long mask) {
    GLDS2("");
}
template <int O>
__device__ __forceinline__ void glds2_sys(unsigned lds, unsigned v0, unsigned v1, const u8* base, unsigned long long mask) {
    GLDS2(" sc0 sc1");
}
__device__ __forceinline__ void glds1(unsigned lds, unsigned v0, const u8* base) {
    asm volatile("s_mov_b32 m0, %0\n\ts_nop 0\n\tglobal_load_lds_dwordx4 %1, %2" :: "s"(lds), "v"(v0), "s"(base) : "memory", "m0");
}

#define VMW(n) asm volatile("s_waitcnt vmcnt(" #n ")" ::: "memory")
#define LGKM0() asm volatile("s_waitcnt lgkmcnt(0)" ::: "memory")
#define PIN() __builtin_amdgcn_sched_barrier(0)

__device__ __forceinline__ void store16_sys(unsigned voff, v4f v, const void* base) {
    asm volatile("global_store_dwordx4 %0, %1, %2 sc0 sc1\n\ts_nop 1" :: "v"(voff), "v"(v), "s"(base) : "memory");
}
__device__ __forceinline__ void store8_sys(unsigned voff, v2f v, const void* base) {
    asm volatile("global_store_dwordx2 %0, %1, %2 sc0 sc1" :: "v"(voff), "v"(v), "s"(base) : "memory");
}
__device__ __forceinline__ void store4_sys(unsigned voff, unsigned v, const void* base) {
    asm volatile("global_store_dword %0, %1, %2 sc0 sc1" :: "v"(voff), "v"(v), "s"(base) : "memory");
}
__device__ __forceinline__ v4f load16(unsigned voff, const void* base) {
    v4f r;
    asm volatile("global_load_dwordx4 %0, %1, %2" : "=v"(r) : "v"(voff), "s"(base) : "memory");
    return r;
}
__device__ __forceinline__ void barrier() {
    asm volatile("s_waitcnt lgkmcnt(0)\n\ts_barrier" ::: "memory");
}
__device__ __forceinline__ unsigned first(unsigned v) { return (unsigned)__builtin_amdgcn_readfirstlane(v); }

__device__ __forceinline__ float act(float acc, float s) {
    const float y = acc * s;
    const float c = __builtin_amdgcn_fmed3f(y, -60.0f, 60.0f);
    const float t = c * -1.44269502f;
    const float r = __builtin_fmaf(c, -1.92596299e-8f, __builtin_fmaf(c, -1.44269502f, -t));
    const float e0 = __builtin_amdgcn_exp2f(t);
    const float d = __builtin_fmaf(e0, r * 0.693147182f, e0) + 1.0f;
    const float i0 = __builtin_amdgcn_rcpf(d);
    const float q = y * i0;
    return __builtin_fmaf(__builtin_fmaf(-q, d, y), i0, q);
}

#define STAMP(i) do {} while (0)
#define CSTAMP(i) do {} while (0)

__device__ __forceinline__ u8* ptr(unsigned lo, unsigned hi) {
    return (u8*)(((unsigned long long)hi << 32) | lo);
}

__global__ __launch_bounds__(NT) void fused_kernel(const Args args) {
    __shared__ __attribute__((aligned(1024))) u8 smem[LDS_BYTES];
    const int b = blockIdx.x;
    const int rel = b & 7, idx = (b >> 3) & 31;
    v16u rec;
    unsigned job;
    unsigned long long t_start;
    asm volatile("s_load_dwordx16 %0, %3, %4\n\t"
                 "s_load_dword %1, %3, %5\n\t"
                 "s_memrealtime %2\n\t"
                 "s_waitcnt lgkmcnt(0)"
                 : "=&s"(rec), "=&s"(job), "=&s"(t_start)
                 : "s"(__builtin_amdgcn_kernarg_segment_ptr()), "s"(rel * 64), "s"(512 + (rel ? 0 : 128) + idx * 4) : "memory");
    const u8* const ax = ptr(rec[0], rec[1]);
    const u8* const aw = ptr(rec[2], rec[3]);
    u8* const aout = ptr(rec[4], rec[5]);
    u8* const dws = ptr(rec[6], rec[7]);
    u8* const myws = ptr(rec[8], rec[9]);
    const int rank = rec[14] & (NR - 1);
    const int tid = threadIdx.x;
    const int lane = tid & 63;
    const int wave = __builtin_amdgcn_readfirstlane(tid >> 6);
    const u8* const cntp = myws + WS_CNT + rel * 128 + idx * 4;
    const unsigned lds0 = (unsigned)(size_t)(LDS_AS u8*)smem;
    const unsigned ldsw = lds0 + LDS_BUF + wave * 16384;
    const unsigned ldsr = lds0 + wave * 4096;
    const int g = lane >> 4, i16 = lane & 15;
    unsigned call;
    float sclx, sclw;
    auto load_scalars = [&]() {
        asm volatile("s_load_dword %0, %3, 0x0\n\t"
                     "s_load_dword %1, %4, 0x0\n\t"
                     "s_load_dword %2, %5, 0x0\n\t"
                     "s_waitcnt lgkmcnt(0)"
                     : "=&s"(call), "=&s"(sclx), "=&s"(sclw)
                     : "s"(cntp), "s"(ptr(rec[10], rec[11])), "s"(ptr(rec[12], rec[13])) : "memory");
        call += 1u;
    };

    if (rel != 0 && idx >= 8) {
        auto help = [&]() {
            const int hw = (idx - 8) * 8 + wave;
            #pragma unroll
            for (int j = 0; j < 4; ++j) {
                const int id = j * 192 + hw;
                if (id < 256) {
                    const unsigned tl = (rec[15] >> (5 * (id >> 6))) & 31u;
                    glds1(ldsw + j * 1024, lane * 16, ax + (size_t)tl * (16 * KD) + (id & 63) * 1024);
                } else {
                    const int k = id - 256;
                    const int row = ((k >> 3) & 7) * 512 + (k >> 6) * 64 + (k & 7) * 8;
                    glds1(ldsw + j * 1024, (lane >> 3) * KD + (lane & 7) * 16, aw + (size_t)row * KD);
                }
            }
        };
        if (idx >= 24) {
            v4f xv[XPW];
            const unsigned xoff = ((idx & 7) * 8 + wave) * (XPW * 1024) + lane * 16;
            #pragma unroll
            for (int i = 0; i < XPW; ++i) xv[i] = load16(xoff + i * 1024, ax + (size_t)(MSH - XM) * KD);
            help();
            load_scalars();
            const u8* dst = dws + WS_XST + (size_t)(call & 1u) * WS_XST_PAR + (size_t)(rel - 1) * (XM * KD);
            VMW(4);
            #pragma unroll
            for (int i = 0; i < XPW; ++i) {
                asm volatile("" : "+v"(xv[i]));
                store16_sys(xoff + i * 1024, xv[i], dst);
            }
            VMW(0);
            barrier();
            if (tid == 0) store4_sys(0, call, dws + WS_XFLAG + ((rel - 1) * 8 + (idx & 7)) * 4);
            STAMP(1);
        } else {
            help();
        }
        VMW(0);
    }

    int nown = job & 31u;
    if (nown < 4) nown = 4;
    int t0 = (job >> 5) & 31u;
    if (t0 + nown > 32) t0 = 32 - nown;
    int nstg = rel ? 0 : (job >> 24) & 7u;
    int s0 = job >> 27;
    if (s0 + nstg > 7 * XT) nstg = 0;
    const int nt = nown + nstg;
    const int strip = (job >> 10) & 7;
    const int xg = (job >> 13) & 7;
    const unsigned delay = ((job >> 16) & 255u) * 5u;

    unsigned wo[4];
    {
        const int row = lane >> 2, pos = lane & 3;
        #pragma unroll
        for (int p = 0; p < 4; ++p)
            wo[p] = (16 * p + row) * KD + ((pos ^ (((row >> 2) & 1) | ((p & 1) << 1))) << 4) - p * 1024;
    }
    unsigned xo[8];
    #pragma unroll
    for (int p = 0; p < 8; ++p) {
        const int m = 8 * (p & 1) + (lane >> 3);
        const int cs = (lane & 7) ^ (m >> 1);
        xo[p] = m * KD + ((((p >> 1) << 7) + (cs << 4)) ^ (xg << 6)) - (p & 3) * 1024 + 4096;
    }
    const u8* const wsrc = aw + (size_t)(wave * 512) * KD + strip * 64;
    const u8* xin = ax + wave * 512 - 4096 + (size_t)t0 * (16 * KD);
    auto issue_x = [&](unsigned buf) {
        glds4(buf, xo[0], xo[1], xo[2], xo[3], xin);
        glds4(buf + 4096, xo[4], xo[5], xo[6], xo[7], xin);
        xin += 16 * KD;
    };
    auto issue_w = [&](int h, unsigned unit) {
        glds4(unit, wo[0], wo[1], wo[2], wo[3], wsrc + (size_t)((h ^ xg) * 64) * KD);
    };

    if (delay) while ((unsigned)__builtin_amdgcn_s_memrealtime() - (unsigned)t_start < delay) __builtin_amdgcn_s_sleep(1);
    STAMP(7); CSTAMP(10);

    issue_x(ldsw + 8192);
    issue_w(0, ldsw);
    asm volatile("s_barrier" ::: "memory");
    issue_w(1, ldsw + 4096);
    issue_w(2, ldsr);
    PIN();
    load_scalars();
    const float scale = sclx * sclw;
    const LDS_AS u8* wrw[4];
    const LDS_AS u8* wrr[4];
    {
        const int q = i16 >> 1, p = i16 & 1;
        const int f = ((q >> 2) & 1) | ((g & 1) << 1);
        #pragma unroll
        for (int j = 0; j < 4; ++j) {
            const unsigned o = g * 1024 + q * 64 + ((j ^ f) << 4) + 8 * p;
            wrw[j] = (const LDS_AS u8*)(size_t)(ldsw + o);
            wrr[j] = (const LDS_AS u8*)(size_t)(ldsr + o);
        }
    }
    const unsigned xr0 = ldsw + i16 * 128 + ((g ^ (i16 >> 1)) << 4);
    const int om = 2 * wave + (lane >> 5);
    const unsigned ovoff = om * (NSH * 4) + (lane & 31) * 8;
    const LDS_AS u8* const prd = (const LDS_AS u8*)(size_t)(lds0 + om * 256 + (((((lane & 31) >> 1) ^ om) << 4) | ((lane & 1) << 3)));
    LDS_AS u8* pwr[4];
    #pragma unroll
    for (int j = 0; j < 4; ++j) pwr[j] = (LDS_AS u8*)(size_t)(ldsr + i16 * 256 + (((4 * j + g) ^ i16) << 4));
    float cinit = 0.f;
    v8i wf[4][4], xf[4];
    v4f acc[4];
    auto read_xs = [&](unsigned ad, int s) {
        const v4i lo = *(const LDS_AS v4i*)((const LDS_AS u8*)(size_t)ad + s * 2048);
        const v4i hi = *(const LDS_AS v4i*)((const LDS_AS u8*)(size_t)(ad ^ 64u) + s * 2048);
        return (v8i){lo[0], lo[1], lo[2], lo[3], hi[0], hi[1], hi[2], hi[3]};
    };
    auto read_x = [&](unsigned ad) {
        #pragma unroll
        for (int s = 0; s < 4; ++s) xf[s] = read_xs(ad, s);
        LGKM0();
    };
    auto read_w = [&](int h, const LDS_AS u8* const* base, int off) {
        #pragma unroll
        for (int j = 0; j < 4; ++j) {
            #pragma unroll
            for (int r = 0; r < 2; ++r) {
                const v2i t = __builtin_amdgcn_ds_read_tr8_b64_v2i32((LDS_AS v2i*)(base[j] + off + r * 512));
                wf[h >> 1][j][(h & 1) * 4 + 2 * r] = t[0];
                wf[h >> 1][j][(h & 1) * 4 + 2 * r + 1] = t[1];
            }
        }
        LGKM0();
    };
    auto mma1 = [&](int s, int j, v4f c) {
        return __builtin_amdgcn_mfma_scale_f32_16x16x128_f8f6f4(wf[s][j], xf[s], c, 0, 0, 0, 0, 0, 0);
    };
    auto mma = [&](int s) {
        #pragma unroll
        for (int j = 0; j < 4; ++j) acc[j] = mma1(s, j, s ? acc[j] : (v4f){0.f, 0.f, 0.f, 0.f});
    };
    VMW(12);
    read_x(xr0 + 8192);
    issue_w(3, ldsw + 8192);
    issue_w(4, ldsw + 12288);
    VMW(16); read_w(0, wrw, 0);      issue_w(5, ldsw);
    VMW(16); read_w(1, wrw, 4096);   issue_w(6, ldsw + 4096);   PIN(); mma(0); PIN();
    VMW(16); read_w(2, wrr, 0);      issue_w(7, ldsr);
    asm volatile("s_barrier" ::: "memory");
    VMW(16); read_w(3, wrw, 8192);                              PIN(); mma(1); PIN();
    VMW(12); read_w(4, wrw, 12288);  issue_x(ldsw + 8192);
    VMW(16); read_w(5, wrw, 0);      glds4(ldsw, xo[0], xo[1], xo[2], xo[3], xin);
                                                                PIN(); mma(2); PIN();
    VMW(16); read_w(6, wrw, 4096);
    VMW(12); read_w(7, wrr, 0);                                 PIN(); mma(3); PIN();
    STAMP(2); CSTAMP(11);

    auto tiles = [&](auto local_) {
        constexpr bool LOCAL = decltype(local_)::value;
        const u8* const obase = aout + strip * 256;
        auto optr = [&](int t) {
            int orow = rank * MSH + (t0 + t) * 16;
            if (LOCAL && t >= nown) {
                const int st = s0 + t - nown, sl = st / XT;
                orow = ((rank - 1 - sl) & (NR - 1)) * MSH + (MSH - XM) + (st - sl * XT) * 16;
            }
            return obase + (size_t)orow * (NSH * 4);
        };
        #pragma unroll 1
        for (int pass = 0;; ++pass) {
            asm volatile("s_barrier" ::: "memory");
            #pragma unroll
            for (int j = 0; j < 4; ++j) *(LDS_AS v4f*)pwr[j] = acc[j];
            barrier();
            {
                v2f p[8];
                #pragma unroll
                for (int k = 0; k < 8; ++k) p[k] = *(const LDS_AS v2f*)(prd + k * 4096);
                const v2f sum = ((p[0] + p[1]) + (p[2] + p[3])) + ((p[4] + p[5]) + (p[6] + p[7]));
                store8_sys(ovoff, (v2f){act(sum[0], scale), act(sum[1], scale)}, optr(pass ? nt - 1 : 0));
            }
            STAMP(16 + (pass ? nt - 1 : 0));
            if (pass) break;
            glds4(ldsw + 4096, xo[4], xo[5], xo[6], xo[7], xin);
            xin += 16 * KD;
            VMW(9);
            read_x(xr0 + 8192);
            glds4(ldsw + 8192, xo[0], xo[1], xo[2], xo[3], xin);
            glds4(ldsw + 12288, xo[4], xo[5], xo[6], xo[7], xin);

            unsigned boff = 0;
            const u8* op = myws + WS_SCR + idx * 32768 + rel * 256;
            #pragma unroll 1
            for (int i = 1; i < nt; ++i) {
                const bool more = i + 3 < nt;
                const unsigned long long mask = (more && !(ABL & 16)) ? ~0ull : 1ull;
                if (more) xin += 16 * KD;
                if (LOCAL && i + 3 == nown && more) {
                    xin = myws + WS_XST + (size_t)(call & 1u) * WS_XST_PAR + wave * 512 - 4096 + (size_t)s0 * (16 * KD);
                    const unsigned long long tw = __builtin_amdgcn_s_memrealtime();
                    while (true) {
                        unsigned f, tm;
                        asm volatile("global_load_dword %0, %2, %3 sc0 sc1\n\t"
                                     "global_load_dword %1, %4, %5 sc0 sc1\n\t"
                                     "s_waitcnt vmcnt(0)" : "=&v"(f), "=&v"(tm)
                                     : "v"((lane < 56 ? lane : 0) * 4), "s"(myws + WS_XFLAG), "v"(0u), "s"(myws + WS_TMO) : "memory");
                        if (__builtin_amdgcn_ballot_w64(f < call) == 0ull) break;
                        if (first(tm) == call || __builtin_amdgcn_s_memrealtime() - tw > DEADLINE_TICKS) {
                            if (lane == 0) store4_sys(0, call, myws + WS_TMO);
                            cinit = __builtin_nanf("");
                            break;
                        }
                        __builtin_amdgcn_s_sleep(4);
                    }
                    STAMP(8);
                }
                const bool sys = LOCAL && i + 3 >= nown;
                const unsigned lb = ldsw + boff, xr = xr0 + boff;
                const float ci = LOCAL ? cinit : 0.f;
                const v4f c0 = {ci, ci, ci, ci};
                v4f n[4];
                v2f p[8];
                #define MM(D, S, J, C) do { PIN(); asm volatile("" : "+v"(xf[S])); D = mma1(S, J, C); \
                                            asm volatile("" : "+v"(D)); PIN(); } while (0)
                #define REQ(O, H, A, B) do { if (ABL & 128) break; if (sys) glds2_sys<O>(lb + H, xo[A], xo[B], xin, mask); \
                                             else glds2<O>(lb + H, xo[A], xo[B], xin, mask); } while (0)
                MM(n[0], 0, 0, c0);
                if (!(ABL & 1)) asm volatile("s_barrier" ::: "memory");
                if (!(ABL & 2)) *(LDS_AS v4f*)pwr[0] = acc[0]; else asm volatile("" :: "v"(acc[0]));
                if (!(ABL & 2)) *(LDS_AS v4f*)pwr[1] = acc[1]; else asm volatile("" :: "v"(acc[1]));
                MM(n[1], 0, 1, c0);
                if (!(ABL & 2)) *(LDS_AS v4f*)pwr[2] = acc[2]; else asm volatile("" :: "v"(acc[2]));
                if (!(ABL & 2)) *(LDS_AS v4f*)pwr[3] = acc[3]; else asm volatile("" :: "v"(acc[3]));
                MM(n[2], 0, 2, c0);
                MM(n[3], 0, 3, c0);
                VMW(15);
                if (!(ABL & 8)) xf[0] = read_xs(xr, 0);
                MM(n[0], 1, 0, n[0]);
                MM(n[1], 1, 1, n[1]);
                if (ABL & 1) LGKM0(); else barrier();
                #pragma unroll
                for (int k = 0; k < 8; ++k) { if (ABL & 4) { p[k] = (v2f){1.f, 2.f}; asm volatile("" : "+v"(p[k])); } else p[k] = *(const LDS_AS v2f*)(prd + k * 4096); }
                REQ(0, 0, 0, 1);
                MM(n[2], 1, 2, n[2]);
                MM(n[3], 1, 3, n[3]);
                VMW(15);
                if (!(ABL & 8)) xf[1] = read_xs(xr, 1);
                MM(n[0], 2, 0, n[0]);
                const v2f sum = ((p[0] + p[1]) + (p[2] + p[3])) + ((p[4] + p[5]) + (p[6] + p[7]));
                MM(n[1], 2, 1, n[1]);
                LGKM0();
                REQ(2048, 0, 2, 3);
                const v2f o = (ABL & 256) ? p[0] : (v2f){act(sum[0], scale), act(sum[1], scale)};
                MM(n[2], 2, 2, n[2]);
                if (!(ABL & 32)) store8_sys(ovoff, o, op); else asm volatile("" :: "v"(o));
                op = optr(i);
                MM(n[3], 2, 3, n[3]);
                VMW(15);
                if (!(ABL & 8)) xf[2] = read_xs(xr, 2);
                MM(acc[0], 3, 0, n[0]);
                VMW(13);
                const v8i y = (ABL & 8) ? xf[3] : read_xs(xr, 3);
                MM(acc[1], 3, 1, n[1]);
                MM(acc[2], 3, 2, n[2]);
                LGKM0();
                REQ(0, 4096, 4, 5);
                MM(acc[3], 3, 3, n[3]);
                REQ(2048, 4096, 6, 7);
                xf[3] = y;
                boff ^= 8192u;
            }
        }
    };
    if (rel) tiles(std::false_type{}); else tiles(std::true_type{});
    STAMP(3); CSTAMP(12);

    VMW(0);
    STAMP(6);
    barrier();
    if (wave == 4) {
        if (lane == 0) {
            if (rel != 0) store4_sys(0, call, dws + WS_OFLAG + ((rel - 1) * 32 + idx) * 4);
            store4_sys(0, call, cntp);
        }
        STAMP(4);
        if (b == 0) {
            const unsigned fo = (lane < 56 ? lane : 0) * 16;
            const unsigned long long tw = __builtin_amdgcn_s_memrealtime();
            while (true) {
                v4u f;
                unsigned tm;
                asm volatile("global_load_dwordx4 %0, %2, %3 sc0 sc1\n\t"
                             "global_load_dword %1, %4, %3 offset:-2048 sc0 sc1\n\t"
                             "s_waitcnt vmcnt(0)" : "=&v"(f), "=&v"(tm) : "v"(fo), "s"(myws + WS_OFLAG), "v"(0u) : "memory");
                const bool miss = f[0] < call || f[1] < call || f[2] < call || f[3] < call;
                if (__builtin_amdgcn_ballot_w64(miss) == 0ull) break;
                if (first(tm) == call || __builtin_amdgcn_s_memrealtime() - tw > DEADLINE_TICKS) {
                    if (lane == 0) {
                        store4_sys(0, call, myws + WS_TMO);
                        store4_sys(0, 0x7fc00000u, aout);
                    }
                    break;
                }
                __builtin_amdgcn_s_sleep(1);
            }
            __builtin_amdgcn_fence(__ATOMIC_ACQUIRE, "");
            STAMP(5);
        }
    }
}
static_assert(WS_OFLAG - WS_TMO == 2048, "layout");

inline unsigned mkjob(int nown, int t0, int strip, int xg, int delay50, int nstg, int s0) {
    return (unsigned)nown | (unsigned)t0 << 5 | (unsigned)strip << 10 | (unsigned)xg << 13 | (unsigned)delay50 << 16 |
           (unsigned)nstg << 24 | (unsigned)s0 << 27;
}

#define DLY1 30
#define DLY2 80
#define DLY3 80
#define DLYL 50

unsigned build_jobs(Args& a) {
    const int dly[4] = {0, DLY1, DLY2, DLY3};
    unsigned first4 = 0;
    for (int idx = 0; idx < 32; ++idx) {
        const int gen = idx >> 3, sub = (idx >> 1) & 3, strip = 2 * gen + (idx & 1);
        {
            const int n = 32 - XT, lo = sub * n / 4, hi = (sub + 1) * n / 4;
            a.jobs[0][idx] = mkjob(hi - lo, lo, strip, gen ? (idx & 7) : 0, dly[gen], 0, 0);
            if (gen == 0) first4 |= (unsigned)lo << (5 * sub);
        }
        {
            const int n = 7 * XT, lo = sub * n / 4, hi = (sub + 1) * n / 4;
            a.jobs[1][idx] = mkjob(8, sub * 8, strip, idx & 7, DLYL, hi - lo, lo);
        }
    }
    return first4;
}

}

extern "C" void kernel_launch(int rank, int nranks, ncclComm_t comm,
                              hipStream_t stream,
                              void* const* d_in, const int* in_sizes, int n_in,
                              void* d_out, int out_size,
                              const int* in_dtypes, int out_dtype,
                              void* const* d_out_all,
                              void* const* d_ws_all, size_t ws_bytes) {
    (void)comm; (void)in_dtypes; (void)out_dtype;
    if (nranks != NR || n_in != 4 || (unsigned)rank >= (unsigned)NR || ws_bytes < WS_NEED ||
        in_sizes[0] != MSH * KD || in_sizes[1] != KD * NR * NSH || out_size != NR * MSH * NSH) {
        fprintf(stderr, "kernel_launch: unexpected problem shape\n");
        return;
    }
    Args a;
    memset(&a, 0, sizeof(a));
    const unsigned first4 = build_jobs(a);
    for (int r = 0; r < NR; ++r) {
        const int dest = (rank + r) % NR;
        Rec& c = a.rec[r];
        c.x = (const u8*)d_in[0];
        c.w = (const u8*)d_in[1] + dest * NSH;
        c.out = (u8*)(r ? d_out_all[dest] : d_out);
        c.dws = (u8*)d_ws_all[dest];
        c.myws = (u8*)d_ws_all[rank];
        c.sx = (const float*)d_in[2];
        c.sw = (const float*)d_in[3];
        c.rank = (unsigned)rank;
        c.first4 = first4;
    }
    hipLaunchKernelGGL(fused_kernel, dim3(NWG), dim3(NT), 0, stream, a);
    HIP_CHECK(hipGetLastError());
}
```

```cpp
#include <hip/hip_runtime.h>
#include <rccl/rccl.h>
#include <cstdio>
#include <cstring>

#define HIP_CHECK(expr) do { hipError_t _e = (expr); if (_e != hipSuccess) { \
    fprintf(stderr, "HIP error %s at %s:%d\n", hipGetErrorString(_e), __FILE__, __LINE__); \
    return; } } while (0)

namespace {

typedef int v2i __attribute__((ext_vector_type(2)));
typedef int v4i __attribute__((ext_vector_type(4)));
typedef int v8i __attribute__((ext_vector_type(8)));
typedef float v4f __attribute__((ext_vector_type(4)));
typedef unsigned v4u __attribute__((ext_vector_type(4)));
typedef unsigned v16u __attribute__((ext_vector_type(16)));
typedef unsigned char u8;
#define LDS_AS __attribute__((address_space(3)))

constexpr int NR = 8;
constexpr int NWG = 256;
constexpr int NT = 256;
constexpr int KD = 4096;
constexpr int MSH = 512;
constexpr int NSH = 512;
#define ABL 0
#define CODEPF 1
#define XPF 7
#define XM_ROWS 48
constexpr int XM = XM_ROWS;
constexpr int XT = XM / 16;
constexpr int XPW = XM / 8;
constexpr unsigned long long DEADLINE_TICKS = 100000000ull;

constexpr int LDS_BUF = 32768;
constexpr int LDS_BYTES = LDS_BUF + 4 * 32768;

constexpr size_t WS_CNT = 0;
constexpr size_t WS_TMO = 2048;
constexpr size_t WS_OFLAG = 4096;
constexpr size_t WS_XFLAG = 8192;
constexpr size_t WS_XST = 65536;
constexpr size_t WS_XST_PAR = (size_t)7 * 64 * KD;
constexpr size_t WS_DBG = WS_XST + 2 * WS_XST_PAR;
constexpr size_t WS_NEED = WS_DBG + 131072;

struct Rec {
    const u8* x;
    const u8* w;
    u8* out;
    u8* dws;
    u8* myws;
    const float* sx;
    const float* sw;
    unsigned rank;
    unsigned first4;
};
struct alignas(64) Args {
    Rec rec[NR];
    unsigned jobs[2][32];
};
static_assert(sizeof(Rec) == 64, "layout");

template <int O>
__device__ __forceinline__ void dma(unsigned v, v4u srd, unsigned s) {
    asm volatile("buffer_load_dwordx4 %0, %1, %2 offen offset:%3 lds" :: "v"(v), "s"(srd), "s"(s), "n"(O) : "memory");
}
template <int O>
__device__ __forceinline__ void dma_after(const v8i& f, unsigned v, v4u srd, unsigned s) {
    asm volatile("buffer_load_dwordx4 %0, %1, %2 offen offset:%3 lds" :: "v"(v), "s"(srd), "s"(s), "n"(O), "v"(f) : "memory");
}
__device__ __forceinline__ void set_m0(unsigned lds) {
    asm volatile("s_mov_b32 m0, %0\n\ts_nop 0" :: "s"(lds) : "memory", "m0");
}
__device__ __forceinline__ void dma4(unsigned lds, const unsigned (&v)[4], v4u srd, unsigned s) {
    set_m0(lds);
    dma<0>(v[0], srd, s); dma<1024>(v[1], srd, s); dma<2048>(v[2], srd, s); dma<3072>(v[3], srd, s);
}
__device__ __forceinline__ void glds1(unsigned lds, unsigned v0, const u8* base) {
    asm volatile("s_mov_b32 m0, %0\n\ts_nop 0\n\tglobal_load_lds_dwordx4 %1, %2" :: "s"(lds), "v"(v0), "s"(base) : "memory", "m0");
}
__device__ __forceinline__ v4u make_srd(const u8* p, unsigned bytes) {
    const unsigned long long a = (unsigned long long)p;
    return (v4u){(unsigned)a, (unsigned)(a >> 32) & 0xffffu, bytes, 0x00020000u};
}

#define VMW(n) asm volatile("s_waitcnt vmcnt(" #n ")" ::: "memory")
#define LGKM0() asm volatile("s_waitcnt lgkmcnt(0)" ::: "memory")
#define PIN() __builtin_amdgcn_sched_barrier(0)
#define KEEP(x) asm volatile("" : "+v"(x))

__device__ __forceinline__ void store16_sys(unsigned voff, v4f v, const void* base) {
    asm volatile("global_store_dwordx4 %0, %1, %2 sc0 sc1\n\ts_nop 1" :: "v"(voff), "v"(v), "s"(base) : "memory");
}
__device__ __forceinline__ void store4_sys(unsigned voff, unsigned v, const void* base) {
    asm volatile("global_store_dword %0, %1, %2 sc0 sc1" :: "v"(voff), "v"(v), "s"(base) : "memory");
}
__device__ __forceinline__ v4f load16(unsigned voff, const void* base) {
    v4f r;
    asm volatile("global_load_dwordx4 %0, %1, %2" : "=v"(r) : "v"(voff), "s"(base) : "memory");
    return r;
}
__device__ __forceinline__ void barrier() {
    asm volatile("s_waitcnt lgkmcnt(0)\n\ts_barrier" ::: "memory");
}
__device__ __forceinline__ unsigned first(unsigned v) { return (unsigned)__builtin_amdgcn_readfirstlane(v); }

__device__ __forceinline__ float act1(float& y, float acc, float s) {
    y = acc * s;
    float t = __builtin_amdgcn_fmed3f(y, -60.0f, 60.0f) * -1.44269504f;
    KEEP(y); KEEP(t);
    return t;
}
__device__ __forceinline__ float act2(float t) {
    float d = __builtin_amdgcn_exp2f(t) + 1.0f;
    KEEP(d);
    return d;
}
__device__ __forceinline__ float act3(float y, float d) {
    float o = y * __builtin_amdgcn_rcpf(d);
    KEEP(o);
    return o;
}

#define STAMP(i) do {} while (0)
#define CSTAMP(i) do {} while (0)

__device__ __forceinline__ u8* ptr(unsigned lo, unsigned hi) {
    return (u8*)(((unsigned long long)hi << 32) | lo);
}

__global__ __launch_bounds__(NT) void fused_kernel(const Args args) {
    __shared__ __attribute__((aligned(1024))) u8 smem[LDS_BYTES];
    const int b = blockIdx.x;
    const int rel = b & 7, idx = (b >> 3) & 31;
    v16u rec;
    unsigned job;
    unsigned long long t_start;
    asm volatile("s_load_dwordx16 %0, %3, %4\n\t"
                 "s_load_dword %1, %3, %5\n\t"
                 "s_memrealtime %2\n\t"
                 "s_waitcnt lgkmcnt(0)"
                 : "=&s"(rec), "=&s"(job), "=&s"(t_start)
                 : "s"(__builtin_amdgcn_kernarg_segment_ptr()), "s"(rel * 64), "s"(512 + (rel ? 0 : 128) + idx * 4) : "memory");
    const u8* const ax = ptr(rec[0], rec[1]);
    const u8* const aw = ptr(rec[2], rec[3]);
    u8* const aout = ptr(rec[4], rec[5]);
    u8* const dws = ptr(rec[6], rec[7]);
    u8* const myws = ptr(rec[8], rec[9]);
    const int rank = rec[14] & (NR - 1);
    const int tid = threadIdx.x;
    const int lane = tid & 63;
    const int wave = __builtin_amdgcn_readfirstlane(tid >> 6);
    const u8* const cntp = myws + WS_CNT + rel * 128 + idx * 4;
    const unsigned lds0 = (unsigned)(size_t)(LDS_AS u8*)smem;
    const unsigned ldsw = lds0 + LDS_BUF + wave * 32768;
    const unsigned ldsr = lds0 + wave * 4096;
    const int g = lane >> 4, i16 = lane & 15;
    unsigned call;
    float sclx, sclw;
    auto load_scalars = [&]() {
        asm volatile("s_load_dword %0, %3, 0x0\n\t"
                     "s_load_dword %1, %4, 0x0\n\t"
                     "s_load_dword %2, %5, 0x0\n\t"
                     "s_waitcnt lgkmcnt(0)"
                     : "=&s"(call), "=&s"(sclx), "=&s"(sclw)
                     : "s"(cntp), "s"(ptr(rec[10], rec[11])), "s"(ptr(rec[12], rec[13])) : "memory");
        call += 1u;
    };

    if (rel == 0) {
        if (wave == 0) __builtin_amdgcn_fence(__ATOMIC_ACQUIRE, "");
    } else if (idx >= 8) {
        if (idx == 8 && wave < 2) {
            const u8* pc;
            unsigned len;
            asm volatile("s_getpc_b64 %0\n.Lpc_here:\n\ts_mov_b32 %1, .Lpc_end-.Lpc_here" : "=s"(pc), "=s"(len));
            unsigned o = (wave * 64 + lane) * 128;
            if (o > len - 16) o = len - 16;
            glds1(ldsw + 8192, o & ~15u, (const u8*)((unsigned long long)pc & ~15ull));
        }
        auto help = [&]() {
            const int hw = (idx - 8) * 4 + wave;
            #pragma unroll
            for (int j = 0; j < 8; ++j) {
                const int id = j * 96 + hw;
                if (id < 256) {
                    const unsigned tl = (rec[15] >> (5 * (id >> 6))) & 31u;
                    glds1(ldsw + j * 1024, lane * 16, ax + (size_t)tl * (16 * KD) + (id & 63) * 1024);
                } else {
                    const int k = id - 256;
                    const int row = ((k >> 3) & 3) * 1024 + (k >> 5) * 64 + (k & 7) * 8;
                    glds1(ldsw + j * 1024, (lane >> 3) * KD + (lane & 7) * 16, aw + (size_t)row * KD);
                }
            }
        };
        if (idx >= 24) {
            v4f xv[XPW];
            const unsigned xoff = ((idx & 7) * 4 + wave) * (XPW * 1024) + lane * 16;
            #pragma unroll
            for (int i = 0; i < XPW; ++i) xv[i] = load16(xoff + i * 1024, ax + (size_t)(MSH - XM) * KD);
            help();
            load_scalars();
            const u8* dst = dws + WS_XST + (size_t)(call & 1u) * WS_XST_PAR + (size_t)(rel - 1) * (XM * KD);
            VMW(8);
            #pragma unroll
            for (int i = 0; i < XPW; ++i) {
                KEEP(xv[i]);
                store16_sys(xoff + i * 1024, xv[i], dst);
            }
            VMW(0);
            barrier();
            if (tid == 0) store4_sys(0, call, dws + WS_XFLAG + ((rel - 1) * 8 + (idx & 7)) * 4);
            STAMP(1);
        } else {
            help();
        }
        VMW(0);
        if (idx < 24) {
            const int hw = (idx - 8) * 4 + wave;
            #pragma unroll 1
            for (int j = 0; j < XPF * 4; ++j) {
                unsigned tl = ((rec[15] >> (5 * (j & 3))) & 31u) + 1 + (j >> 2);
                if (tl > 31) tl = 31;
                glds1(ldsw + (j & 7) * 1024, lane * 16, ax + (size_t)tl * (16 * KD) + hw * 1024);
                VMW(7);
            }
            VMW(0);
        }
    }

    int nown = job & 31u;
    if (nown < 4) nown = 4;
    int t0 = (job >> 5) & 31u;
    if (t0 + nown > 32) t0 = 32 - nown;
    int nstg = rel ? 0 : (job >> 24) & 7u;
    int s0 = job >> 27;
    if (s0 + nstg > 7 * XT) nstg = 0;
    const int nt = nown + nstg;
    const int strip = (job >> 10) & 7;
    const int xg = (job >> 13) & 7;
    const unsigned delay = ((job >> 16) & 255u) * 5u;

    unsigned wo[4];
    {
        const int row = lane >> 2, pos = lane & 3;
        #pragma unroll
        for (int p = 0; p < 4; ++p)
            wo[p] = (16 * p + row) * KD + ((pos ^ (((row >> 2) & 1) | ((p & 1) << 1))) << 4) - p * 1024;
    }
    unsigned xo[4];
    #pragma unroll
    for (int p = 0; p < 4; ++p) {
        const int m = 8 * (p & 1) + (lane >> 3);
        const int cs = (lane & 7) ^ (m >> 1);
        xo[p] = m * KD + ((((p >> 1) ^ (xg & 1)) << 7) + (cs << 4)) - p * 1024 + 4096;
    }
    unsigned xq[4];
    #pragma unroll
    for (int q = 0; q < 4; ++q) xq[q] = (q ^ (xg >> 1)) << 8;
    const v4u wsrd = make_srd(aw + (size_t)(wave * 1024) * KD + strip * 64, 1024 * KD);
    constexpr unsigned XWIN = 17 * KD + 4096;
    const u8* xin = ax + wave * 1024 - 4096 + (size_t)t0 * (16 * KD);
    v4u xsrd = make_srd(xin, XWIN);
    auto next_x = [&]() { xin += 16 * KD; xsrd = make_srd(xin, XWIN); };
    auto issue_w = [&](int h, unsigned unit) { dma4(unit, wo, wsrd, (unsigned)(h ^ (xg << 1)) * (64 * KD)); };

    if (delay) while ((unsigned)__builtin_amdgcn_s_memrealtime() - (unsigned)t_start < delay) __builtin_amdgcn_s_sleep(1);
    STAMP(7); CSTAMP(10);

    #pragma unroll
    for (int q = 0; q < 4; ++q) dma4(ldsw + q * 4096, xo, xsrd, xq[q]);
    next_x();
    issue_w(0, ldsw + 16384);
    issue_w(1, ldsw + 20480);
    asm volatile("s_barrier" ::: "memory");
    issue_w(2, ldsw + 24576);
    issue_w(3, ldsw + 28672);
    issue_w(4, ldsr);
    issue_w(5, ldsr + 16384);
    PIN();
    load_scalars();
    float scale = sclx * sclw;
    const LDS_AS u8* wrw[4];
    const LDS_AS u8* wrr[4];
    {
        const int q = i16 >> 1, p = i16 & 1;
        const int f = ((q >> 2) & 1) | ((g & 1) << 1);
        #pragma unroll
        for (int j = 0; j < 4; ++j) {
            const unsigned o = g * 1024 + q * 64 + ((j ^ f) << 4) + 8 * p;
            wrw[j] = (const LDS_AS u8*)(size_t)(ldsw + o);
            wrr[j] = (const LDS_AS u8*)(size_t)(ldsr + o);
        }
    }
    const unsigned xr0 = ldsw + i16 * 128 + ((g ^ (i16 >> 1)) << 4);
    const int om = 4 * wave + g;
    const unsigned ovoff = om * (NSH * 4) + i16 * 16;
    const unsigned prd0 = lds0 + om * 256 + ((i16 ^ om) << 4);
    unsigned pwr0[4];
    #pragma unroll
    for (int j = 0; j < 4; ++j) pwr0[j] = ldsr + i16 * 256 + (((4 * j + g) ^ i16) << 4);
    v8i wf[8][4], xf[4];
    v4f acc[4];
    auto read_xs = [&](unsigned ad, int s) {
        const v4i lo = *(const LDS_AS v4i*)((const LDS_AS u8*)(size_t)ad + s * 2048);
        const v4i hi = *(const LDS_AS v4i*)((const LDS_AS u8*)(size_t)(ad ^ 64u) + s * 2048);
        return (v8i){lo[0], lo[1], lo[2], lo[3], hi[0], hi[1], hi[2], hi[3]};
    };
    auto read_w = [&](int h, const LDS_AS u8* const* base, int off) {
        #pragma unroll
        for (int j = 0; j < 4; ++j) {
            #pragma unroll
            for (int r = 0; r < 2; ++r) {
                const v2i t = __builtin_amdgcn_ds_read_tr8_b64_v2i32((LDS_AS v2i*)(base[j] + off + r * 512));
                wf[h >> 1][j][(h & 1) * 4 + 2 * r] = t[0];
                wf[h >> 1][j][(h & 1) * 4 + 2 * r + 1] = t[1];
            }
        }
        LGKM0();
        STAMP(42 + h);
        if (h & 1) {
            #pragma unroll
            for (int j = 0; j < 4; ++j) {
                if (h < 12) asm volatile("" : "+a"(wf[h >> 1][j]));
                else asm volatile("" : "+v"(wf[h >> 1][j]));
            }
        }
    };
    const v4f zero = {0.f, 0.f, 0.f, 0.f};
    {
        v8i x0[8];
        auto mma = [&](int s) {
            PIN();
            #pragma unroll
            for (int j = 0; j < 4; ++j)
                acc[j] = __builtin_amdgcn_mfma_scale_f32_16x16x128_f8f6f4(wf[s][j], x0[s], s ? acc[j] : zero, 0, 0, 0, 0, 0, 0);
            PIN();
        };
        STAMP(40);
        VMW(24);
        STAMP(41);
        #pragma unroll
        for (int s = 0; s < 8; ++s) x0[s] = read_xs(xr0, s);
        LGKM0();
        issue_w(6, ldsw);
        issue_w(7, ldsw + 4096);
        issue_w(8, ldsw + 8192);
        issue_w(9, ldsw + 12288);
        VMW(36); read_w(0, wrw, 16384);   issue_w(10, ldsw + 16384);
        VMW(36); read_w(1, wrw, 20480);   issue_w(11, ldsw + 20480);              mma(0);
        VMW(36); read_w(2, wrw, 24576);   issue_w(12, ldsw + 24576);
        VMW(36); read_w(3, wrw, 28672);   issue_w(13, ldsw + 28672);              mma(1);
        VMW(36); read_w(4, wrr, 0);       issue_w(14, ldsr);
        VMW(36); read_w(5, wrr, 16384);   issue_w(15, ldsr + 16384);              mma(2);
        asm volatile("s_barrier" ::: "memory");
        VMW(36); read_w(6, wrw, 0);       dma4(ldsw, xo, xsrd, xq[0]);
        VMW(36); read_w(7, wrw, 4096);    dma4(ldsw + 4096, xo, xsrd, xq[1]);     mma(3);
        VMW(36); read_w(8, wrw, 8192);    dma4(ldsw + 8192, xo, xsrd, xq[2]);
        VMW(36); read_w(9, wrw, 12288);   dma4(ldsw + 12288, xo, xsrd, xq[3]);    mma(4);
        next_x();
        VMW(36); read_w(10, wrw, 16384);  dma4(ldsw + 16384, xo, xsrd, xq[0]);
        VMW(36); read_w(11, wrw, 20480);  dma4(ldsw + 20480, xo, xsrd, xq[1]);    mma(5);
        VMW(36); read_w(12, wrw, 24576);  dma4(ldsw + 24576, xo, xsrd, xq[2]);
        VMW(36); read_w(13, wrw, 28672);  dma4(ldsw + 28672, xo, xsrd, xq[3]);    mma(6);
        next_x();
        VMW(36); read_w(14, wrr, 0);
        VMW(32); read_w(15, wrr, 16384);                                          mma(7);
        STAMP(2); CSTAMP(11);
        VMW(30); xf[0] = read_xs(xr0, 0);
        VMW(28); xf[1] = read_xs(xr0, 1);
        set_m0(ldsw);
        dma_after<0>(xf[0], xo[0], xsrd, xq[0]);
        dma<1024>(xo[1], xsrd, xq[0]);
    }

    const u8* const obase = aout + strip * 256;
    auto optr = [&](int t) {
        int orow = rank * MSH + (t0 + t) * 16;
        if (t >= nown) {
            const int st = s0 + t - nown, sl = st / XT;
            orow = ((rank - 1 - sl) & (NR - 1)) * MSH + (MSH - XM) + (st - sl * XT) * 16;
        }
        return obase + (size_t)orow * (NSH * 4);
    };
    auto pw = [&](int j, unsigned bo) { if (ABL & 2) asm volatile("" :: "a"(acc[j])); else *(LDS_AS v4f*)(size_t)(pwr0[j] + bo) = acc[j]; };
    auto pr = [&](int k, unsigned bo) { return *(const LDS_AS v4f*)((const LDS_AS u8*)(size_t)(prd0 + bo) + k * 4096); };
    auto sum4 = [&](const v4f (&p)[4], int e) {
        float a = p[0][e] + p[1][e], c = p[2][e] + p[3][e];
        KEEP(a); KEEP(c);
        float s = a + c;
        KEEP(s);
        return s;
    };

    unsigned boff = 0;
    #pragma unroll 1
    for (int i = 1; i < nt; ++i) {
        const unsigned lb = ldsw + boff, xr = xr0 + boff, xn = xr0 + (boff ^ 16384u);
        v4f n[4], p[4];
        float y[4], t[4];
        #define MM(D, S, J, C) do { PIN(); KEEP(xf[(S) & 3]); \
            D = __builtin_amdgcn_mfma_scale_f32_16x16x128_f8f6f4(wf[S][J], xf[(S) & 3], C, 0, 0, 0, 0, 0, 0); \
            asm volatile("" : "+a"(D)); PIN(); } while (0)
        #define RD(S) do { VMW(28); xf[((S) + 2) & 3] = read_xs((S) < 6 ? xr : xn, ((S) + 2) & 7); } while (0)
        #define RQA(S) dma_after<(((S) + 1) & 1) * 2048>(xf[((S) + 1) & 3], xo[(((S) + 1) & 1) * 2], xsrd, xq[(((S) + 1) >> 1) & 3])
        #define RQB(S) dma<(((S) + 1) & 1) * 2048 + 1024>(xo[(((S) + 1) & 1) * 2 + 1], xsrd, xq[(((S) + 1) >> 1) & 3])
        MM(n[0], 0, 0, zero);  RD(0);  pw(0, boff);
        MM(n[1], 0, 1, zero);  RQA(0); pw(1, boff);
        MM(n[2], 0, 2, zero);          pw(2, boff);
        MM(n[3], 0, 3, zero);  RQB(0); pw(3, boff);
        MM(n[0], 1, 0, n[0]);  RD(1);  set_m0(lb + 4096);
        MM(n[1], 1, 1, n[1]);  RQA(1);
        const u8* const op = optr(i - 1);
        MM(n[2], 1, 2, n[2]);
        MM(n[3], 1, 3, n[3]);  RQB(1);
        MM(n[0], 2, 0, n[0]);  RD(2);
        MM(n[1], 2, 1, n[1]);  RQA(2);
        if (!(ABL & 1)) asm volatile("s_waitcnt lgkmcnt(2)\n\ts_barrier" ::: "memory");
        p[0] = pr(0, boff); p[1] = pr(1, boff);
        MM(n[2], 2, 2, n[2]);
        p[2] = pr(2, boff); p[3] = pr(3, boff);
        MM(n[3], 2, 3, n[3]);  RQB(2);
        MM(n[0], 3, 0, n[0]);  RD(3);  set_m0(lb + 8192);
        MM(n[1], 3, 1, n[1]);  RQA(3);
        MM(n[2], 3, 2, n[2]);          y[0] = sum4(p, 0);
        MM(n[3], 3, 3, n[3]);  RQB(3); y[1] = sum4(p, 1);
        MM(n[0], 4, 0, n[0]);  RD(4);  t[0] = act1(y[0], y[0], scale);
        MM(n[1], 4, 1, n[1]);  RQA(4); y[2] = sum4(p, 2);                  t[0] = act2(t[0]);
        MM(n[2], 4, 2, n[2]);          t[1] = act1(y[1], y[1], scale);     y[0] = act3(y[0], t[0]);
        MM(n[3], 4, 3, n[3]);  RQB(4); y[3] = sum4(p, 3);                  t[1] = act2(t[1]);
        MM(n[0], 5, 0, n[0]);  RD(5);  set_m0(lb + 12288);                 y[1] = act3(y[1], t[1]);
        MM(n[1], 5, 1, n[1]);  RQA(5); t[2] = act1(y[2], y[2], scale);
        MM(n[2], 5, 2, n[2]);          t[3] = act1(y[3], y[3], scale);     t[2] = act2(t[2]);
        MM(n[3], 5, 3, n[3]);  RQB(5); t[3] = act2(t[3]);                  y[2] = act3(y[2], t[2]);
        MM(n[0], 6, 0, n[0]);  RD(6);  y[3] = act3(y[3], t[3]);
        MM(n[1], 6, 1, n[1]);  RQA(6);
        MM(n[2], 6, 2, n[2]);  if (!(ABL & 32)) store16_sys(ovoff, (v4f){y[0], y[1], y[2], y[3]}, op); else asm volatile("" :: "v"(y[0]), "v"(y[1]), "v"(y[2]), "v"(y[3]), "s"(op));
        MM(n[3], 6, 3, n[3]);  RQB(6);
        boff ^= 16384u;
        next_x();
        if (i + 3 >= nt || (ABL & 128)) xsrd[2] = 0;
        MM(acc[0], 7, 0, n[0]);  RD(7);  set_m0(ldsw + boff);
        if (i + 3 == nown && nstg) {
            xin = myws + WS_XST + (size_t)(call & 1u) * WS_XST_PAR + wave * 1024 - 4096 + (size_t)s0 * (16 * KD);
            xsrd = make_srd(xin, XWIN);
            const unsigned long long tw = __builtin_amdgcn_s_memrealtime();
            while (true) {
                unsigned f, tm;
                asm volatile("global_load_dword %0, %2, %3 sc0 sc1\n\t"
                             "global_load_dword %1, %4, %5 sc0 sc1\n\t"
                             "s_waitcnt vmcnt(0)" : "=&v"(f), "=&v"(tm)
                             : "v"((lane < 56 ? lane : 0) * 4), "s"(myws + WS_XFLAG), "v"(0u), "s"(myws + WS_TMO) : "memory");
                if (__builtin_amdgcn_ballot_w64(f < call) == 0ull) break;
                if (first(tm) == call || __builtin_amdgcn_s_memrealtime() - tw > DEADLINE_TICKS) {
                    if (lane == 0) store4_sys(0, call, myws + WS_TMO);
                    scale = __builtin_nanf("");
                    break;
                }
                __builtin_amdgcn_s_sleep(4);
            }
            STAMP(8);
        }
        MM(acc[1], 7, 1, n[1]);  RQA(7);
        MM(acc[2], 7, 2, n[2]);
        MM(acc[3], 7, 3, n[3]);  RQB(7);
    }
    {
        #pragma unroll
        for (int j = 0; j < 4; ++j) pw(j, boff);
        barrier();
        v4f p[4];
        #pragma unroll
        for (int k = 0; k < 4; ++k) p[k] = pr(k, boff);
        float y[4];
        #pragma unroll
        for (int e = 0; e < 4; ++e) {
            const float t = act1(y[e], sum4(p, e), scale);
            y[e] = act3(y[e], act2(t));
        }
        store16_sys(ovoff, (v4f){y[0], y[1], y[2], y[3]}, optr(nt - 1));
        STAMP(15 + nt);
    }
    STAMP(3); CSTAMP(12);

    VMW(0);
    STAMP(6);
    barrier();
    asm volatile(".Lpc_end:");
    if (wave == 2) {
        if (lane == 0) {
            if (rel != 0) store4_sys(0, call, dws + WS_OFLAG + ((rel - 1) * 32 + idx) * 4);
            store4_sys(0, call, cntp);
        }
        STAMP(4);
        if (b == 0) {
            const unsigned fo = (lane < 56 ? lane : 0) * 16;
            const unsigned long long tw = __builtin_amdgcn_s_memrealtime();
            while (true) {
                v4u f;
                unsigned tm;
                asm volatile("global_load_dwordx4 %0, %2, %3 sc0 sc1\n\t"
                             "global_load_dword %1, %4, %3 offset:-2048 sc0 sc1\n\t"
                             "s_waitcnt vmcnt(0)" : "=&v"(f), "=&v"(tm) : "v"(fo), "s"(myws + WS_OFLAG), "v"(0u) : "memory");
                const bool miss = f[0] < call || f[1] < call || f[2] < call || f[3] < call;
                if (__builtin_amdgcn_ballot_w64(miss) == 0ull) break;
                if (first(tm) == call || __builtin_amdgcn_s_memrealtime() - tw > DEADLINE_TICKS) {
                    if (lane == 0) {
                        store4_sys(0, call, myws + WS_TMO);
                        store4_sys(0, 0x7fc00000u, aout);
                    }
                    break;
                }
                __builtin_amdgcn_s_sleep(1);
            }
            __builtin_amdgcn_fence(__ATOMIC_ACQUIRE, "");
            STAMP(5);
        }
    }
}
static_assert(WS_OFLAG - WS_TMO == 2048, "layout");

inline unsigned mkjob(int nown, int t0, int strip, int xg, int delay50, int nstg, int s0) {
    return (unsigned)nown | (unsigned)t0 << 5 | (unsigned)strip << 10 | (unsigned)xg << 13 | (unsigned)delay50 << 16 |
           (unsigned)nstg << 24 | (unsigned)s0 << 27;
}

#define DLY1 30
#define DLY2 80
#define DLY3 80
#define DLYL 50

unsigned build_jobs(Args& a) {
    const int dly[4] = {0, DLY1, DLY2, DLY3};
    unsigned first4 = 0;
    for (int idx = 0; idx < 32; ++idx) {
        const int gen = idx >> 3, sub = (idx >> 1) & 3, strip = 2 * gen + (idx & 1);
        {
            const int n = 32 - XT, lo = sub * n / 4, hi = (sub + 1) * n / 4;
            a.jobs[0][idx] = mkjob(hi - lo, lo, strip, gen ? (idx & 7) : 0, dly[gen], 0, 0);
            if (gen == 0) first4 |= (unsigned)lo << (5 * sub);
        }
        {
            const int n = 7 * XT, lo = sub * n / 4, hi = (sub + 1) * n / 4;
            a.jobs[1][idx] = mkjob(8, sub * 8, strip, idx & 7, DLYL, hi - lo, lo);
        }
    }
    return first4;
}

}

extern "C" void kernel_launch(int rank, int nranks, ncclComm_t comm,
                              hipStream_t stream,
                              void* const* d_in, const int* in_sizes, int n_in,
                              void* d_out, int out_size,
                              const int* in_dtypes, int out_dtype,
                              void* const* d_out_all,
                              void* const* d_ws_all, size_t ws_bytes) {
    (void)comm; (void)in_dtypes; (void)out_dtype;
    if (nranks != NR || n_in != 4 || (unsigned)rank >= (unsigned)NR || ws_bytes < WS_NEED ||
        in_sizes[0] != MSH * KD || in_sizes[1] != KD * NR * NSH || out_size != NR * MSH * NSH) {
        fprintf(stderr, "kernel_launch: unexpected problem shape\n");
        return;
    }
    Args a;
    memset(&a, 0, sizeof(a));
    const unsigned first4 = build_jobs(a);
    for (int r = 0; r < NR; ++r) {
        const int dest = (rank + r) % NR;
        Rec& c = a.rec[r];
        c.x = (const u8*)d_in[0];
        c.w = (const u8*)d_in[1] + dest * NSH;
        c.out = (u8*)(r ? d_out_all[dest] : d_out);
        c.dws = (u8*)d_ws_all[dest];
        c.myws = (u8*)d_ws_all[rank];
        c.sx = (const float*)d_in[2];
        c.sw = (const float*)d_in[3];
        c.rank = (unsigned)rank;
        c.first4 = first4;
    }
    hipLaunchKernelGGL(fused_kernel, dim3(NWG), dim3(NT), 0, stream, a);
    HIP_CHECK(hipGetLastError());
}
```

```cpp
#include <hip/hip_runtime.h>
#include <rccl/rccl.h>
#include <cstdio>
#include <cstring>

#define HIP_CHECK(expr) do { hipError_t _e = (expr); if (_e != hipSuccess) { \
    fprintf(stderr, "HIP error %s at %s:%d\n", hipGetErrorString(_e), __FILE__, __LINE__); \
    return; } } while (0)

namespace {

typedef int v2i __attribute__((ext_vector_type(2)));
typedef int v4i __attribute__((ext_vector_type(4)));
typedef int v8i __attribute__((ext_vector_type(8)));
typedef float v4f __attribute__((ext_vector_type(4)));
typedef unsigned v4u __attribute__((ext_vector_type(4)));
typedef unsigned v16u __attribute__((ext_vector_type(16)));
typedef unsigned char u8;
#define LDS_AS __attribute__((address_space(3)))

constexpr int NR = 8;
constexpr int NWG = 256;
constexpr int NT = 256;
constexpr int KD = 4096;
constexpr int MSH = 512;
constexpr int NSH = 512;
#define CODEPF 1
#define XPF 2
#define XPD 23
#define DLYP 0
#define DLYW 0
#define XGA 2
#define ST0 2
#define XM_ROWS 48
constexpr int XM = XM_ROWS;
constexpr int XT = XM / 16;
constexpr int XPW = XM / 8;
constexpr unsigned long long DEADLINE_TICKS = 100000000ull;

constexpr int LDS_BUF = 32768;
constexpr int LDS_BYTES = LDS_BUF + 4 * 32768;

constexpr size_t WS_CNT = 0;
constexpr size_t WS_TMO = 2048;
constexpr size_t WS_OFLAG = 4096;
constexpr size_t WS_XFLAG = 8192;
constexpr size_t WS_XST = 65536;
constexpr size_t WS_XST_PAR = (size_t)7 * 64 * KD;
constexpr size_t WS_DBG = WS_XST + 2 * WS_XST_PAR;
constexpr size_t WS_NEED = WS_DBG + 131072;

struct Rec {
    const u8* x;
    const u8* w;
    u8* out;
    u8* dws;
    u8* myws;
    const float* sx;
    const float* sw;
    unsigned rank;
    unsigned first4;
};
struct alignas(64) Args {
    Rec rec[NR];
    unsigned jobs[2][32];
};
static_assert(sizeof(Rec) == 64, "layout");

template <int O>
__device__ __forceinline__ void dma(unsigned v, v4u srd, unsigned s) {
    asm volatile("buffer_load_dwordx4 %0, %1, %2 offen offset:%3 lds" :: "v"(v), "s"(srd), "s"(s), "n"(O) : "memory");
}
template <int O>
__device__ __forceinline__ void dma_after(const v8i& f, unsigned v, v4u srd, unsigned s) {
    asm volatile("buffer_load_dwordx4 %0, %1, %2 offen offset:%3 lds" :: "v"(v), "s"(srd), "s"(s), "n"(O), "v"(f) : "memory");
}
__device__ __forceinline__ void set_m0(unsigned lds) {
    asm volatile("s_mov_b32 m0, %0\n\ts_nop 0" :: "s"(lds) : "memory", "m0");
}
__device__ __forceinline__ void dma4(unsigned lds, const unsigned (&v)[4], v4u srd, unsigned s) {
    set_m0(lds);
    dma<0>(v[0], srd, s); dma<1024>(v[1], srd, s); dma<2048>(v[2], srd, s); dma<3072>(v[3], srd, s);
}
__device__ __forceinline__ void glds1(unsigned lds, unsigned v0, const u8* base) {
    asm volatile("s_mov_b32 m0, %0\n\ts_nop 0\n\tglobal_load_lds_dwordx4 %1, %2" :: "s"(lds), "v"(v0), "s"(base) : "memory", "m0");
}
__device__ __forceinline__ v4u make_srd(const u8* p, unsigned bytes) {
    const unsigned long long a = (unsigned long long)p;
    return (v4u){(unsigned)a, (unsigned)(a >> 32) & 0xffffu, bytes, 0x00020000u};
}

#define VMW(n) asm volatile("s_waitcnt vmcnt(" #n ")" ::: "memory")
#define LGKM0() asm volatile("s_waitcnt lgkmcnt(0)" ::: "memory")
#define PIN() __builtin_amdgcn_sched_barrier(0)
#define KEEP(x) asm volatile("" : "+v"(x))

__device__ __forceinline__ void store16_sys(unsigned voff, v4f v, const void* base) {
    asm volatile("global_store_dwordx4 %0, %1, %2 sc0 sc1\n\ts_nop 1" :: "v"(voff), "v"(v), "s"(base) : "memory");
}
__device__ __forceinline__ void store4_sys(unsigned voff, unsigned v, const void* base) {
    asm volatile("global_store_dword %0, %1, %2 sc0 sc1" :: "v"(voff), "v"(v), "s"(base) : "memory");
}
__device__ __forceinline__ v4f load16(unsigned voff, const void* base) {
    v4f r;
    asm volatile("global_load_dwordx4 %0, %1, %2" : "=v"(r) : "v"(voff), "s"(base) : "memory");
    return r;
}
__device__ __forceinline__ void barrier() {
    asm volatile("s_waitcnt lgkmcnt(0)\n\ts_barrier" ::: "memory");
}
__device__ __forceinline__ unsigned first(unsigned v) { return (unsigned)__builtin_amdgcn_readfirstlane(v); }

__device__ __forceinline__ float act1(float& y, float acc, float s) {
    y = acc * s;
    float t = __builtin_amdgcn_fmed3f(y, -60.0f, 60.0f) * -1.44269504f;
    KEEP(y); KEEP(t);
    return t;
}
__device__ __forceinline__ float act2(float t) {
    float d = __builtin_amdgcn_exp2f(t) + 1.0f;
    KEEP(d);
    return d;
}
__device__ __forceinline__ float act3(float y, float d) {
    float o = y * __builtin_amdgcn_rcpf(d);
    KEEP(o);
    return o;
}

#define STAMP(i) do {} while (0)
#define CSTAMP(i) do {} while (0)

__device__ __forceinline__ u8* ptr(unsigned lo, unsigned hi) {
    return (u8*)(((unsigned long long)hi << 32) | lo);
}

__global__ __launch_bounds__(NT) void fused_kernel(const Args args) {
    __shared__ __attribute__((aligned(1024))) u8 smem[LDS_BYTES];
    const int b = blockIdx.x;
    const int rel = b & 7, idx = (b >> 3) & 31;
    v16u rec;
    unsigned job;
    unsigned long long t_start;
    asm volatile("s_load_dwordx16 %0, %3, %4\n\t"
                 "s_load_dword %1, %3, %5\n\t"
                 "s_memrealtime %2\n\t"
                 "s_waitcnt lgkmcnt(0)"
                 : "=&s"(rec), "=&s"(job), "=&s"(t_start)
                 : "s"(__builtin_amdgcn_kernarg_segment_ptr()), "s"(rel * 64), "s"(512 + (rel ? 0 : 128) + idx * 4) : "memory");
    const u8* const ax = ptr(rec[0], rec[1]);
    const u8* const aw = ptr(rec[2], rec[3]);
    u8* const aout = ptr(rec[4], rec[5]);
    u8* const dws = ptr(rec[6], rec[7]);
    u8* const myws = ptr(rec[8], rec[9]);
    const int rank = rec[14] & (NR - 1);
    const int tid = threadIdx.x;
    const int lane = tid & 63;
    const int wave = __builtin_amdgcn_readfirstlane(tid >> 6);
    const u8* const cntp = myws + WS_CNT + rel * 128 + idx * 4;
    const unsigned lds0 = (unsigned)(size_t)(LDS_AS u8*)smem;
    const unsigned ldsw = lds0 + LDS_BUF + wave * 32768;
    const unsigned ldsr = lds0 + wave * 4096;
    const int g = lane >> 4, i16 = lane & 15;
    unsigned cnt;
    float sclx, sclw;
    asm volatile("s_load_dword %0, %3, 0x0\n\t"
                 "s_load_dword %1, %4, 0x0\n\t"
                 "s_load_dword %2, %5, 0x0"
                 : "=&s"(cnt), "=&s"(sclx), "=&s"(sclw)
                 : "s"(cntp), "s"(ptr(rec[10], rec[11])), "s"(ptr(rec[12], rec[13])) : "memory");
    auto call_no = [&]() {
        unsigned c;
        asm volatile("s_waitcnt lgkmcnt(0)\n\ts_add_u32 %0, %1, 1" : "=s"(c) : "s"(cnt) : "memory", "scc");
        return c;
    };

    auto wait_until = [&](unsigned ticks) {
        while ((unsigned)__builtin_amdgcn_s_memrealtime() - (unsigned)t_start < ticks) __builtin_amdgcn_s_sleep(1);
    };
    const bool staged = rel == 0 && (job >> 24 & 1u);
    if (staged && wave == 0) __builtin_amdgcn_fence(__ATOMIC_ACQUIRE, "");
    if (idx >= 16) {
        const int hw = (idx - 16) * 4 + wave;
        if (idx == 16 && wave >= 2) {
            const u8* pc;
            unsigned len;
            asm volatile("s_getpc_b64 %0\n.Lpc_here:\n\ts_mov_b32 %1, .Lpc_end-.Lpc_here" : "=s"(pc), "=s"(len));
            unsigned o = ((wave - 2) * 64 + lane) * 128;
            if (o > len - 16) o = len - 16;
            glds1(ldsw + 31 * 1024, o & ~15u, (const u8*)((unsigned long long)pc & ~15ull));
        }
        auto help_x = [&](int n) {
            #pragma unroll
            for (int j = 0; j < n; ++j) {
                const unsigned tl = (rec[15] >> (5 * j)) & 31u;
                glds1(ldsw + j * 1024, lane * 16, ax + (size_t)tl * (16 * KD) + hw * 1024);
            }
        };
        auto help_w = [&]() {
            const unsigned q = (lane >> 3) & 1;
            #pragma unroll
            for (int j = 0; j < 16; ++j)
                glds1(ldsw + 4096 + j * 1024, (((j >> 1 & 3) | (j & 1) << 2 | (j & 8)) ^ (q << 3)) * (64 * KD) + (lane >> 4) * KD + (lane & 15) * 16,
                      aw + (size_t)((hw >> 4) * 1024 + (hw & 15) * 4) * KD);
        };
        if (rel == 0) {
            if (DLYP) wait_until(DLYP * 5);
            help_x(2);
            const unsigned q = (lane >> 3) & 3;
            #pragma unroll 1
            for (int j = 0; j < 32; ++j)
                glds1(ldsw + j * 1024, ((j >> 1) ^ (q << 2)) * (64 * KD) + (lane >> 5) * KD + (lane & 31) * 16,
                      aw + (size_t)((hw >> 4) * 1024 + (hw & 15) * 4 + (j & 1) * 2) * KD);
            VMW(0);
            STAMP(9);
        } else if (idx >= 24) {
            v4f xv[XPW];
            const unsigned xoff = ((idx & 7) * 4 + wave) * (XPW * 1024) + lane * 16;
            #pragma unroll
            for (int i = 0; i < XPW; ++i) xv[i] = load16(xoff + i * 1024, ax + (size_t)(MSH - XM) * KD);
            help_x(4);
            help_w();
            const unsigned call = call_no();
            const u8* dst = dws + WS_XST + (size_t)(call & 1u) * WS_XST_PAR + (size_t)(rel - 1) * (XM * KD);
            VMW(20);
            STAMP(9);
            #pragma unroll
            for (int i = 0; i < XPW; ++i) {
                KEEP(xv[i]);
                store16_sys(xoff + i * 1024, xv[i], dst);
            }
            VMW(0);
            barrier();
            if (tid == 0) store4_sys(0, call, dws + WS_XFLAG + ((rel - 1) * 8 + (idx & 7)) * 4);
            STAMP(1);
        } else {
            help_x(4);
            help_w();
            VMW(8);
            STAMP(9);
            #pragma unroll 1
            for (int j = 0; j < XPF * 8; ++j) {
                unsigned tl = ((rec[15] >> (5 * ((j >> 1) & 3))) & 31u) + 1 + (j >> 3);
                if (tl > 31) tl = 31;
                glds1(ldsw + (j & 31) * 1024, lane * 16, ax + (size_t)tl * (16 * KD) + (hw * 2 + (j & 1)) * 1024);
                asm volatile("s_waitcnt vmcnt(%0)" :: "n"(XPD) : "memory");
            }
            #pragma unroll 1
            for (int j = 0; j < 32; ++j) {
                glds1(ldsw + j * 1024, (lane >> 4) * KD + (lane & 15) * 16, aw + (size_t)(j * 128 + hw * 4) * KD + 256);
                asm volatile("s_waitcnt vmcnt(%0)" :: "n"(XPD) : "memory");
            }
            VMW(0);
            STAMP(1);
        }
    }

    int nt = job & 31u;
    if (nt < 5) nt = 5;
    int t0 = (job >> 5) & 31u;
    if (t0 + nt > (staged ? 7 * XT : 32)) t0 = 0;
    if (nt > 7 * XT) nt = staged ? 7 * XT : nt;
    const int strip = (job >> 10) & 7;
    const int xg = (job >> 13) & 7;
    const unsigned delay = ((job >> 16) & 255u) * 5u;

    unsigned wo[4];
    {
        const int row = lane >> 2, pos = lane & 3;
        #pragma unroll
        for (int p = 0; p < 4; ++p)
            wo[p] = (16 * p + row) * KD + ((pos ^ (((row >> 2) & 1) | ((p & 1) << 1))) << 4) - p * 1024;
    }
    unsigned xo[4];
    #pragma unroll
    for (int p = 0; p < 4; ++p) {
        const int m = 8 * (p & 1) + (lane >> 3);
        const int cs = (lane & 7) ^ (m >> 1);
        xo[p] = m * KD + ((((p >> 1) ^ (xg & 1)) << 7) + (cs << 4)) - p * 1024 + 4096;
    }
    unsigned xq[4];
    #pragma unroll
    for (int q = 0; q < 4; ++q) xq[q] = (q ^ (xg >> 1)) << 8;
    const v4u wsrd = make_srd(aw + (size_t)(wave * 1024) * KD + strip * 64, 1024 * KD);
    constexpr unsigned XWIN = 17 * KD + 4096;
    const u8* xin = ax;
    unsigned orow = rank * MSH + t0 * 16;
    int ocnt = 32;
    float poison = 0.f;
    if (staged) {
        const unsigned call = call_no();
        xin = myws + WS_XST + (size_t)(call & 1u) * WS_XST_PAR;
        const int sl = t0 / XT;
        ocnt = XT - (t0 - sl * XT);
        orow = ((rank - 1 - sl) & (NR - 1)) * MSH + MSH - ocnt * 16;
        const unsigned long long tw = __builtin_amdgcn_s_memrealtime();
        while (true) {
            unsigned f, tm;
            asm volatile("global_load_dword %0, %2, %3 sc0 sc1\n\t"
                         "global_load_dword %1, %4, %5 sc0 sc1\n\t"
                         "s_waitcnt vmcnt(0)" : "=&v"(f), "=&v"(tm)
                         : "v"((lane < 56 ? lane : 0) * 4), "s"(myws + WS_XFLAG), "v"(0u), "s"(myws + WS_TMO) : "memory");
            if (__builtin_amdgcn_ballot_w64(f < call) == 0ull) break;
            if (first(tm) == call || __builtin_amdgcn_s_memrealtime() - tw > DEADLINE_TICKS) {
                if (lane == 0) store4_sys(0, call, myws + WS_TMO);
                poison = __builtin_nanf("");
                break;
            }
            __builtin_amdgcn_s_sleep(8);
        }
        barrier();
        STAMP(8);
    }
    orow *= NSH * 4;
    xin += wave * 1024 - 4096 + (size_t)t0 * (16 * KD);
    v4u xsrd = make_srd(xin, XWIN);
    auto next_x = [&]() { xin += 16 * KD; xsrd = make_srd(xin, XWIN); };
    auto next_o = [&]() {
        orow += 16 * NSH * 4;
        if (--ocnt == 0) { orow = (orow - (MSH + XM) * NSH * 4) & (NR * MSH * NSH * 4 - 1); ocnt = XT; }
    };
    auto issue_w = [&](int h, unsigned unit) { dma4(unit, wo, wsrd, (unsigned)(h ^ (xg << 1)) * (64 * KD)); };

    if (delay) {
        wait_until(delay);
        STAMP(7); CSTAMP(10);
    }

    #pragma unroll
    for (int q = 0; q < 4; ++q) dma4(ldsw + q * 4096, xo, xsrd, xq[q]);
    next_x();
    issue_w(0, ldsw + 16384);
    issue_w(1, ldsw + 20480);
    asm volatile("s_barrier" ::: "memory");
    issue_w(2, ldsw + 24576);
    issue_w(3, ldsw + 28672);
    issue_w(4, ldsr);
    issue_w(5, ldsr + 16384);
    PIN();
    STAMP(14);
    const LDS_AS u8* wrw[4];
    const LDS_AS u8* wrr[4];
    {
        const int q = i16 >> 1, p = i16 & 1;
        const int f = ((q >> 2) & 1) | ((g & 1) << 1);
        #pragma unroll
        for (int j = 0; j < 4; ++j) {
            const unsigned o = g * 1024 + q * 64 + ((j ^ f) << 4) + 8 * p;
            wrw[j] = (const LDS_AS u8*)(size_t)(ldsw + o);
            wrr[j] = (const LDS_AS u8*)(size_t)(ldsr + o);
        }
    }
    const unsigned xr0 = ldsw + i16 * 128 + ((g ^ (i16 >> 1)) << 4);
    const int om = 4 * wave + g;
    const unsigned ovoff = om * (NSH * 4) + i16 * 16;
    const unsigned prd0 = lds0 + om * 256 + ((i16 ^ om) << 4);
    unsigned pwr0[4];
    #pragma unroll
    for (int j = 0; j < 4; ++j) pwr0[j] = ldsr + i16 * 256 + (((4 * j + g) ^ i16) << 4);
    v8i wf[8][4], xf[4];
    v4f acc[4];
    float scale;
    auto read_xs = [&](unsigned ad, int s) {
        const v4i lo = *(const LDS_AS v4i*)((const LDS_AS u8*)(size_t)ad + s * 2048);
        const v4i hi = *(const LDS_AS v4i*)((const LDS_AS u8*)(size_t)(ad ^ 64u) + s * 2048);
        return (v8i){lo[0], lo[1], lo[2], lo[3], hi[0], hi[1], hi[2], hi[3]};
    };
    auto read_w = [&](int h, const LDS_AS u8* const* base, int off) {
        #pragma unroll
        for (int j = 0; j < 4; ++j) {
            #pragma unroll
            for (int r = 0; r < 2; ++r) {
                const v2i t = __builtin_amdgcn_ds_read_tr8_b64_v2i32((LDS_AS v2i*)(base[j] + off + r * 512));
                wf[h >> 1][j][(h & 1) * 4 + 2 * r] = t[0];
                wf[h >> 1][j][(h & 1) * 4 + 2 * r + 1] = t[1];
            }
        }
        LGKM0();
        STAMP(42 + h);
        if (h & 1) {
            #pragma unroll
            for (int j = 0; j < 4; ++j) {
                if (h < 12) asm volatile("" : "+a"(wf[h >> 1][j]));
                else asm volatile("" : "+v"(wf[h >> 1][j]));
            }
        }
    };
    const v4f zero = {0.f, 0.f, 0.f, 0.f};
    const u8* const obase = aout + strip * 256;
    auto pw = [&](int j, unsigned bo) { *(LDS_AS v4f*)(size_t)(pwr0[j] + bo) = acc[j]; };
    auto pr = [&](int k, unsigned bo) { return *(const LDS_AS v4f*)((const LDS_AS u8*)(size_t)(prd0 + bo) + k * 4096); };
    auto sum4 = [&](const v4f (&p)[4], int e) {
        float a = p[0][e] + p[1][e], c = p[2][e] + p[3][e];
        KEEP(a); KEEP(c);
        float s = a + c;
        KEEP(s);
        return s;
    };
    auto finish = [&](const v4f (&p)[4]) {
        float y[4];
        #pragma unroll
        for (int e = 0; e < 4; ++e) {
            const float t = act1(y[e], sum4(p, e), scale);
            y[e] = act3(y[e], act2(t));
        }
        return (v4f){y[0], y[1], y[2], y[3]};
    };
    {
        v8i x0[8];
        auto mma = [&](int s) {
            PIN();
            #pragma unroll
            for (int j = 0; j < 4; ++j)
                acc[j] = __builtin_amdgcn_mfma_scale_f32_16x16x128_f8f6f4(wf[s][j], x0[s], s ? acc[j] : zero, 0, 0, 0, 0, 0, 0);
            PIN();
        };
        STAMP(40);
        VMW(24);
        STAMP(41);
        #pragma unroll
        for (int s = 0; s < 8; ++s) x0[s] = read_xs(xr0, s);
        LGKM0();
        issue_w(6, ldsw);            VMW(24); read_w(0, wrw, 16384);
        issue_w(7, ldsw + 4096);     VMW(24); read_w(1, wrw, 20480);                          mma(0);
        issue_w(8, ldsw + 8192);     VMW(24); read_w(2, wrw, 24576);
        issue_w(9, ldsw + 12288);    VMW(24); read_w(3, wrw, 28672);                          mma(1);
        issue_w(10, ldsw + 16384);   VMW(24); read_w(4, wrr, 0);
        issue_w(11, ldsw + 20480);   VMW(20); read_w(5, wrr, 16384);  read_w(6, wrw, 0);      mma(2);
        issue_w(12, ldsw + 24576);   VMW(16); read_w(7, wrw, 4096);   read_w(8, wrw, 8192);   mma(3);
        issue_w(13, ldsw + 28672);   VMW(16); read_w(9, wrw, 12288);                          mma(4);
        issue_w(14, ldsr);           VMW(16); read_w(10, wrw, 16384);
        issue_w(15, ldsr + 16384);   VMW(16); read_w(11, wrw, 20480);                         mma(5);
        asm volatile("s_barrier" ::: "memory");
        dma4(ldsw, xo, xsrd, xq[0]);          VMW(12); read_w(12, wrw, 24576); read_w(13, wrw, 28672); mma(6);
        dma4(ldsw + 4096, xo, xsrd, xq[1]);   VMW(8);  read_w(14, wrr, 0);     read_w(15, wrr, 16384); mma(7);
        STAMP(2); CSTAMP(11);
        asm volatile("s_waitcnt lgkmcnt(0)\n\tv_mov_b32 %0, %1\n\tv_mul_f32 %0, %2, %0" : "=v"(scale) : "s"(sclx), "s"(sclw));
        scale += poison;
        dma4(ldsw + 8192, xo, xsrd, xq[2]);
        #pragma unroll
        for (int j = 0; j < 4; ++j) pw(j, 0);
        PIN();
        dma4(ldsw + 12288, xo, xsrd, xq[3]);
        next_x();
        barrier();
        v4f p[4];
        #pragma unroll
        for (int k = 0; k < 4; ++k) p[k] = pr(k, 0);
        PIN();
        dma4(ldsw + 16384, xo, xsrd, xq[0]);
        VMW(12);
        #pragma unroll
        for (int s = 0; s < 4; ++s) x0[s] = read_xs(xr0, s);
        PIN();
        dma4(ldsw + 20480, xo, xsrd, xq[1]);
        mma(0);
        const v4f o = finish(p);
        mma(1);
        dma4(ldsw + 24576, xo, xsrd, xq[2]);
        VMW(12);
        #pragma unroll
        for (int s = 4; s < 8; ++s) x0[s] = read_xs(xr0, s);
        mma(2);
        dma4(ldsw + 28672, xo, xsrd, xq[3]);
        next_x();
        mma(3);
        LGKM0();
        mma(4);
        dma4(ldsw, xo, xsrd, xq[0]);
        mma(5);
        dma4(ldsw + 4096, xo, xsrd, xq[1]);
        mma(6);
        dma4(ldsw + 8192, xo, xsrd, xq[2]);
        mma(7);
        dma4(ldsw + 12288, xo, xsrd, xq[3]);
        next_x();
        VMW(28);
        xf[0] = read_xs(xr0 + 16384, 0);
        xf[1] = read_xs(xr0 + 16384, 1);
        set_m0(ldsw + 16384);
        dma_after<0>(xf[0], xo[0], xsrd, xq[0]);
        dma<1024>(xo[1], xsrd, xq[0]);
        store16_sys(ovoff + orow, o, obase);
        STAMP(16);
        next_o();
        STAMP(13);
    }

    unsigned boff = 16384;
    #pragma unroll 1
    for (int i = 2; i < nt; ++i) {
        const unsigned lb = ldsw + boff, xr = xr0 + boff, xn = xr0 + (boff ^ 16384u);
        v4f n[4], p[4];
        float y[4], t[4];
        #define MM(D, S, J, C) do { PIN(); KEEP(xf[(S) & 3]); \
            D = __builtin_amdgcn_mfma_scale_f32_16x16x128_f8f6f4(wf[S][J], xf[(S) & 3], C, 0, 0, 0, 0, 0, 0); \
            asm volatile("" : "+a"(D)); PIN(); } while (0)
        #define RD(S) do { VMW(28); xf[((S) + 2) & 3] = read_xs((S) < 6 ? xr : xn, ((S) + 2) & 7); } while (0)
        #define RQA(S) dma_after<(((S) + 1) & 1) * 2048>(xf[((S) + 1) & 3], xo[(((S) + 1) & 1) * 2], xsrd, xq[(((S) + 1) >> 1) & 3])
        #define RQB(S) dma<(((S) + 1) & 1) * 2048 + 1024>(xo[(((S) + 1) & 1) * 2 + 1], xsrd, xq[(((S) + 1) >> 1) & 3])
        MM(n[0], 0, 0, zero);  RD(0);  pw(0, boff);
        MM(n[1], 0, 1, zero);  RQA(0); pw(1, boff);
        MM(n[2], 0, 2, zero);          pw(2, boff);
        MM(n[3], 0, 3, zero);  RQB(0); pw(3, boff);
        MM(n[0], 1, 0, n[0]);  RD(1);  set_m0(lb + 4096);
        MM(n[1], 1, 1, n[1]);  RQA(1);
        const unsigned ov = ovoff + orow;
        MM(n[2], 1, 2, n[2]);
        MM(n[3], 1, 3, n[3]);  RQB(1);
        MM(n[0], 2, 0, n[0]);  RD(2);
        MM(n[1], 2, 1, n[1]);  RQA(2);
        asm volatile("s_waitcnt lgkmcnt(2)\n\ts_barrier" ::: "memory");
        p[0] = pr(0, boff); p[1] = pr(1, boff);
        MM(n[2], 2, 2, n[2]);
        p[2] = pr(2, boff); p[3] = pr(3, boff);
        MM(n[3], 2, 3, n[3]);  RQB(2);
        MM(n[0], 3, 0, n[0]);  RD(3);  set_m0(lb + 8192);
        MM(n[1], 3, 1, n[1]);  RQA(3);
        MM(n[2], 3, 2, n[2]);          y[0] = sum4(p, 0);
        MM(n[3], 3, 3, n[3]);  RQB(3); y[1] = sum4(p, 1);
        MM(n[0], 4, 0, n[0]);  RD(4);  t[0] = act1(y[0], y[0], scale);
        MM(n[1], 4, 1, n[1]);  RQA(4); y[2] = sum4(p, 2);                  t[0] = act2(t[0]);
        MM(n[2], 4, 2, n[2]);          t[1] = act1(y[1], y[1], scale);     y[0] = act3(y[0], t[0]);
        MM(n[3], 4, 3, n[3]);  RQB(4); y[3] = sum4(p, 3);                  t[1] = act2(t[1]);
        MM(n[0], 5, 0, n[0]);  RD(5);  set_m0(lb + 12288);                 y[1] = act3(y[1], t[1]);
        MM(n[1], 5, 1, n[1]);  RQA(5); t[2] = act1(y[2], y[2], scale);
        MM(n[2], 5, 2, n[2]);          t[3] = act1(y[3], y[3], scale);     t[2] = act2(t[2]);
        MM(n[3], 5, 3, n[3]);  RQB(5); t[3] = act2(t[3]);                  y[2] = act3(y[2], t[2]);
        MM(n[0], 6, 0, n[0]);  RD(6);  y[3] = act3(y[3], t[3]);
        MM(n[1], 6, 1, n[1]);  RQA(6);
        MM(n[2], 6, 2, n[2]);
        store16_sys(ov, (v4f){y[0], y[1], y[2], y[3]}, obase);
        next_o();
        MM(n[3], 6, 3, n[3]);  RQB(6);
        boff ^= 16384u;
        next_x();
        if (i + 3 >= nt) xsrd[2] = 0;
        MM(acc[0], 7, 0, n[0]);  RD(7);  set_m0(ldsw + boff);
        MM(acc[1], 7, 1, n[1]);  RQA(7);
        STAMP(15 + i);
        MM(acc[2], 7, 2, n[2]);
        MM(acc[3], 7, 3, n[3]);  RQB(7);
    }
    {
        #pragma unroll
        for (int j = 0; j < 4; ++j) pw(j, boff);
        barrier();
        v4f p[4];
        #pragma unroll
        for (int k = 0; k < 4; ++k) p[k] = pr(k, boff);
        store16_sys(ovoff + orow, finish(p), obase);
        STAMP(15 + nt);
    }
    STAMP(3); CSTAMP(12);

    VMW(0);
    STAMP(6);
    barrier();
    asm volatile(".Lpc_end:");
    if (wave == 2) {
        const unsigned call = call_no();
        if (lane == 0) {
            if (rel != 0) store4_sys(0, call, dws + WS_OFLAG + ((rel - 1) * 32 + idx) * 4);
            store4_sys(0, call, cntp);
        }
        STAMP(4);
        if (b == 0) {
            const unsigned fo = (lane < 56 ? lane : 0) * 16;
            const unsigned long long tw = __builtin_amdgcn_s_memrealtime();
            while (true) {
                v4u f;
                unsigned tm;
                asm volatile("global_load_dwordx4 %0, %2, %3 sc0 sc1\n\t"
                             "global_load_dword %1, %4, %3 offset:-2048 sc0 sc1\n\t"
                             "s_waitcnt vmcnt(0)" : "=&v"(f), "=&v"(tm) : "v"(fo), "s"(myws + WS_OFLAG), "v"(0u) : "memory");
                const bool miss = f[0] < call || f[1] < call || f[2] < call || f[3] < call;
                if (__builtin_amdgcn_ballot_w64(miss) == 0ull) break;
                if (first(tm) == call || __builtin_amdgcn_s_memrealtime() - tw > DEADLINE_TICKS) {
                    if (lane == 0) {
                        store4_sys(0, call, myws + WS_TMO);
                        store4_sys(0, 0x7fc00000u, aout);
                    }
                    break;
                }
                __builtin_amdgcn_s_sleep(1);
            }
            __builtin_amdgcn_fence(__ATOMIC_ACQUIRE, "");
            STAMP(5);
        }
    }
}
static_assert(WS_OFLAG - WS_TMO == 2048, "layout");

inline unsigned mkjob(int nt, int t0, int strip, int xg, int delay50, int staged) {
    return (unsigned)nt | (unsigned)t0 << 5 | (unsigned)strip << 10 | (unsigned)xg << 13 | (unsigned)delay50 << 16 |
           (unsigned)staged << 24;
}

#define DLY1 0
#define DLYE 120
#define DLY2 140
#define DLY3 140
#define DLYL 20
#define DLYM 110

unsigned build_jobs(Args& a) {
    unsigned first4 = 0;
    const int n = 32 - XT;
    for (int idx = 0; idx < 32; ++idx) {
        {
            const int strip = (idx >> 4) * 4 + (idx & 3), sub = 3 - ((idx >> 2) & 3);
            const int lo = sub * n / 4, hi = (sub + 1) * n / 4;
            const int dly = idx < 8 ? 0 : idx < 16 ? DLY1 : idx < 20 ? DLYE : idx < 24 ? DLY2 : DLY3;
            a.jobs[0][idx] = mkjob(hi - lo, lo, strip, (idx & 3) * XGA, dly, 0);
            if (idx < 16) first4 |= (unsigned)lo << (5 * sub);
        }
        {
            const int strip = idx & 7, e = (idx >> 3) & 1, ns = 7 * XT;
            if (idx < 16) a.jobs[1][idx] = mkjob(16, e * 16, strip, idx & 7, DLYL, 0);
            else a.jobs[1][idx] = mkjob(e ? ns - ns / 2 : ns / 2, e ? ns / 2 : 0, strip, idx & 7, DLYM, 1);
        }
    }
    return first4;
}

}

extern "C" void kernel_launch(int rank, int nranks, ncclComm_t comm,
                              hipStream_t stream,
                              void* const* d_in, const int* in_sizes, int n_in,
                              void* d_out, int out_size,
                              const int* in_dtypes, int out_dtype,
                              void* const* d_out_all,
                              void* const* d_ws_all, size_t ws_bytes) {
    (void)comm; (void)in_dtypes; (void)out_dtype;
    if (nranks != NR || n_in != 4 || (unsigned)rank >= (unsigned)NR || ws_bytes < WS_NEED ||
        in_sizes[0] != MSH * KD || in_sizes[1] != KD * NR * NSH || out_size != NR * MSH * NSH) {
        fprintf(stderr, "kernel_launch: unexpected problem shape\n");
        return;
    }
    Args a;
    memset(&a, 0, sizeof(a));
    const unsigned first4 = build_jobs(a);
    for (int r = 0; r < NR; ++r) {
        const int dest = (rank + r) % NR;
        Rec& c = a.rec[r];
        c.x = (const u8*)d_in[0];
        c.w = (const u8*)d_in[1] + dest * NSH;
        c.out = (u8*)(r ? d_out_all[dest] : d_out);
        c.dws = (u8*)d_ws_all[dest];
        c.myws = (u8*)d_ws_all[rank];
        c.sx = (const float*)d_in[2];
        c.sw = (const float*)d_in[3];
        c.rank = (unsigned)rank;
        c.first4 = r ? first4 : 16u << 5;
    }
    hipLaunchKernelGGL(fused_kernel, dim3(NWG), dim3(NT), 0, stream, a);
    HIP_CHECK(hipGetLastError());
}
```

```cpp
#include <hip/hip_runtime.h>
#include <rccl/rccl.h>
#include <cstdio>
#include <cstring>

#define HIP_CHECK(expr) do { hipError_t _e = (expr); if (_e != hipSuccess) { \
    fprintf(stderr, "HIP error %s at %s:%d\n", hipGetErrorString(_e), __FILE__, __LINE__); \
    return; } } while (0)

namespace {

typedef int v2i __attribute__((ext_vector_type(2)));
typedef int v4i __attribute__((ext_vector_type(4)));
typedef int v8i __attribute__((ext_vector_type(8)));
typedef float v4f __attribute__((ext_vector_type(4)));
typedef unsigned v4u __attribute__((ext_vector_type(4)));
typedef unsigned v16u __attribute__((ext_vector_type(16)));
typedef unsigned char u8;
#define LDS_AS __attribute__((address_space(3)))

constexpr int NR = 8;
constexpr int NWG = 256;
constexpr int NT = 256;
constexpr int KD = 4096;
constexpr int MSH = 512;
constexpr int NSH = 512;
#define CODEPF 1
#define XPF 2
#define XPD 23
#define DLYP 0
#define DLYW 0
#define XGA 2
#define SH 8
#define T0E 0
#define XM_ROWS 48
constexpr int XM = XM_ROWS;
constexpr int XT = XM / 16;
constexpr int XPW = XM / 8;
constexpr unsigned long long DEADLINE_TICKS = 100000000ull;

constexpr int LDS_BUF = 32768;
constexpr int LDS_BYTES = LDS_BUF + 4 * 32768;

constexpr size_t WS_CNT = 0;
constexpr size_t WS_TMO = 2048;
constexpr size_t WS_OFLAG = 4096;
constexpr size_t WS_XFLAG = 8192;
constexpr size_t WS_XST = 65536;
constexpr size_t WS_XST_PAR = (size_t)7 * 64 * KD;
constexpr size_t WS_DBG = WS_XST + 2 * WS_XST_PAR;
constexpr size_t WS_NEED = WS_DBG + 131072;

struct Rec {
    const u8* x;
    const u8* w;
    u8* out;
    u8* dws;
    u8* myws;
    const float* sx;
    const float* sw;
    unsigned rank;
    unsigned first4;
};
struct alignas(64) Args {
    Rec rec[NR];
    unsigned jobs[2][32];
};
static_assert(sizeof(Rec) == 64, "layout");

template <int O>
__device__ __forceinline__ void dma(unsigned v, v4u srd, unsigned s) {
    asm volatile("buffer_load_dwordx4 %0, %1, %2 offen offset:%3 lds" :: "v"(v), "s"(srd), "s"(s), "n"(O) : "memory");
}
template <int O>
__device__ __forceinline__ void dma_after(const v8i& f, unsigned v, v4u srd, unsigned s) {
    asm volatile("buffer_load_dwordx4 %0, %1, %2 offen offset:%3 lds" :: "v"(v), "s"(srd), "s"(s), "n"(O), "v"(f) : "memory");
}
__device__ __forceinline__ void set_m0(unsigned lds) {
    asm volatile("s_mov_b32 m0, %0\n\ts_nop 0" :: "s"(lds) : "memory", "m0");
}
__device__ __forceinline__ void dma4(unsigned lds, const unsigned (&v)[4], v4u srd, unsigned s) {
    set_m0(lds);
    dma<0>(v[0], srd, s); dma<1024>(v[1], srd, s); dma<2048>(v[2], srd, s); dma<3072>(v[3], srd, s);
}
__device__ __forceinline__ void glds1(unsigned lds, unsigned v0, const u8* base) {
    asm volatile("s_mov_b32 m0, %0\n\ts_nop 0\n\tglobal_load_lds_dwordx4 %1, %2" :: "s"(lds), "v"(v0), "s"(base) : "memory", "m0");
}
__device__ __forceinline__ v4u make_srd(const u8* p, unsigned bytes) {
    const unsigned long long a = (unsigned long long)p;
    return (v4u){(unsigned)a, (unsigned)(a >> 32) & 0xffffu, bytes, 0x00020000u};
}

#define VMW(n) asm volatile("s_waitcnt vmcnt(" #n ")" ::: "memory")
#define LGKM0() asm volatile("s_waitcnt lgkmcnt(0)" ::: "memory")
#define PIN() __builtin_amdgcn_sched_barrier(0)
#define KEEP(x) asm volatile("" : "+v"(x))
#define HOLD(a) asm volatile("" :: "v"(a))
#define VMN(n) asm volatile("s_waitcnt vmcnt(%0)" :: "n"(n) : "memory")

__device__ __forceinline__ void store16_sys(unsigned voff, v4f v, const void* base) {
    asm volatile("global_store_dwordx4 %0, %1, %2 sc0 sc1\n\ts_nop 1" :: "v"(voff), "v"(v), "s"(base) : "memory");
}
__device__ __forceinline__ void store4_sys(unsigned voff, unsigned v, const void* base) {
    asm volatile("global_store_dword %0, %1, %2 sc0 sc1" :: "v"(voff), "v"(v), "s"(base) : "memory");
}
__device__ __forceinline__ v4f load16(unsigned voff, const void* base) {
    v4f r;
    asm volatile("global_load_dwordx4 %0, %1, %2" : "=v"(r) : "v"(voff), "s"(base) : "memory");
    return r;
}
__device__ __forceinline__ void barrier() {
    asm volatile("s_waitcnt lgkmcnt(0)\n\ts_barrier" ::: "memory");
}
__device__ __forceinline__ unsigned first(unsigned v) { return (unsigned)__builtin_amdgcn_readfirstlane(v); }

__device__ __forceinline__ float act1(float& y, float acc, float s) {
    y = acc * s;
    float t = __builtin_amdgcn_fmed3f(y, -60.0f, 60.0f) * -1.44269504f;
    KEEP(y); KEEP(t);
    return t;
}
__device__ __forceinline__ float act2(float t) {
    float d = __builtin_amdgcn_exp2f(t) + 1.0f;
    KEEP(d);
    return d;
}
__device__ __forceinline__ float act3(float y, float d) {
    float o = y * __builtin_amdgcn_rcpf(d);
    KEEP(o);
    return o;
}

#define STAMP(i) do {} while (0)
#define CSTAMP(i) do {} while (0)

__device__ __forceinline__ u8* ptr(unsigned lo, unsigned hi) {
    return (u8*)(((unsigned long long)hi << 32) | lo);
}

__global__ __launch_bounds__(NT) void fused_kernel(const Args args) {
    __shared__ __attribute__((aligned(1024))) u8 smem[LDS_BYTES];
    const int b = blockIdx.x;
    const int rel = b & 7, idx = (b >> 3) & 31;
    v16u rec;
    unsigned job;
    unsigned long long t_start;
    asm volatile("s_load_dwordx16 %0, %3, %4\n\t"
                 "s_load_dword %1, %3, %5\n\t"
                 "s_memrealtime %2\n\t"
                 "s_waitcnt lgkmcnt(0)"
                 : "=&s"(rec), "=&s"(job), "=&s"(t_start)
                 : "s"(__builtin_amdgcn_kernarg_segment_ptr()), "s"(rel * 64), "s"(512 + (rel ? 0 : 128) + idx * 4) : "memory");
    const u8* const ax = ptr(rec[0], rec[1]);
    const u8* const aw = ptr(rec[2], rec[3]);
    u8* const aout = ptr(rec[4], rec[5]);
    u8* const dws = ptr(rec[6], rec[7]);
    u8* const myws = ptr(rec[8], rec[9]);
    const int rank = rec[14] & (NR - 1);
    const int tid = threadIdx.x;
    const int lane = tid & 63;
    const int wave = __builtin_amdgcn_readfirstlane(tid >> 6);
    const u8* const cntp = myws + WS_CNT + rel * 128 + idx * 4;
    const unsigned lds0 = (unsigned)(size_t)(LDS_AS u8*)smem;
    const unsigned ldsw = lds0 + LDS_BUF + wave * 32768;
    const unsigned ldsr = lds0 + wave * 4096;
    const int g = lane >> 4, i16 = lane & 15;
    unsigned cnt;
    float sclx, sclw;
    asm volatile("s_load_dword %0, %3, 0x0\n\t"
                 "s_load_dword %1, %4, 0x0\n\t"
                 "s_load_dword %2, %5, 0x0"
                 : "=&s"(cnt), "=&s"(sclx), "=&s"(sclw)
                 : "s"(cntp), "s"(ptr(rec[10], rec[11])), "s"(ptr(rec[12], rec[13])) : "memory");
    auto call_no = [&]() {
        unsigned c;
        asm volatile("s_waitcnt lgkmcnt(0)\n\ts_add_u32 %0, %1, 1" : "=s"(c) : "s"(cnt) : "memory", "scc");
        return c;
    };

    auto wait_until = [&](unsigned ticks) {
        while ((unsigned)__builtin_amdgcn_s_memrealtime() - (unsigned)t_start < ticks) __builtin_amdgcn_s_sleep(1);
    };
    const bool staged = rel == 0 && (job >> 24 & 1u);
    if (staged && wave == 0) __builtin_amdgcn_fence(__ATOMIC_ACQUIRE, "");
    if (idx >= 16) {
        const int hw = (idx - 16) * 4 + wave;
        if (idx == 16 && wave >= 2) {
            const u8* pc;
            unsigned len;
            asm volatile("s_getpc_b64 %0\n.Lpc_here:\n\ts_mov_b32 %1, .Lpc_end-.Lpc_here" : "=s"(pc), "=s"(len));
            unsigned o = ((wave - 2) * 64 + lane) * 128;
            if (o > len - 16) o = len - 16;
            glds1(ldsw + 31 * 1024, o & ~15u, (const u8*)((unsigned long long)pc & ~15ull));
        }
        auto help_x = [&](int n) {
            #pragma unroll
            for (int j = 0; j < n; ++j) {
                const unsigned tl = (rec[15] >> (5 * j)) & 31u;
                glds1(ldsw + j * 1024, lane * 16, ax + (size_t)tl * (16 * KD) + hw * 1024);
            }
        };
        auto help_w = [&]() {
            const unsigned q = (lane >> 3) & 1;
            #pragma unroll
            for (int j = 0; j < 16; ++j)
                glds1(ldsw + 4096 + j * 1024, (((j >> 1 & 3) | (j & 1) << 2 | (j & 8)) ^ (q << 3)) * (64 * KD) + (lane >> 4) * KD + (lane & 15) * 16,
                      aw + (size_t)((hw >> 4) * 1024 + (hw & 15) * 4) * KD);
        };
        if (rel == 0) {
            if (DLYP) wait_until(DLYP * 5);
            help_x(2);
            const unsigned q = (lane >> 3) & 3;
            #pragma unroll 1
            for (int j = 0; j < 32; ++j)
                glds1(ldsw + j * 1024, ((j >> 1) ^ (q << 2)) * (64 * KD) + (lane >> 5) * KD + (lane & 31) * 16,
                      aw + (size_t)((hw >> 4) * 1024 + (hw & 15) * 4 + (j & 1) * 2) * KD);
            VMW(0);
            STAMP(9);
        } else if (idx >= 24) {
            v4f xv[XPW];
            const unsigned xoff = ((idx & 7) * 4 + wave) * (XPW * 1024) + lane * 16;
            #pragma unroll
            for (int i = 0; i < XPW; ++i) xv[i] = load16(xoff + i * 1024, ax + (size_t)(MSH - XM) * KD);
            help_x(4);
            help_w();
            const unsigned call = call_no();
            const u8* dst = dws + WS_XST + (size_t)(call & 1u) * WS_XST_PAR + (size_t)(rel - 1) * (XM * KD);
            VMW(20);
            STAMP(9);
            #pragma unroll
            for (int i = 0; i < XPW; ++i) {
                KEEP(xv[i]);
                store16_sys(xoff + i * 1024, xv[i], dst);
            }
            VMW(0);
            barrier();
            if (tid == 0) store4_sys(0, call, dws + WS_XFLAG + ((rel - 1) * 8 + (idx & 7)) * 4);
            STAMP(1);
        } else {
            help_x(4);
            help_w();
            VMW(8);
            STAMP(9);
            #pragma unroll 1
            for (int j = 0; j < XPF * 8; ++j) {
                unsigned tl = ((rec[15] >> (5 * ((j >> 1) & 3))) & 31u) + 1 + (j >> 3);
                if (tl > 31) tl = 31;
                glds1(ldsw + (j & 31) * 1024, lane * 16, ax + (size_t)tl * (16 * KD) + (hw * 2 + (j & 1)) * 1024);
                asm volatile("s_waitcnt vmcnt(%0)" :: "n"(XPD) : "memory");
            }
            if (DLYW) wait_until(DLYW * 5);
            #pragma unroll 1
            for (int j = 0; j < 32; ++j) {
                glds1(ldsw + j * 1024, (lane >> 4) * KD + (lane & 15) * 16, aw + (size_t)(j * 128 + hw * 4) * KD + 256);
                asm volatile("s_waitcnt vmcnt(%0)" :: "n"(XPD) : "memory");
            }
            VMW(0);
            STAMP(1);
        }
    }

    int nt = job & 31u;
    if (nt < 5) nt = 5;
    int t0 = (job >> 5) & 31u;
    if (t0 + nt > (staged ? 7 * XT + 1 : 32)) { t0 = 0; nt = 5; }
    const int strip = (job >> 10) & 7;
    const int xg = (job >> 13) & 7;
    const unsigned delay = ((job >> 16) & 255u) * 5u;

    unsigned wo[4];
    {
        const int row = lane >> 2, pos = lane & 3;
        #pragma unroll
        for (int p = 0; p < 4; ++p)
            wo[p] = (16 * p + row) * KD + ((pos ^ (((row >> 2) & 1) | ((p & 1) << 1))) << 4) - p * 1024;
    }
    unsigned xo[4];
    #pragma unroll
    for (int p = 0; p < 4; ++p) {
        const int m = 8 * (p & 1) + (lane >> 3);
        const int cs = (lane & 7) ^ (m >> 1);
        xo[p] = m * KD + ((((p >> 1) ^ (xg & 1)) << 7) + (cs << 4)) - p * 1024 + 4096;
    }
    unsigned xq[4];
    #pragma unroll
    for (int q = 0; q < 4; ++q) xq[q] = (q ^ (xg >> 1)) << 8;
    const v4u wsrd = make_srd(aw + (size_t)(wave * 1024) * KD + strip * 64, 1024 * KD);
    constexpr unsigned XWIN = 17 * KD + 4096;
    const int tf = staged ? 30 + (job >> 25 & 1u) : t0;
    unsigned orow = (rank * MSH + tf * 16) * (NSH * 4);
    int ocnt = 32;
    float poison = 0.f;
    const u8* xin = ax + (wave * 1024 - 4096) + (size_t)tf * (16 * KD);
    v4u xsrd = make_srd(xin, XWIN);
    auto next_x = [&]() { xin += 16 * KD; xsrd = make_srd(xin, XWIN); };
    auto next_o = [&]() {
        orow += 16 * NSH * 4;
        if (--ocnt == 0) { orow = (orow - (MSH + XM) * NSH * 4) & (NR * MSH * NSH * 4 - 1); ocnt = XT; }
    };
    auto issue_w = [&](int h, unsigned unit) { dma4(unit, wo, wsrd, (unsigned)(h ^ (xg << 1)) * (64 * KD)); };

    if (delay) {
        wait_until(delay);
        STAMP(7); CSTAMP(10);
    }

    #pragma unroll
    for (int q = 0; q < 4; ++q) dma4(ldsw + q * 4096, xo, xsrd, xq[q]);
    next_x();
    issue_w(0, ldsw + 16384);
    issue_w(1, ldsw + 20480);
    asm volatile("s_barrier" ::: "memory");
    issue_w(2, ldsw + 24576);
    issue_w(3, ldsw + 28672);
    issue_w(4, ldsr);
    issue_w(5, ldsr + 16384);
    PIN();
    STAMP(14);
    const LDS_AS u8* wrw[4];
    const LDS_AS u8* wrr[4];
    {
        const int q = i16 >> 1, p = i16 & 1;
        const int f = ((q >> 2) & 1) | ((g & 1) << 1);
        #pragma unroll
        for (int j = 0; j < 4; ++j) {
            const unsigned o = g * 1024 + q * 64 + ((j ^ f) << 4) + 8 * p;
            wrw[j] = (const LDS_AS u8*)(size_t)(ldsw + o);
            wrr[j] = (const LDS_AS u8*)(size_t)(ldsr + o);
        }
    }
    const unsigned xr0 = ldsw + i16 * 128 + ((g ^ (i16 >> 1)) << 4);
    const int om = 4 * wave + g;
    const unsigned ovoff = om * (NSH * 4) + i16 * 16;
    const unsigned prd0 = lds0 + om * 256 + ((i16 ^ om) << 4);
    unsigned pwr0[4];
    #pragma unroll
    for (int j = 0; j < 4; ++j) pwr0[j] = ldsr + i16 * 256 + (((4 * j + g) ^ i16) << 4);
    v8i wf[8][4], xf[4];
    v4f acc[4];
    float scale;
    v4f oh;
    unsigned ovh;
    auto read_xs = [&](unsigned ad, int s) {
        const v4i lo = *(const LDS_AS v4i*)((const LDS_AS u8*)(size_t)ad + s * 2048);
        const v4i hi = *(const LDS_AS v4i*)((const LDS_AS u8*)(size_t)(ad ^ 64u) + s * 2048);
        return (v8i){lo[0], lo[1], lo[2], lo[3], hi[0], hi[1], hi[2], hi[3]};
    };
    auto read_w = [&](int h, const LDS_AS u8* const* base, int off) {
        #pragma unroll
        for (int j = 0; j < 4; ++j) {
            #pragma unroll
            for (int r = 0; r < 2; ++r) {
                const v2i t = __builtin_amdgcn_ds_read_tr8_b64_v2i32((LDS_AS v2i*)(base[j] + off + r * 512));
                wf[h >> 1][j][(h & 1) * 4 + 2 * r] = t[0];
                wf[h >> 1][j][(h & 1) * 4 + 2 * r + 1] = t[1];
            }
        }
        LGKM0();
        STAMP(42 + h);
        if (h & 1) {
            #pragma unroll
            for (int j = 0; j < 4; ++j) {
                if (h < 12) asm volatile("" : "+a"(wf[h >> 1][j]));
                else asm volatile("" : "+v"(wf[h >> 1][j]));
            }
        }
    };
    const v4f zero = {0.f, 0.f, 0.f, 0.f};
    const u8* const obase = aout + strip * 256;
    auto pw = [&](int j, unsigned bo) { *(LDS_AS v4f*)(size_t)(pwr0[j] + bo) = acc[j]; };
    auto pr = [&](int k, unsigned bo) { return *(const LDS_AS v4f*)((const LDS_AS u8*)(size_t)(prd0 + bo) + k * 4096); };
    auto sum4 = [&](const v4f (&p)[4], int e) {
        float a = p[0][e] + p[1][e], c = p[2][e] + p[3][e];
        KEEP(a); KEEP(c);
        float s = a + c;
        KEEP(s);
        return s;
    };
    auto finish = [&](const v4f (&p)[4]) {
        float y[4];
        #pragma unroll
        for (int e = 0; e < 4; ++e) {
            const float t = act1(y[e], sum4(p, e), scale);
            y[e] = act3(y[e], act2(t));
        }
        return (v4f){y[0], y[1], y[2], y[3]};
    };
    {
        v8i x0[8];
        auto mma = [&](int s) {
            PIN();
            #pragma unroll
            for (int j = 0; j < 4; ++j)
                acc[j] = __builtin_amdgcn_mfma_scale_f32_16x16x128_f8f6f4(wf[s][j], x0[s], s ? acc[j] : zero, 0, 0, 0, 0, 0, 0);
            PIN();
        };
        STAMP(40);
        VMW(24);
        STAMP(41);
        #pragma unroll
        for (int s = 0; s < 8; ++s) x0[s] = read_xs(xr0, s);
        LGKM0();
        issue_w(6, ldsw);            VMW(24); read_w(0, wrw, 16384);
        issue_w(7, ldsw + 4096);     VMW(24); read_w(1, wrw, 20480);                          mma(0);
        issue_w(8, ldsw + 8192);     VMW(24); read_w(2, wrw, 24576);
        issue_w(9, ldsw + 12288);    VMW(24); read_w(3, wrw, 28672);                          mma(1);
        issue_w(10, ldsw + 16384);   VMW(24); read_w(4, wrr, 0);
        issue_w(11, ldsw + 20480);   VMW(20); read_w(5, wrr, 16384);  read_w(6, wrw, 0);      mma(2);
        issue_w(12, ldsw + 24576);   VMW(16); read_w(7, wrw, 4096);   read_w(8, wrw, 8192);   mma(3);
        issue_w(13, ldsw + 28672);   VMW(16); read_w(9, wrw, 12288);                          mma(4);
        issue_w(14, ldsr);           VMW(16); read_w(10, wrw, 16384);
        issue_w(15, ldsr + 16384);   VMW(16); read_w(11, wrw, 20480);                         mma(5);
        asm volatile("s_barrier" ::: "memory");
        ovh = ovoff + orow;
        unsigned orow1 = orow + 16 * NSH * 4;
        if (staged) {
            const unsigned call = call_no();
            xin = myws + WS_XST + (size_t)(call & 1u) * WS_XST_PAR + (wave * 1024 - 4096) + (size_t)t0 * (16 * KD);
            xsrd = make_srd(xin, XWIN);
            const int sl = t0 / XT;
            ocnt = XT - (t0 - sl * XT);
            orow1 = (((rank - 1 - sl) & (NR - 1)) * MSH + MSH - ocnt * 16) * (NSH * 4);
            const unsigned long long tw = __builtin_amdgcn_s_memrealtime();
            while (true) {
                unsigned f, tm;
                asm volatile("global_load_dword %0, %2, %3 sc0 sc1\n\t"
                             "global_load_dword %1, %4, %5 sc0 sc1\n\t"
                             "s_waitcnt vmcnt(0)" : "=&v"(f), "=&v"(tm)
                             : "v"((lane < 56 ? lane : 0) * 4), "s"(myws + WS_XFLAG), "v"(0u), "s"(myws + WS_TMO) : "memory");
                if (__builtin_amdgcn_ballot_w64(f < call) == 0ull) break;
                if (first(tm) == call || __builtin_amdgcn_s_memrealtime() - tw > DEADLINE_TICKS) {
                    if (lane == 0) store4_sys(0, call, myws + WS_TMO);
                    poison = __builtin_nanf("");
                    break;
                }
                __builtin_amdgcn_s_sleep(8);
            }
            STAMP(8);
        }
        orow = orow1;
        dma4(ldsw, xo, xsrd, xq[0]);          VMW(12); read_w(12, wrw, 24576); read_w(13, wrw, 28672); mma(6);
        dma4(ldsw + 4096, xo, xsrd, xq[1]);   VMW(8);  read_w(14, wrr, 0);     read_w(15, wrr, 16384); mma(7);
        STAMP(2); CSTAMP(11);
        asm volatile("s_waitcnt lgkmcnt(0)\n\tv_mov_b32 %0, %1\n\tv_mul_f32 %0, %2, %0" : "=v"(scale) : "s"(sclx), "s"(sclw));
        scale += poison;
        dma4(ldsw + 8192, xo, xsrd, xq[2]);
        dma4(ldsw + 12288, xo, xsrd, xq[3]);
        next_x();
        #pragma unroll
        for (int q = 0; q < (8 - SH) / 2; ++q) dma4(ldsw + 16384 + q * 4096, xo, xsrd, xq[q]);
        set_m0(ldsw + 16384 + (8 - SH) / 2 * 4096);
        dma<0>(xo[0], xsrd, xq[(8 - SH) / 2]);
        dma<1024>(xo[1], xsrd, xq[(8 - SH) / 2]);
        VMN(30 - 2 * SH);
        xf[0] = read_xs(xr0, 0);
        xf[1] = read_xs(xr0, 1);
        PIN();
        oh = zero;
        STAMP(13);
    }

    unsigned boff = 0;
    unsigned long long live = T0E ? 0ull : ~0ull;
    #pragma unroll 1
    for (int i = 1; i < nt; ++i) {
        const unsigned lb = ldsw + boff, lo = ldsw + (boff ^ 16384u), xr = xr0 + boff, xn = xr0 + (boff ^ 16384u);
        v4f n[4], p[4];
        float y[4], t[4];
        #define MM(D, S, J, C) do { PIN(); KEEP(xf[(S) & 3]); \
            D = __builtin_amdgcn_mfma_scale_f32_16x16x128_f8f6f4(wf[S][J], xf[(S) & 3], C, 0, 0, 0, 0, 0, 0); \
            asm volatile("" : "+a"(D)); PIN(); } while (0)
        #define SQ(S) (((S) + 9 - SH) & 7)
        #define RD(S) do { VMN(28 - 2 * SH); xf[((S) + 2) & 3] = read_xs((S) < 6 ? xr : xn, ((S) + 2) & 7); \
            if (SQ(S) == 0) { next_x(); if (i + 2 >= nt) xsrd[2] = 0; } \
            if ((SQ(S) & 1) == 0) set_m0(((S) + 1 >= SH ? lb : lo) + SQ(S) * 2048); } while (0)
        #define RQA(S) dma<(SQ(S) & 1) * 2048>(xo[(SQ(S) & 1) * 2], xsrd, xq[SQ(S) >> 1])
        #define RQB(S) dma<(SQ(S) & 1) * 2048 + 1024>(xo[(SQ(S) & 1) * 2 + 1], xsrd, xq[SQ(S) >> 1])
        #define OUT() do { oh = (v4f){y[0], y[1], y[2], y[3]}; \
            asm volatile("s_mov_b64 exec, %3\n\tglobal_store_dwordx4 %0, %1, %2 sc0 sc1\n\ts_mov_b64 exec, -1\n\ts_nop 0" \
                         :: "v"(ovh), "v"(oh), "s"(obase), "s"(live) : "memory"); \
            live = ~0ull; } while (0)
        MM(n[0], 0, 0, zero);  RD(0);  pw(0, boff);
        MM(n[1], 0, 1, zero);  RQA(0); pw(1, boff);
        MM(n[2], 0, 2, zero);          pw(2, boff);
        MM(n[3], 0, 3, zero);  RQB(0); pw(3, boff);
        MM(n[0], 1, 0, n[0]);  RD(1);
        MM(n[1], 1, 1, n[1]);  RQA(1);
        MM(n[2], 1, 2, n[2]);
        MM(n[3], 1, 3, n[3]);  RQB(1);
        MM(n[0], 2, 0, n[0]);  RD(2);
        MM(n[1], 2, 1, n[1]);  RQA(2);
        asm volatile("s_waitcnt lgkmcnt(2)\n\ts_barrier" ::: "memory");
        p[0] = pr(0, boff); p[1] = pr(1, boff);
        MM(n[2], 2, 2, n[2]);
        p[2] = pr(2, boff); p[3] = pr(3, boff);
        MM(n[3], 2, 3, n[3]);  RQB(2);
        MM(n[0], 3, 0, n[0]);  RD(3);
        MM(n[1], 3, 1, n[1]);  RQA(3);
        MM(n[2], 3, 2, n[2]);          y[0] = sum4(p, 0);
        MM(n[3], 3, 3, n[3]);  RQB(3); y[1] = sum4(p, 1);
        MM(n[0], 4, 0, n[0]);  RD(4);  t[0] = act1(y[0], y[0], scale);
        MM(n[1], 4, 1, n[1]);  RQA(4); y[2] = sum4(p, 2);                  t[0] = act2(t[0]);
        MM(n[2], 4, 2, n[2]);          t[1] = act1(y[1], y[1], scale);     HOLD(oh); y[0] = act3(y[0], t[0]);
        MM(n[3], 4, 3, n[3]);  RQB(4); y[3] = sum4(p, 3);                  t[1] = act2(t[1]);
        MM(n[0], 5, 0, n[0]);  RD(5);                                      y[1] = act3(y[1], t[1]);
        MM(n[1], 5, 1, n[1]);  RQA(5); t[2] = act1(y[2], y[2], scale);
        MM(n[2], 5, 2, n[2]);          t[3] = act1(y[3], y[3], scale);     t[2] = act2(t[2]);
        MM(n[3], 5, 3, n[3]);  RQB(5); t[3] = act2(t[3]);                  y[2] = act3(y[2], t[2]);
        MM(n[0], 6, 0, n[0]);  RD(6);  y[3] = act3(y[3], t[3]);
        MM(n[1], 6, 1, n[1]);  RQA(6);
        MM(n[2], 6, 2, n[2]);  OUT();
        MM(n[3], 6, 3, n[3]);  RQB(6);
        MM(acc[0], 7, 0, n[0]);  RD(7);
        MM(acc[1], 7, 1, n[1]);  RQA(7);
        MM(acc[2], 7, 2, n[2]);
        boff ^= 16384u;
        MM(acc[3], 7, 3, n[3]);  RQB(7);
        ovh = ovoff + orow;
        next_o();
    }
    {
        #pragma unroll
        for (int j = 0; j < 4; ++j) pw(j, boff);
        barrier();
        v4f p[4];
        #pragma unroll
        for (int k = 0; k < 4; ++k) p[k] = pr(k, boff);
        store16_sys(ovh, finish(p), obase);
        HOLD(oh);
        STAMP(15 + nt);
    }
    STAMP(3); CSTAMP(12);

    VMW(0);
    STAMP(6);
    barrier();
    asm volatile(".Lpc_end:");
    if (wave == 2) {
        const unsigned call = call_no();
        if (lane == 0) {
            if (rel != 0) store4_sys(0, call, dws + WS_OFLAG + ((rel - 1) * 32 + idx) * 4);
            store4_sys(0, call, cntp);
        }
        STAMP(4);
        if (b == 0) {
            const unsigned fo = (lane < 56 ? lane : 0) * 16;
            const unsigned long long tw = __builtin_amdgcn_s_memrealtime();
            while (true) {
                v4u f;
                unsigned tm;
                asm volatile("global_load_dwordx4 %0, %2, %3 sc0 sc1\n\t"
                             "global_load_dword %1, %4, %3 offset:-2048 sc0 sc1\n\t"
                             "s_waitcnt vmcnt(0)" : "=&v"(f), "=&v"(tm) : "v"(fo), "s"(myws + WS_OFLAG), "v"(0u) : "memory");
                const bool miss = f[0] < call || f[1] < call || f[2] < call || f[3] < call;
                if (__builtin_amdgcn_ballot_w64(miss) == 0ull) break;
                if (first(tm) == call || __builtin_amdgcn_s_memrealtime() - tw > DEADLINE_TICKS) {
                    if (lane == 0) {
                        store4_sys(0, call, myws + WS_TMO);
                        store4_sys(0, 0x7fc00000u, aout);
                    }
                    break;
                }
                __builtin_amdgcn_s_sleep(1);
            }
            __builtin_amdgcn_fence(__ATOMIC_ACQUIRE, "");
            STAMP(5);
        }
    }
}
static_assert(WS_OFLAG - WS_TMO == 2048, "layout");

inline unsigned mkjob(int nt, int t0, int strip, int xg, int delay50, int staged) {
    return (unsigned)nt | (unsigned)t0 << 5 | (unsigned)strip << 10 | (unsigned)xg << 13 | (unsigned)delay50 << 16 |
           (unsigned)staged << 24;
}

#define DLY1 0
#define DLYE 120
#define DLY2 140
#define DLY3 140
#define DLYL 20
#define DLYM 90

unsigned build_jobs(Args& a) {
    unsigned first4 = 0;
    const int n = 32 - XT;
    for (int idx = 0; idx < 32; ++idx) {
        {
            const int strip = (idx >> 4) * 4 + (idx & 3), sub = 3 - ((idx >> 2) & 3);
            const int lo = sub * n / 4, hi = (sub + 1) * n / 4;
            const int dly = idx < 8 ? 0 : idx < 16 ? DLY1 : idx < 20 ? DLYE : idx < 24 ? DLY2 : DLY3;
            a.jobs[0][idx] = mkjob(hi - lo, lo, strip, (idx & 3) * XGA, dly, 0);
            if (idx < 16) first4 |= (unsigned)lo << (5 * sub);
        }
        {
            const int strip = idx & 7, e = (idx >> 3) & 1, ns = 7 * XT;
            if (idx < 16) a.jobs[1][idx] = mkjob(15, e * 15, strip, idx & 7, DLYL, 0);
            else a.jobs[1][idx] = mkjob(1 + (e ? ns - ns / 2 : ns / 2), e ? ns / 2 : 0, strip, idx & 7, DLYM, 1 + 2 * e);
        }
    }
    return first4;
}

}

extern "C" void kernel_launch(int rank, int nranks, ncclComm_t comm,
                              hipStream_t stream,
                              void* const* d_in, const int* in_sizes, int n_in,
                              void* d_out, int out_size,
                              const int* in_dtypes, int out_dtype,
                              void* const* d_out_all,
                              void* const* d_ws_all, size_t ws_bytes) {
    (void)comm; (void)in_dtypes; (void)out_dtype;
    if (nranks != NR || n_in != 4 || (unsigned)rank >= (unsigned)NR || ws_bytes < WS_NEED ||
        in_sizes[0] != MSH * KD || in_sizes[1] != KD * NR * NSH || out_size != NR * MSH * NSH) {
        fprintf(stderr, "kernel_launch: unexpected problem shape\n");
        return;
    }
    Args a;
    memset(&a, 0, sizeof(a));
    const unsigned first4 = build_jobs(a);
    for (int r = 0; r < NR; ++r) {
        const int dest = (rank + r) % NR;
        Rec& c = a.rec[r];
        c.x = (const u8*)d_in[0];
        c.w = (const u8*)d_in[1] + dest * NSH;
        c.out = (u8*)(r ? d_out_all[dest] : d_out);
        c.dws = (u8*)d_ws_all[dest];
        c.myws = (u8*)d_ws_all[rank];
        c.sx = (const float*)d_in[2];
        c.sw = (const float*)d_in[3];
        c.rank = (unsigned)rank;
        c.first4 = r ? first4 : 15u << 5;
    }
    hipLaunchKernelGGL(fused_kernel, dim3(NWG), dim3(NT), 0, stream, a);
    HIP_CHECK(hipGetLastError());
}
```

```cpp
#include <hip/hip_runtime.h>
#include <rccl/rccl.h>
#include <cstdio>
#include <cstring>

#define HIP_CHECK(expr) do { hipError_t _e = (expr); if (_e != hipSuccess) { \
    fprintf(stderr, "HIP error %s at %s:%d\n", hipGetErrorString(_e), __FILE__, __LINE__); \
    return; } } while (0)

namespace {

typedef int v2i __attribute__((ext_vector_type(2)));
typedef int v4i __attribute__((ext_vector_type(4)));
typedef int v8i __attribute__((ext_vector_type(8)));
typedef float v4f __attribute__((ext_vector_type(4)));
typedef unsigned v4u __attribute__((ext_vector_type(4)));
typedef unsigned v16u __attribute__((ext_vector_type(16)));
typedef unsigned char u8;
#define LDS_AS __attribute__((address_space(3)))

constexpr int NR = 8;
constexpr int NWG = 256;
constexpr int NT = 256;
constexpr int KD = 4096;
constexpr int MSH = 512;
constexpr int NSH = 512;
#define CODEPF 1
#define XPF 2
#define XPD 23
#define DLYP 0
#define DLYW 0
#define XGA 2
#define SH 8
#define FINV 0
#define SPARSE 0
#define SPW 1
#define PX 1
#define PJ 4
#define LH 0
#define LXG 1
#define XM_ROWS 64
constexpr int XM = XM_ROWS;
constexpr int XT = XM / 16;
constexpr int XPW = XM / 8;
constexpr unsigned long long DEADLINE_TICKS = 100000000ull;

constexpr int LDS_BUF = 32768;
constexpr int LDS_BYTES = LDS_BUF + 4 * 32768;

constexpr size_t WS_CNT = 0;
constexpr size_t WS_TMO = 2048;
constexpr size_t WS_OFLAG = 4096;
constexpr size_t WS_XFLAG = 8192;
constexpr size_t WS_XST = 65536;
constexpr size_t WS_XST_PAR = (size_t)7 * 64 * KD;
constexpr size_t WS_DBG = WS_XST + 2 * WS_XST_PAR;
constexpr size_t WS_NEED = WS_DBG + 131072;

struct Rec {
    const u8* x;
    const u8* w;
    u8* out;
    u8* dws;
    u8* myws;
    const float* sx;
    const float* sw;
    unsigned rank;
    unsigned first4;
};
struct alignas(64) Args {
    Rec rec[NR];
    unsigned jobs[2][32];
};
static_assert(sizeof(Rec) == 64, "layout");

template <int O>
__device__ __forceinline__ void dma(unsigned v, v4u srd, unsigned s) {
    asm volatile("buffer_load_dwordx4 %0, %1, %2 offen offset:%3 lds" :: "v"(v), "s"(srd), "s"(s), "n"(O) : "memory");
}
template <int O>
__device__ __forceinline__ void dma_after(const v8i& f, unsigned v, v4u srd, unsigned s) {
    asm volatile("buffer_load_dwordx4 %0, %1, %2 offen offset:%3 lds" :: "v"(v), "s"(srd), "s"(s), "n"(O), "v"(f) : "memory");
}
__device__ __forceinline__ void set_m0(unsigned lds) {
    asm volatile("s_mov_b32 m0, %0\n\ts_nop 0" :: "s"(lds) : "memory", "m0");
}
__device__ __forceinline__ void dma4(unsigned lds, const unsigned (&v)[4], v4u srd, unsigned s) {
    set_m0(lds);
    dma<0>(v[0], srd, s); dma<1024>(v[1], srd, s); dma<2048>(v[2], srd, s); dma<3072>(v[3], srd, s);
}
__device__ __forceinline__ void glds1(unsigned lds, unsigned v0, const u8* base) {
    asm volatile("s_mov_b32 m0, %0\n\ts_nop 0\n\tglobal_load_lds_dwordx4 %1, %2" :: "s"(lds), "v"(v0), "s"(base) : "memory", "m0");
}
__device__ __forceinline__ void touch(unsigned lds, unsigned v0, const u8* base) {
    asm volatile("s_mov_b32 m0, %0\n\ts_nop 0\n\tglobal_load_lds_dword %1, %2" :: "s"(lds), "v"(v0), "s"(base) : "memory", "m0");
}
__device__ __forceinline__ v4u make_srd(const u8* p, unsigned bytes) {
    const unsigned long long a = (unsigned long long)p;
    return (v4u){(unsigned)a, (unsigned)(a >> 32) & 0xffffu, bytes, 0x00020000u};
}

#define VMW(n) asm volatile("s_waitcnt vmcnt(" #n ")" ::: "memory")
#define LGKM0() asm volatile("s_waitcnt lgkmcnt(0)" ::: "memory")
#define PIN() __builtin_amdgcn_sched_barrier(0)
#define KEEP(x) asm volatile("" : "+v"(x))
#define HOLD(a) asm volatile("" :: "v"(a))
#define VMN(n) asm volatile("s_waitcnt vmcnt(%0)" :: "n"(n) : "memory")

__device__ __forceinline__ void store16_sys(unsigned voff, v4f v, const void* base) {
    asm volatile("global_store_dwordx4 %0, %1, %2 sc0 sc1\n\ts_nop 1" :: "v"(voff), "v"(v), "s"(base) : "memory");
}
__device__ __forceinline__ void store4_sys(unsigned voff, unsigned v, const void* base) {
    asm volatile("global_store_dword %0, %1, %2 sc0 sc1" :: "v"(voff), "v"(v), "s"(base) : "memory");
}
__device__ __forceinline__ v4f load16(unsigned voff, const void* base) {
    v4f r;
    asm volatile("global_load_dwordx4 %0, %1, %2" : "=v"(r) : "v"(voff), "s"(base) : "memory");
    return r;
}
__device__ __forceinline__ void barrier() {
    asm volatile("s_waitcnt lgkmcnt(0)\n\ts_barrier" ::: "memory");
}
__device__ __forceinline__ unsigned first(unsigned v) { return (unsigned)__builtin_amdgcn_readfirstlane(v); }

__device__ __forceinline__ float act1(float& y, float acc, float s) {
    y = acc * s;
    float t = __builtin_amdgcn_fmed3f(y, -60.0f, 60.0f) * -1.44269504f;
    KEEP(y); KEEP(t);
    return t;
}
__device__ __forceinline__ float act2(float t) {
    float d = __builtin_amdgcn_exp2f(t) + 1.0f;
    KEEP(d);
    return d;
}
__device__ __forceinline__ float act3(float y, float d) {
    float o = y * __builtin_amdgcn_rcpf(d);
    KEEP(o);
    return o;
}

#define STAMP(i) do {} while (0)
#define CSTAMP(i) do {} while (0)

__device__ __forceinline__ u8* ptr(unsigned lo, unsigned hi) {
    return (u8*)(((unsigned long long)hi << 32) | lo);
}

__global__ __launch_bounds__(NT) void fused_kernel(const Args args) {
    __shared__ __attribute__((aligned(1024))) u8 smem[LDS_BYTES];
    const int b = blockIdx.x;
    const int rel = b & 7, idx = (b >> 3) & 31;
    v16u rec;
    unsigned job;
    unsigned long long t_start;
    asm volatile("s_load_dwordx16 %0, %3, %4\n\t"
                 "s_load_dword %1, %3, %5\n\t"
                 "s_memrealtime %2\n\t"
                 "s_waitcnt lgkmcnt(0)"
                 : "=&s"(rec), "=&s"(job), "=&s"(t_start)
                 : "s"(__builtin_amdgcn_kernarg_segment_ptr()), "s"(rel * 64), "s"(512 + (rel ? 0 : 128) + idx * 4) : "memory");
    const u8* const ax = ptr(rec[0], rec[1]);
    const u8* const aw = ptr(rec[2], rec[3]);
    u8* const aout = ptr(rec[4], rec[5]);
    u8* const dws = ptr(rec[6], rec[7]);
    u8* const myws = ptr(rec[8], rec[9]);
    const int rank = rec[14] & (NR - 1);
    const unsigned slot = min((rec[14] >> 8) & 7u, 6u);
    const int tid = threadIdx.x;
    const int lane = tid & 63;
    const int wave = __builtin_amdgcn_readfirstlane(tid >> 6);
    const u8* const cntp = myws + WS_CNT + rel * 128 + idx * 4;
    const unsigned lds0 = (unsigned)(size_t)(LDS_AS u8*)smem;
    const unsigned ldsw = lds0 + LDS_BUF + wave * 32768;
    const unsigned ldsr = lds0 + wave * 4096;
    const int g = lane >> 4, i16 = lane & 15;
    unsigned cnt;
    float sclx, sclw;
    asm volatile("s_load_dword %0, %3, 0x0\n\t"
                 "s_load_dword %1, %4, 0x0\n\t"
                 "s_load_dword %2, %5, 0x0"
                 : "=&s"(cnt), "=&s"(sclx), "=&s"(sclw)
                 : "s"(cntp), "s"(ptr(rec[10], rec[11])), "s"(ptr(rec[12], rec[13])) : "memory");
    auto call_no = [&]() {
        unsigned c;
        asm volatile("s_waitcnt lgkmcnt(0)\n\ts_add_u32 %0, %1, 1" : "=s"(c) : "s"(cnt) : "memory", "scc");
        return c;
    };

    auto wait_until = [&](unsigned ticks) {
        while ((unsigned)__builtin_amdgcn_s_memrealtime() - (unsigned)t_start < ticks) __builtin_amdgcn_s_sleep(1);
    };
    const bool staged = rel == 0 && (job >> 24 & 1u);
    if (staged && wave == 0) __builtin_amdgcn_fence(__ATOMIC_ACQUIRE, "");
    if (idx >= 16) {
        const int hw = (idx - 16) * 4 + wave;
        if (idx == 16 && wave >= 2) {
            const u8* pc;
            unsigned len;
            asm volatile("s_getpc_b64 %0\n.Lpc_here:\n\ts_mov_b32 %1, .Lpc_end-.Lpc_here" : "=s"(pc), "=s"(len));
            unsigned o = ((wave - 2) * 64 + lane) * 128;
            if (o > len - 16) o = len - 16;
            glds1(ldsw + 31 * 1024, o & ~15u, (const u8*)((unsigned long long)pc & ~15ull));
        }
        auto help_x = [&](int n, int hw) {
            #pragma unroll
            for (int j = 0; j < n; ++j) {
                const unsigned tl = (rec[15] >> (5 * j)) & 31u;
                glds1(ldsw + j * 1024, lane * 16, ax + (size_t)tl * (16 * KD) + hw * 1024);
            }
        };
        auto help_w = [&](int j, int hw) {
            const unsigned q = (lane >> 3) & 1;
            glds1(ldsw + 4096 + j * 1024, (((j >> 1 & 3) | (j & 1) << 2 | (j & 8)) ^ (q << 3)) * (64 * KD) + (lane >> 4) * KD + (lane & 15) * 16,
                  aw + (size_t)((hw >> 4) * 1024 + (hw & 15) * 4) * KD);
        };
        auto touch_x = [&](int hw, int n) {
            const unsigned tl = (rec[15] >> (5 * (lane >> 3 & 3))) & 31u;
            if (lane < n) touch(ldsw, tl * (16 * KD) + (lane >> 5) * 32768 + (lane & 7) * 128, ax + hw * 1024);
        };
        auto touch_w = [&](int k, int hw) {
            const unsigned j = lane >> 3, q = lane & 1;
            touch(ldsw, (((j >> 1) | (j & 1) << 2 | (q ^ k) << 3) * 64 + (lane >> 1 & 3)) * KD + q * 128,
                  aw + (size_t)((hw >> 4) * 1024 + (hw & 15) * 4) * KD);
        };
        if (rel == 0) {
            if (DLYP) wait_until(DLYP * 5);
            if (LH) help_x(2, hw);
            const unsigned q = (lane >> 3) & 3;
            #pragma unroll 1
            for (int j = 0; j < LH; ++j)
                glds1(ldsw + j * 1024, ((j >> 1) ^ (q << 2)) * (64 * KD) + (lane >> 5) * KD + (lane & 31) * 16,
                      aw + (size_t)((hw >> 4) * 1024 + (hw & 15) * 4 + (j & 1) * 2) * KD);
            VMW(0);
            STAMP(9);
        } else if (idx >= 24) {
            v4f xv[XPW];
            const unsigned xoff = ((idx & 7) * 4 + wave) * (XPW * 1024) + lane * 16;
            #pragma unroll
            for (int i = 0; i < XPW; ++i) xv[i] = load16(xoff + i * 1024, ax + (size_t)(MSH - XM) * KD);
            if (PX) help_x(4, hw);
            #pragma unroll
            for (int j = 0; j < PJ; ++j) help_w(j, hw);
            const unsigned call = call_no();
            const u8* dst = dws + WS_XST + (size_t)(call & 1u) * WS_XST_PAR + (size_t)slot * (XM * KD);
            VMN(SPARSE ? PX + PJ / 8 : 4 * PX + PJ);
            STAMP(9);
            #pragma unroll
            for (int i = 0; i < XPW; ++i) {
                KEEP(xv[i]);
                store16_sys(xoff + i * 1024, xv[i], dst);
            }
            VMW(0);
            barrier();
            if (tid == 0) store4_sys(0, call, dws + WS_XFLAG + (slot * 8 + (idx & 7)) * 4);
            STAMP(1);
        } else {
            help_x(4, hw);
            if (!PX) help_x(4, hw + 32);
            #pragma unroll
            for (int j = 0; j < 16; ++j) {
                help_w(j, hw);
                if (j >= PJ) help_w(j, hw + 32);
            }
            VMW(8);
            STAMP(9);
            #pragma unroll 1
            for (int j = 0; j < XPF * 8; ++j) {
                unsigned tl = ((rec[15] >> (5 * ((j >> 1) & 3))) & 31u) + 1 + (j >> 3);
                if (tl > 31) tl = 31;
                glds1(ldsw + (j & 31) * 1024, lane * 16, ax + (size_t)tl * (16 * KD) + (hw * 2 + (j & 1)) * 1024);
                asm volatile("s_waitcnt vmcnt(%0)" :: "n"(XPD) : "memory");
            }
            if (DLYW) wait_until(DLYW * 5);
            #pragma unroll 1
            for (int j = 0; j < 32; ++j) {
                glds1(ldsw + j * 1024, (lane >> 4) * KD + (lane & 15) * 16, aw + (size_t)(j * 128 + hw * 4) * KD + 256);
                asm volatile("s_waitcnt vmcnt(%0)" :: "n"(XPD) : "memory");
            }
            VMW(0);
            STAMP(1);
        }
    }

    int nt = job & 31u;
    if (nt < 5) nt = 5;
    int t0 = (job >> 5) & 31u;
    if (t0 + nt > (staged ? 7 * XT + 1 : 32)) { t0 = 0; nt = 5; }
    const int strip = (job >> 10) & 7;
    const int xg = (job >> 13) & 7;
    const unsigned delay = ((job >> 16) & 255u) * 5u;

    unsigned wo[4];
    {
        const int row = lane >> 2, pos = lane & 3;
        #pragma unroll
        for (int p = 0; p < 4; ++p)
            wo[p] = (16 * p + row) * KD + ((pos ^ (((row >> 2) & 1) | ((p & 1) << 1))) << 4) - p * 1024;
    }
    unsigned xo[4];
    #pragma unroll
    for (int p = 0; p < 4; ++p) {
        const int m = 8 * (p & 1) + (lane >> 3);
        const int cs = (lane & 7) ^ (m >> 1);
        xo[p] = m * KD + ((((p >> 1) ^ (xg & 1)) << 7) + (cs << 4)) - p * 1024 + 4096;
    }
    unsigned xq[4];
    #pragma unroll
    for (int q = 0; q < 4; ++q) xq[q] = (q ^ (xg >> 1)) << 8;
    const v4u wsrd = make_srd(aw + (size_t)(wave * 1024) * KD + strip * 64, 1024 * KD);
    constexpr unsigned XWIN = 17 * KD + 4096;
    const int tf = staged ? 30 + (job >> 25 & 1u) : t0;
    unsigned orow = (rank * MSH + tf * 16) * (NSH * 4);
    int ocnt = 32;
    float poison = 0.f;
    const u8* xin = ax + (wave * 1024 - 4096) + (size_t)tf * (16 * KD);
    v4u xsrd = make_srd(xin, XWIN);
    auto next_x = [&]() { xin += 16 * KD; xsrd = make_srd(xin, XWIN); };
    auto next_o = [&]() {
        orow += 16 * NSH * 4;
        if (--ocnt == 0) { orow = (orow - (MSH + XM) * NSH * 4) & (NR * MSH * NSH * 4 - 1); ocnt = XT; }
    };
    auto issue_w = [&](int h, unsigned unit) { dma4(unit, wo, wsrd, (unsigned)(h ^ (xg << 1)) * (64 * KD)); };

    if (delay) {
        wait_until(delay);
        STAMP(7); CSTAMP(10);
    }

    #pragma unroll
    for (int q = 0; q < 4; ++q) dma4(ldsw + q * 4096, xo, xsrd, xq[q]);
    next_x();
    issue_w(0, ldsw + 16384);
    issue_w(1, ldsw + 20480);
    asm volatile("s_barrier" ::: "memory");
    issue_w(2, ldsw + 24576);
    issue_w(3, ldsw + 28672);
    issue_w(4, ldsr);
    issue_w(5, ldsr + 16384);
    PIN();
    STAMP(14);
    const LDS_AS u8* wrw[4];
    const LDS_AS u8* wrr[4];
    {
        const int q = i16 >> 1, p = i16 & 1;
        const int f = ((q >> 2) & 1) | ((g & 1) << 1);
        #pragma unroll
        for (int j = 0; j < 4; ++j) {
            const unsigned o = g * 1024 + q * 64 + ((j ^ f) << 4) + 8 * p;
            wrw[j] = (const LDS_AS u8*)(size_t)(ldsw + o);
            wrr[j] = (const LDS_AS u8*)(size_t)(ldsr + o);
        }
    }
    const unsigned xr0 = ldsw + i16 * 128 + ((g ^ (i16 >> 1)) << 4);
    const int om = 4 * wave + g;
    const unsigned ovoff = om * (NSH * 4) + i16 * 16;
    const unsigned prd0 = lds0 + om * 256 + ((i16 ^ om) << 4);
    unsigned pwr0[4];
    #pragma unroll
    for (int j = 0; j < 4; ++j) pwr0[j] = ldsr + i16 * 256 + (((4 * j + g) ^ i16) << 4);
    v8i wf[8][4], xf[4];
    v4f acc[4];
    float scale;
    v4f oh;
    unsigned ovh;
    auto read_xs = [&](unsigned ad, int s) {
        const v4i lo = *(const LDS_AS v4i*)((const LDS_AS u8*)(size_t)ad + s * 2048);
        const v4i hi = *(const LDS_AS v4i*)((const LDS_AS u8*)(size_t)(ad ^ 64u) + s * 2048);
        return (v8i){lo[0], lo[1], lo[2], lo[3], hi[0], hi[1], hi[2], hi[3]};
    };
    auto read_w = [&](int h, const LDS_AS u8* const* base, int off) {
        #pragma unroll
        for (int j = 0; j < 4; ++j) {
            #pragma unroll
            for (int r = 0; r < 2; ++r) {
                const v2i t = __builtin_amdgcn_ds_read_tr8_b64_v2i32((LDS_AS v2i*)(base[j] + off + r * 512));
                wf[h >> 1][j][(h & 1) * 4 + 2 * r] = t[0];
                wf[h >> 1][j][(h & 1) * 4 + 2 * r + 1] = t[1];
            }
        }
        LGKM0();
        STAMP(42 + h);
        if (h & 1) {
            #pragma unroll
            for (int j = 0; j < 4; ++j) {
                if (h < 12) asm volatile("" : "+a"(wf[h >> 1][j]));
                else asm volatile("" : "+v"(wf[h >> 1][j]));
            }
        }
    };
    const v4f zero = {0.f, 0.f, 0.f, 0.f};
    const u8* const obase = aout + strip * 256;
    auto pw = [&](int j, unsigned bo) { *(LDS_AS v4f*)(size_t)(pwr0[j] + bo) = acc[j]; };
    auto pr = [&](int k, unsigned bo) { return *(const LDS_AS v4f*)((const LDS_AS u8*)(size_t)(prd0 + bo) + k * 4096); };
    auto sum4 = [&](const v4f (&p)[4], int e) {
        float a = p[0][e] + p[1][e], c = p[2][e] + p[3][e];
        KEEP(a); KEEP(c);
        float s = a + c;
        KEEP(s);
        return s;
    };
    auto finish = [&](const v4f (&p)[4]) {
        float y[4];
        #pragma unroll
        for (int e = 0; e < 4; ++e) {
            const float t = act1(y[e], sum4(p, e), scale);
            y[e] = act3(y[e], act2(t));
        }
        return (v4f){y[0], y[1], y[2], y[3]};
    };
    {
        v8i x0[8];
        auto mma = [&](int s) {
            PIN();
            #pragma unroll
            for (int j = 0; j < 4; ++j)
                acc[j] = __builtin_amdgcn_mfma_scale_f32_16x16x128_f8f6f4(wf[s][j], x0[s], s ? acc[j] : zero, 0, 0, 0, 0, 0, 0);
            PIN();
        };
        STAMP(40);
        VMW(24);
        STAMP(41);
        #pragma unroll
        for (int s = 0; s < 8; ++s) x0[s] = read_xs(xr0, s);
        LGKM0();
        issue_w(6, ldsw);            VMW(24); read_w(0, wrw, 16384);
        issue_w(7, ldsw + 4096);     VMW(24); read_w(1, wrw, 20480);                          mma(0);
        issue_w(8, ldsw + 8192);     VMW(24); read_w(2, wrw, 24576);
        issue_w(9, ldsw + 12288);    VMW(24); read_w(3, wrw, 28672);                          mma(1);
        issue_w(10, ldsw + 16384);   VMW(24); read_w(4, wrr, 0);
        issue_w(11, ldsw + 20480);   VMW(20); read_w(5, wrr, 16384);  read_w(6, wrw, 0);      mma(2);
        issue_w(12, ldsw + 24576);   VMW(16); read_w(7, wrw, 4096);   read_w(8, wrw, 8192);   mma(3);
        issue_w(13, ldsw + 28672);   VMW(16); read_w(9, wrw, 12288);                          mma(4);
        issue_w(14, ldsr);           VMW(16); read_w(10, wrw, 16384);
        issue_w(15, ldsr + 16384);   VMW(16); read_w(11, wrw, 20480);                         mma(5);
        asm volatile("s_barrier" ::: "memory");
        ovh = ovoff + orow;
        unsigned orow1 = orow + 16 * NSH * 4;
        if (staged) {
            const unsigned call = call_no();
            xin = myws + WS_XST + (size_t)(call & 1u) * WS_XST_PAR + (wave * 1024 - 4096) + (size_t)t0 * (16 * KD);
            xsrd = make_srd(xin, XWIN);
            const int sl = t0 / XT;
            ocnt = XT - (t0 - sl * XT);
            orow1 = (((rank - 1 - sl) & (NR - 1)) * MSH + MSH - ocnt * 16) * (NSH * 4);
            const unsigned long long tw = __builtin_amdgcn_s_memrealtime();
            while (true) {
                unsigned f, tm;
                asm volatile("global_load_dword %0, %2, %3 sc0 sc1\n\t"
                             "global_load_dword %1, %4, %5 sc0 sc1\n\t"
                             "s_waitcnt vmcnt(0)" : "=&v"(f), "=&v"(tm)
                             : "v"((lane < 56 ? lane : 0) * 4), "s"(myws + WS_XFLAG), "v"(0u), "s"(myws + WS_TMO) : "memory");
                if (__builtin_amdgcn_ballot_w64(f < call) == 0ull) break;
                if (first(tm) == call || __builtin_amdgcn_s_memrealtime() - tw > DEADLINE_TICKS) {
                    if (lane == 0) store4_sys(0, call, myws + WS_TMO);
                    poison = __builtin_nanf("");
                    break;
                }
                __builtin_amdgcn_s_sleep(8);
            }
            STAMP(8);
        }
        orow = orow1;
        dma4(ldsw, xo, xsrd, xq[0]);          VMW(12); read_w(12, wrw, 24576); read_w(13, wrw, 28672); mma(6);
        dma4(ldsw + 4096, xo, xsrd, xq[1]);   VMW(8);  read_w(14, wrr, 0);     read_w(15, wrr, 16384); mma(7);
        STAMP(2); CSTAMP(11);
        asm volatile("s_waitcnt lgkmcnt(0)\n\tv_mov_b32 %0, %1\n\tv_mul_f32 %0, %2, %0" : "=v"(scale) : "s"(sclx), "s"(sclw));
        scale += poison;
        dma4(ldsw + 8192, xo, xsrd, xq[2]);
        dma4(ldsw + 12288, xo, xsrd, xq[3]);
        next_x();
        #pragma unroll
        for (int q = 0; q < (8 - SH) / 2; ++q) dma4(ldsw + 16384 + q * 4096, xo, xsrd, xq[q]);
        set_m0(ldsw + 16384 + (8 - SH) / 2 * 4096);
        dma<0>(xo[0], xsrd, xq[(8 - SH) / 2]);
        dma<1024>(xo[1], xsrd, xq[(8 - SH) / 2]);
        VMN(30 - 2 * SH);
        xf[0] = read_xs(xr0, 0);
        xf[1] = read_xs(xr0, 1);
        PIN();
        oh = zero;
        STAMP(13);
    }

    unsigned boff = 0;
    #pragma unroll 1
    for (int i = 1; i < nt; ++i) {
        const unsigned lb = ldsw + boff, lo = ldsw + (boff ^ 16384u), xr = xr0 + boff, xn = xr0 + (boff ^ 16384u);
        v4f n[4], p[4];
        float y[4], t[4];
        #define MM(D, S, J, C) do { PIN(); KEEP(xf[(S) & 3]); \
            D = __builtin_amdgcn_mfma_scale_f32_16x16x128_f8f6f4(wf[S][J], xf[(S) & 3], C, 0, 0, 0, 0, 0, 0); \
            asm volatile("" : "+a"(D)); PIN(); } while (0)
        #define SQ(S) (((S) + 9 - SH) & 7)
        #define RD(S) do { VMN(28 - 2 * SH); xf[((S) + 2) & 3] = read_xs((S) < 6 ? xr : xn, ((S) + 2) & 7); \
            if (SQ(S) == 0) { next_x(); if (i + 2 >= nt) xsrd[2] = 0; } \
            if ((SQ(S) & 1) == 0) set_m0(((S) + 1 >= SH ? lb : lo) + SQ(S) * 2048); } while (0)
        #define RQA(S) dma<(SQ(S) & 1) * 2048>(xo[(SQ(S) & 1) * 2], xsrd, xq[SQ(S) >> 1])
        #define RQB(S) dma<(SQ(S) & 1) * 2048 + 1024>(xo[(SQ(S) & 1) * 2 + 1], xsrd, xq[SQ(S) >> 1])
        #define OUT() do { oh = (v4f){y[0], y[1], y[2], y[3]}; store16_sys(ovh, oh, obase); } while (0)
        MM(n[0], 0, 0, zero);  RD(0);  pw(0, boff);
        MM(n[1], 0, 1, zero);  RQA(0); pw(1, boff);
        MM(n[2], 0, 2, zero);          pw(2, boff);
        MM(n[3], 0, 3, zero);  RQB(0); pw(3, boff);
        MM(n[0], 1, 0, n[0]);  RD(1);
        MM(n[1], 1, 1, n[1]);  RQA(1);
        MM(n[2], 1, 2, n[2]);
        MM(n[3], 1, 3, n[3]);  RQB(1);
        MM(n[0], 2, 0, n[0]);  RD(2);
        MM(n[1], 2, 1, n[1]);  RQA(2);
        asm volatile("s_waitcnt lgkmcnt(2)\n\ts_barrier" ::: "memory");
        p[0] = pr(0, boff); p[1] = pr(1, boff);
        MM(n[2], 2, 2, n[2]);
        p[2] = pr(2, boff); p[3] = pr(3, boff);
        MM(n[3], 2, 3, n[3]);  RQB(2);
        MM(n[0], 3, 0, n[0]);  RD(3);
        MM(n[1], 3, 1, n[1]);  RQA(3);
        MM(n[2], 3, 2, n[2]);          y[0] = sum4(p, 0);
        MM(n[3], 3, 3, n[3]);  RQB(3); y[1] = sum4(p, 1);
        MM(n[0], 4, 0, n[0]);  RD(4);  t[0] = act1(y[0], y[0], scale);
        MM(n[1], 4, 1, n[1]);  RQA(4); y[2] = sum4(p, 2);                  t[0] = act2(t[0]);
        MM(n[2], 4, 2, n[2]);          t[1] = act1(y[1], y[1], scale);     HOLD(oh); y[0] = act3(y[0], t[0]);
        MM(n[3], 4, 3, n[3]);  RQB(4); y[3] = sum4(p, 3);                  t[1] = act2(t[1]);
        MM(n[0], 5, 0, n[0]);  RD(5);                                      y[1] = act3(y[1], t[1]);
        MM(n[1], 5, 1, n[1]);  RQA(5); t[2] = act1(y[2], y[2], scale);
        MM(n[2], 5, 2, n[2]);          t[3] = act1(y[3], y[3], scale);     t[2] = act2(t[2]);
        MM(n[3], 5, 3, n[3]);  RQB(5); t[3] = act2(t[3]);                  y[2] = act3(y[2], t[2]);
        MM(n[0], 6, 0, n[0]);  RD(6);  y[3] = act3(y[3], t[3]);
        MM(n[1], 6, 1, n[1]);  RQA(6);
        MM(n[2], 6, 2, n[2]);  OUT();
        MM(n[3], 6, 3, n[3]);  RQB(6);
        MM(acc[0], 7, 0, n[0]);  RD(7);
        MM(acc[1], 7, 1, n[1]);  RQA(7);
        MM(acc[2], 7, 2, n[2]);
        boff ^= 16384u;
        MM(acc[3], 7, 3, n[3]);  RQB(7);
        ovh = ovoff + orow;
        next_o();
    }
    {
        #pragma unroll
        for (int j = 0; j < 4; ++j) pw(j, boff);
        barrier();
        v4f p[4];
        #pragma unroll
        for (int k = 0; k < 4; ++k) p[k] = pr(k, boff);
        store16_sys(ovh, finish(p), obase);
        HOLD(oh);
        STAMP(15 + nt);
    }
    STAMP(3); CSTAMP(12);

    VMW(0);
    STAMP(6);
    barrier();
    asm volatile(".Lpc_end:");
    if (wave == 2) {
        const unsigned call = call_no();
        if (lane == 0) {
            if (rel != 0) store4_sys(0, call, dws + WS_OFLAG + (slot * 32 + idx) * 4);
            store4_sys(0, call, cntp);
        }
        STAMP(4);
        if (b == 0) {
            const unsigned fo = (lane < 56 ? lane : 0) * 16;
            const unsigned long long tw = __builtin_amdgcn_s_memrealtime();
            while (true) {
                v4u f;
                unsigned tm;
                asm volatile("global_load_dwordx4 %0, %2, %3 sc0 sc1\n\t"
                             "global_load_dword %1, %4, %3 offset:-2048 sc0 sc1\n\t"
                             "s_waitcnt vmcnt(0)" : "=&v"(f), "=&v"(tm) : "v"(fo), "s"(myws + WS_OFLAG), "v"(0u) : "memory");
                const bool miss = f[0] < call || f[1] < call || f[2] < call || f[3] < call;
                if (__builtin_amdgcn_ballot_w64(miss) == 0ull) break;
                if (first(tm) == call || __builtin_amdgcn_s_memrealtime() - tw > DEADLINE_TICKS) {
                    if (lane == 0) {
                        store4_sys(0, call, myws + WS_TMO);
                        store4_sys(0, 0x7fc00000u, aout);
                    }
                    break;
                }
                __builtin_amdgcn_s_sleep(1);
            }
            STAMP(5);
        }
    }
}
static_assert(WS_OFLAG - WS_TMO == 2048, "layout");

inline unsigned mkjob(int nt, int t0, int strip, int xg, int delay50, int staged) {
    return (unsigned)nt | (unsigned)t0 << 5 | (unsigned)strip << 10 | (unsigned)xg << 13 | (unsigned)delay50 << 16 |
           (unsigned)staged << 24;
}

#define GROUP 0xb2
#define DLY1 0
#define DLYE 120
#define DLY2 140
#define DLY3 140
#define DLYL 20
#define DLYM 20

unsigned build_jobs(Args& a) {
    unsigned first4 = 0;
    const int n = 32 - XT;
    for (int idx = 0; idx < 32; ++idx) {
        {
            const int strip = (idx >> 4) * 4 + (idx & 3), sub = 3 - ((idx >> 2) & 3);
            const int lo = sub * n / 4, hi = (sub + 1) * n / 4;
            const int dly = idx < 8 ? 0 : idx < 16 ? DLY1 : idx < 20 ? DLYE : idx < 24 ? DLY2 : DLY3;
            a.jobs[0][idx] = mkjob(hi - lo, lo, strip, (idx & 3) * XGA, dly, 0);
            if (idx < 16) first4 |= (unsigned)lo << (5 * sub);
        }
        {
            const int strip = idx & 7, e = (idx >> 3) & 1, ns = 7 * XT;
            const int xg = LXG ? (idx >> 3) * 2 + (idx & 1) : idx & 7;
            if (idx < 16) a.jobs[1][idx] = mkjob(15, e * 15, strip, xg, DLYL, 0);
            else a.jobs[1][idx] = mkjob(1 + (e ? ns - ns / 2 : ns / 2), e ? ns / 2 : 0, strip, xg, DLYM, 1 + 2 * e);
        }
    }
    return first4;
}

}

extern "C" void kernel_launch(int rank, int nranks, ncclComm_t comm,
                              hipStream_t stream,
                              void* const* d_in, const int* in_sizes, int n_in,
                              void* d_out, int out_size,
                              const int* in_dtypes, int out_dtype,
                              void* const* d_out_all,
                              void* const* d_ws_all, size_t ws_bytes) {
    (void)comm; (void)in_dtypes; (void)out_dtype;
    if (nranks != NR || n_in != 4 || (unsigned)rank >= (unsigned)NR || ws_bytes < WS_NEED ||
        in_sizes[0] != MSH * KD || in_sizes[1] != KD * NR * NSH || out_size != NR * MSH * NSH) {
        fprintf(stderr, "kernel_launch: unexpected problem shape\n");
        return;
    }
    Args a;
    memset(&a, 0, sizeof(a));
    const unsigned first4 = build_jobs(a);
    int order[NR], n = 0;
    for (int pass = 0; pass < 2; ++pass)
        for (int r = 0; r < NR; ++r) {
            const int d = (rank + r) % NR;
            const bool same = ((GROUP >> d) & 1) == ((GROUP >> rank) & 1);
            if (same == (pass == 0)) order[n++] = d;
        }
    for (int r = 0; r < NR; ++r) {
        const int dest = order[r];
        Rec& c = a.rec[r];
        c.x = (const u8*)d_in[0];
        c.w = (const u8*)d_in[1] + dest * NSH;
        c.out = (u8*)(r ? d_out_all[dest] : d_out);
        c.dws = (u8*)d_ws_all[dest];
        c.myws = (u8*)d_ws_all[rank];
        c.sx = (const float*)d_in[2];
        c.sw = (const float*)d_in[3];
        c.rank = (unsigned)rank | (unsigned)((dest - rank - 1) & (NR - 1)) << 8;
        c.first4 = r ? first4 : 15u << 5;
    }
    hipLaunchKernelGGL(fused_kernel, dim3(NWG), dim3(NT), 0, stream, a);
    HIP_CHECK(hipGetLastError());
}
```

```cpp
#include <hip/hip_runtime.h>
#include <rccl/rccl.h>
#include <cstdio>
#include <cstring>

#define HIP_CHECK(expr) do { hipError_t _e = (expr); if (_e != hipSuccess) { \
    fprintf(stderr, "HIP error %s at %s:%d\n", hipGetErrorString(_e), __FILE__, __LINE__); \
    return; } } while (0)

namespace {

typedef int v2i __attribute__((ext_vector_type(2)));
typedef int v4i __attribute__((ext_vector_type(4)));
typedef int v8i __attribute__((ext_vector_type(8)));
typedef float v4f __attribute__((ext_vector_type(4)));
typedef unsigned v4u __attribute__((ext_vector_type(4)));
typedef unsigned v16u __attribute__((ext_vector_type(16)));
typedef unsigned char u8;
#define LDS_AS __attribute__((address_space(3)))

constexpr int NR = 8;
constexpr int NWG = 256;
constexpr int NT = 256;
constexpr int KD = 4096;
constexpr int MSH = 512;
constexpr int NSH = 512;
#define CODEPF 1
#define XPF 2
#define XPD 23
#define DLYP 0
#define DLYW 0
#define XGA 2
#define SH 8
#define FINV 0
#define SPARSE 0
#define SPW 1
#define PX 1
#define PJ 4
#define EARLYX 1
#define STEP 1
#define LH 0
#define LXG 1
#define XM_ROWS 64
constexpr int XM = XM_ROWS;
constexpr int XT = XM / 16;
constexpr int XPW = XM / 8;
constexpr unsigned long long DEADLINE_TICKS = 100000000ull;

constexpr int LDS_BUF = 32768;
constexpr int LDS_BYTES = LDS_BUF + 4 * 32768;

constexpr size_t WS_CNT = 0;
constexpr size_t WS_TMO = 2048;
constexpr size_t WS_OFLAG = 4096;
constexpr size_t WS_XFLAG = 8192;
constexpr size_t WS_XST = 65536;
constexpr size_t WS_XST_PAR = (size_t)7 * 64 * KD;
constexpr size_t WS_DBG = WS_XST + 2 * WS_XST_PAR;
constexpr size_t WS_NEED = WS_DBG + 131072;

struct Rec {
    const u8* x;
    const u8* w;
    u8* out;
    u8* dws;
    u8* myws;
    const float* sx;
    const float* sw;
    unsigned rank;
    unsigned first4;
};
struct alignas(64) Args {
    Rec rec[NR];
    unsigned jobs[2][32];
};
static_assert(sizeof(Rec) == 64, "layout");

template <int O>
__device__ __forceinline__ void dma(unsigned v, v4u srd, unsigned s) {
    asm volatile("buffer_load_dwordx4 %0, %1, %2 offen offset:%3 lds" :: "v"(v), "s"(srd), "s"(s), "n"(O) : "memory");
}
template <int O>
__device__ __forceinline__ void dma_after(const v8i& f, unsigned v, v4u srd, unsigned s) {
    asm volatile("buffer_load_dwordx4 %0, %1, %2 offen offset:%3 lds" :: "v"(v), "s"(srd), "s"(s), "n"(O), "v"(f) : "memory");
}
__device__ __forceinline__ void set_m0(unsigned lds) {
    asm volatile("s_mov_b32 m0, %0\n\ts_nop 0" :: "s"(lds) : "memory", "m0");
}
__device__ __forceinline__ void dma4(unsigned lds, const unsigned (&v)[4], v4u srd, unsigned s) {
    set_m0(lds);
    dma<0>(v[0], srd, s); dma<1024>(v[1], srd, s); dma<2048>(v[2], srd, s); dma<3072>(v[3], srd, s);
}
__device__ __forceinline__ void glds1(unsigned lds, unsigned v0, const u8* base) {
    asm volatile("s_mov_b32 m0, %0\n\ts_nop 0\n\tglobal_load_lds_dwordx4 %1, %2" :: "s"(lds), "v"(v0), "s"(base) : "memory", "m0");
}
__device__ __forceinline__ void touch(unsigned lds, unsigned v0, const u8* base) {
    asm volatile("s_mov_b32 m0, %0\n\ts_nop 0\n\tglobal_load_lds_dword %1, %2" :: "s"(lds), "v"(v0), "s"(base) : "memory", "m0");
}
__device__ __forceinline__ v4u make_srd(const u8* p, unsigned bytes) {
    const unsigned long long a = (unsigned long long)p;
    return (v4u){(unsigned)a, (unsigned)(a >> 32) & 0xffffu, bytes, 0x00020000u};
}

#define VMW(n) asm volatile("s_waitcnt vmcnt(" #n ")" ::: "memory")
#define LGKM0() asm volatile("s_waitcnt lgkmcnt(0)" ::: "memory")
#define PIN() __builtin_amdgcn_sched_barrier(0)
#define KEEP(x) asm volatile("" : "+v"(x))
#define HOLD(a) asm volatile("" :: "v"(a))
#define VMN(n) asm volatile("s_waitcnt vmcnt(%0)" :: "n"(n) : "memory")

__device__ __forceinline__ void store16_sys(unsigned voff, v4f v, const void* base) {
    asm volatile("global_store_dwordx4 %0, %1, %2 sc0 sc1\n\ts_nop 1" :: "v"(voff), "v"(v), "s"(base) : "memory");
}
__device__ __forceinline__ void store4_sys(unsigned voff, unsigned v, const void* base) {
    asm volatile("global_store_dword %0, %1, %2 sc0 sc1" :: "v"(voff), "v"(v), "s"(base) : "memory");
}
__device__ __forceinline__ v4f load16(unsigned voff, const void* base) {
    v4f r;
    asm volatile("global_load_dwordx4 %0, %1, %2" : "=v"(r) : "v"(voff), "s"(base) : "memory");
    return r;
}
__device__ __forceinline__ void barrier() {
    asm volatile("s_waitcnt lgkmcnt(0)\n\ts_barrier" ::: "memory");
}
__device__ __forceinline__ unsigned first(unsigned v) { return (unsigned)__builtin_amdgcn_readfirstlane(v); }

__device__ __forceinline__ float act1(float& y, float acc, float s) {
    y = acc * s;
    float t = __builtin_amdgcn_fmed3f(y, -60.0f, 60.0f) * -1.44269504f;
    KEEP(y); KEEP(t);
    return t;
}
__device__ __forceinline__ float act2(float t) {
    float d = __builtin_amdgcn_exp2f(t) + 1.0f;
    KEEP(d);
    return d;
}
__device__ __forceinline__ float act3(float y, float d) {
    float o = y * __builtin_amdgcn_rcpf(d);
    KEEP(o);
    return o;
}

#define ESTAMP(i) do {} while (0)
#define STAMP(i) do {} while (0)
#define CSTAMP(i) do {} while (0)

__device__ __forceinline__ u8* ptr(unsigned lo, unsigned hi) {
    return (u8*)(((unsigned long long)hi << 32) | lo);
}

__global__ __launch_bounds__(NT) void fused_kernel(const Args args) {
    __shared__ __attribute__((aligned(1024))) u8 smem[LDS_BYTES];
    const int b = blockIdx.x;
    const int rel = b & 7, idx = (b >> 3) & 31;
    v16u rec;
    unsigned job;
    unsigned long long t_start;
    asm volatile("s_load_dwordx16 %0, %3, %4\n\t"
                 "s_load_dword %1, %3, %5\n\t"
                 "s_memrealtime %2\n\t"
                 "s_waitcnt lgkmcnt(0)"
                 : "=&s"(rec), "=&s"(job), "=&s"(t_start)
                 : "s"(__builtin_amdgcn_kernarg_segment_ptr()), "s"(rel * 64), "s"(512 + (rel ? 0 : 128) + idx * 4) : "memory");
    const u8* const ax = ptr(rec[0], rec[1]);
    const u8* const aw = ptr(rec[2], rec[3]);
    u8* const aout = ptr(rec[4], rec[5]);
    u8* const dws = ptr(rec[6], rec[7]);
    u8* const myws = ptr(rec[8], rec[9]);
    const int rank = rec[14] & (NR - 1);
    const unsigned slot = min((rec[14] >> 8) & 7u, 6u);
    const int tid = threadIdx.x;
    const int lane = tid & 63;
    const int wave = __builtin_amdgcn_readfirstlane(tid >> 6);
    const u8* const cntp = myws + WS_CNT + rel * 128 + idx * 4;
    const unsigned lds0 = (unsigned)(size_t)(LDS_AS u8*)smem;
    const unsigned ldsw = lds0 + LDS_BUF + wave * 32768;
    const unsigned ldsr = lds0 + wave * 4096;
    const int g = lane >> 4, i16 = lane & 15;
    v4f xv[XPW];
    const unsigned xoff = ((idx & 7) * 4 + wave) * (XPW * 1024) + lane * 16;
    static_assert(XPW == 8, "eight pieces");
    asm volatile("s_cmp_lt_u32 %9, 24\n\t"
                 "s_cbranch_scc1 .Lno_rows\n\t"
                 "global_load_dwordx4 %0, %8, %10 offset:-4096\n\t"
                 "global_load_dwordx4 %1, %8, %10 offset:-3072\n\t"
                 "global_load_dwordx4 %2, %8, %10 offset:-2048\n\t"
                 "global_load_dwordx4 %3, %8, %10 offset:-1024\n\t"
                 "global_load_dwordx4 %4, %8, %10\n\t"
                 "global_load_dwordx4 %5, %8, %10 offset:1024\n\t"
                 "global_load_dwordx4 %6, %8, %10 offset:2048\n\t"
                 "global_load_dwordx4 %7, %8, %10 offset:3072\n"
                 ".Lno_rows:"
                 : "=&v"(xv[0]), "=&v"(xv[1]), "=&v"(xv[2]), "=&v"(xv[3]), "=&v"(xv[4]), "=&v"(xv[5]), "=&v"(xv[6]), "=&v"(xv[7])
                 : "v"(xoff + ((MSH - XM) * KD + 4096)), "s"(rel ? idx : 0), "s"(ax) : "memory", "scc");
    unsigned cnt;
    float sclx, sclw;
    asm volatile("s_load_dword %0, %3, 0x0\n\t"
                 "s_load_dword %1, %4, 0x0\n\t"
                 "s_load_dword %2, %5, 0x0"
                 : "=&s"(cnt), "=&s"(sclx), "=&s"(sclw)
                 : "s"(cntp), "s"(ptr(rec[10], rec[11])), "s"(ptr(rec[12], rec[13])) : "memory");
    auto call_no = [&]() {
        unsigned c;
        asm volatile("s_waitcnt lgkmcnt(0)\n\ts_add_u32 %0, %1, 1" : "=s"(c) : "s"(cnt) : "memory", "scc");
        return c;
    };

    auto wait_until = [&](unsigned ticks) {
        while ((unsigned)__builtin_amdgcn_s_memrealtime() - (unsigned)t_start < ticks) __builtin_amdgcn_s_sleep(1);
    };
    const bool staged = rel == 0 && (job >> 24 & 1u);
    if (staged && wave == 0) __builtin_amdgcn_fence(__ATOMIC_ACQUIRE, "");
    if (idx >= 16) {
        const int hw = (idx - 16) * 4 + wave;
        if (idx == 16 && wave >= 2) {
            const u8* pc;
            unsigned len;
            asm volatile("s_getpc_b64 %0\n.Lpc_here:\n\ts_mov_b32 %1, .Lpc_end-.Lpc_here" : "=s"(pc), "=s"(len));
            unsigned o = ((wave - 2) * 64 + lane) * 128;
            if (o > len - 16) o = len - 16;
            glds1(ldsw + 31 * 1024, o & ~15u, (const u8*)((unsigned long long)pc & ~15ull));
        }
        auto help_x = [&](int n, int hw) {
            #pragma unroll
            for (int j = 0; j < n; ++j) {
                const unsigned tl = (rec[15] >> (5 * j)) & 31u;
                glds1(ldsw + j * 1024, lane * 16, ax + (size_t)tl * (16 * KD) + hw * 1024);
            }
        };
        auto help_w = [&](int j, int hw) {
            const unsigned q = (lane >> 3) & 1;
            glds1(ldsw + 4096 + j * 1024, (((j >> 1 & 3) | (j & 1) << 2 | (j & 8)) ^ (q << 3)) * (64 * KD) + (lane >> 4) * KD + (lane & 15) * 16,
                  aw + (size_t)((hw >> 4) * 1024 + (hw & 15) * 4) * KD);
        };
        auto touch_x = [&](int hw, int n) {
            const unsigned tl = (rec[15] >> (5 * (lane >> 3 & 3))) & 31u;
            if (lane < n) touch(ldsw, tl * (16 * KD) + (lane >> 5) * 32768 + (lane & 7) * 128, ax + hw * 1024);
        };
        auto touch_w = [&](int k, int hw) {
            const unsigned j = lane >> 3, q = lane & 1;
            touch(ldsw, (((j >> 1) | (j & 1) << 2 | (q ^ k) << 3) * 64 + (lane >> 1 & 3)) * KD + q * 128,
                  aw + (size_t)((hw >> 4) * 1024 + (hw & 15) * 4) * KD);
        };
        if (rel == 0) {
            if (DLYP) wait_until(DLYP * 5);
            if (LH) help_x(2, hw);
            const unsigned q = (lane >> 3) & 3;
            #pragma unroll 1
            for (int j = 0; j < LH; ++j)
                glds1(ldsw + j * 1024, ((j >> 1) ^ (q << 2)) * (64 * KD) + (lane >> 5) * KD + (lane & 31) * 16,
                      aw + (size_t)((hw >> 4) * 1024 + (hw & 15) * 4 + (j & 1) * 2) * KD);
            VMW(0);
            STAMP(9);
        } else if (idx >= 24) {
            ESTAMP(35);
            if (PX) help_x(4, hw);
            ESTAMP(36);
            #pragma unroll
            for (int j = 0; j < PJ; ++j) help_w(j, hw);
            ESTAMP(37);
            const unsigned call = call_no();
            const u8* dst = dws + WS_XST + (size_t)(call & 1u) * WS_XST_PAR + (size_t)slot * (XM * KD);
            VMN((SPARSE ? PX + PJ / 8 : 4 * PX + PJ) + (STEP ? XPW - 1 : 0));
            STAMP(9);
            #pragma unroll
            for (int i = 0; i < XPW; ++i) {
                if (STEP && i) VMN(4 * PX + PJ + XPW - 1 - i);
                KEEP(xv[i]);
                store16_sys(xoff + i * 1024, xv[i], dst);
            }
            VMW(0);
            barrier();
            if (tid == 0) store4_sys(0, call, dws + WS_XFLAG + (slot * 8 + (idx & 7)) * 4);
            STAMP(1);
        } else {
            ESTAMP(35);
            help_x(4, hw);
            ESTAMP(36);
            if (!PX) help_x(4, hw + 32);
            #pragma unroll
            for (int j = 0; j < 16; ++j) {
                help_w(j, hw);
                if (j >= PJ) help_w(j, hw + 32);
            }
            ESTAMP(37);
            VMW(8);
            STAMP(9);
            #pragma unroll 1
            for (int j = 0; j < XPF * 8; ++j) {
                unsigned tl = ((rec[15] >> (5 * ((j >> 1) & 3))) & 31u) + 1 + (j >> 3);
                if (tl > 31) tl = 31;
                glds1(ldsw + (j & 31) * 1024, lane * 16, ax + (size_t)tl * (16 * KD) + (hw * 2 + (j & 1)) * 1024);
                asm volatile("s_waitcnt vmcnt(%0)" :: "n"(XPD) : "memory");
            }
            if (DLYW) wait_until(DLYW * 5);
            #pragma unroll 1
            for (int j = 0; j < 32; ++j) {
                glds1(ldsw + j * 1024, (lane >> 4) * KD + (lane & 15) * 16, aw + (size_t)(j * 128 + hw * 4) * KD + 256);
                asm volatile("s_waitcnt vmcnt(%0)" :: "n"(XPD) : "memory");
            }
            VMW(0);
            STAMP(1);
        }
    }

    int nt = job & 31u;
    if (nt < 5) nt = 5;
    int t0 = (job >> 5) & 31u;
    if (t0 + nt > (staged ? 7 * XT + 1 : 32)) { t0 = 0; nt = 5; }
    const int strip = (job >> 10) & 7;
    const int xg = (job >> 13) & 7;
    const unsigned delay = ((job >> 16) & 255u) * 5u;

    unsigned wo[4];
    {
        const int row = lane >> 2, pos = lane & 3;
        #pragma unroll
        for (int p = 0; p < 4; ++p)
            wo[p] = (16 * p + row) * KD + ((pos ^ (((row >> 2) & 1) | ((p & 1) << 1))) << 4) - p * 1024;
    }
    unsigned xo[4];
    #pragma unroll
    for (int p = 0; p < 4; ++p) {
        const int m = 8 * (p & 1) + (lane >> 3);
        const int cs = (lane & 7) ^ (m >> 1);
        xo[p] = m * KD + ((((p >> 1) ^ (xg & 1)) << 7) + (cs << 4)) - p * 1024 + 4096;
    }
    unsigned xq[4];
    #pragma unroll
    for (int q = 0; q < 4; ++q) xq[q] = (q ^ (xg >> 1)) << 8;
    const v4u wsrd = make_srd(aw + (size_t)(wave * 1024) * KD + strip * 64, 1024 * KD);
    constexpr unsigned XWIN = 17 * KD + 4096;
    const int tf = staged ? 30 + (job >> 25 & 1u) : t0;
    unsigned orow = (rank * MSH + tf * 16) * (NSH * 4);
    int ocnt = 32;
    float poison = 0.f;
    const u8* xin = ax + (wave * 1024 - 4096) + (size_t)tf * (16 * KD);
    v4u xsrd = make_srd(xin, XWIN);
    auto next_x = [&]() { xin += 16 * KD; xsrd = make_srd(xin, XWIN); };
    auto next_o = [&]() {
        orow += 16 * NSH * 4;
        if (--ocnt == 0) { orow = (orow - (MSH + XM) * NSH * 4) & (NR * MSH * NSH * 4 - 1); ocnt = XT; }
    };
    auto issue_w = [&](int h, unsigned unit) { dma4(unit, wo, wsrd, (unsigned)(h ^ (xg << 1)) * (64 * KD)); };

    if (delay) {
        wait_until(delay);
        STAMP(7); CSTAMP(10);
    }

    if (idx < 16) { ESTAMP(35); }
    #pragma unroll
    for (int q = 0; q < 4; ++q) dma4(ldsw + q * 4096, xo, xsrd, xq[q]);
    if (idx < 16) { ESTAMP(36); }
    next_x();
    issue_w(0, ldsw + 16384);
    issue_w(1, ldsw + 20480);
    if (idx < 16) { ESTAMP(37); }
    asm volatile("s_barrier" ::: "memory");
    if (idx < 16) { ESTAMP(39); }
    issue_w(2, ldsw + 24576);
    issue_w(3, ldsw + 28672);
    issue_w(4, ldsr);
    issue_w(5, ldsr + 16384);
    PIN();
    STAMP(14);
    const LDS_AS u8* wrw[4];
    const LDS_AS u8* wrr[4];
    {
        const int q = i16 >> 1, p = i16 & 1;
        const int f = ((q >> 2) & 1) | ((g & 1) << 1);
        #pragma unroll
        for (int j = 0; j < 4; ++j) {
            const unsigned o = g * 1024 + q * 64 + ((j ^ f) << 4) + 8 * p;
            wrw[j] = (const LDS_AS u8*)(size_t)(ldsw + o);
            wrr[j] = (const LDS_AS u8*)(size_t)(ldsr + o);
        }
    }
    const unsigned xr0 = ldsw + i16 * 128 + ((g ^ (i16 >> 1)) << 4);
    const int om = 4 * wave + g;
    const unsigned ovoff = om * (NSH * 4) + i16 * 16;
    const unsigned prd0 = lds0 + om * 256 + ((i16 ^ om) << 4);
    unsigned pwr0[4];
    #pragma unroll
    for (int j = 0; j < 4; ++j) pwr0[j] = ldsr + i16 * 256 + (((4 * j + g) ^ i16) << 4);
    v8i wf[8][4], xf[4];
    v4f acc[4];
    float scale;
    v4f oh;
    unsigned ovh;
    auto read_xs = [&](unsigned ad, int s) {
        const v4i lo = *(const LDS_AS v4i*)((const LDS_AS u8*)(size_t)ad + s * 2048);
        const v4i hi = *(const LDS_AS v4i*)((const LDS_AS u8*)(size_t)(ad ^ 64u) + s * 2048);
        return (v8i){lo[0], lo[1], lo[2], lo[3], hi[0], hi[1], hi[2], hi[3]};
    };
    auto read_w = [&](int h, const LDS_AS u8* const* base, int off) {
        #pragma unroll
        for (int j = 0; j < 4; ++j) {
            #pragma unroll
            for (int r = 0; r < 2; ++r) {
                const v2i t = __builtin_amdgcn_ds_read_tr8_b64_v2i32((LDS_AS v2i*)(base[j] + off + r * 512));
                wf[h >> 1][j][(h & 1) * 4 + 2 * r] = t[0];
                wf[h >> 1][j][(h & 1) * 4 + 2 * r + 1] = t[1];
            }
        }
        LGKM0();
        STAMP(42 + h);
        if (h & 1) {
            #pragma unroll
            for (int j = 0; j < 4; ++j) {
                if (h < 12) asm volatile("" : "+a"(wf[h >> 1][j]));
                else asm volatile("" : "+v"(wf[h >> 1][j]));
            }
        }
    };
    const v4f zero = {0.f, 0.f, 0.f, 0.f};
    const u8* const obase = aout + strip * 256;
    auto pw = [&](int j, unsigned bo) { *(LDS_AS v4f*)(size_t)(pwr0[j] + bo) = acc[j]; };
    auto pr = [&](int k, unsigned bo) { return *(const LDS_AS v4f*)((const LDS_AS u8*)(size_t)(prd0 + bo) + k * 4096); };
    auto sum4 = [&](const v4f (&p)[4], int e) {
        float a = p[0][e] + p[1][e], c = p[2][e] + p[3][e];
        KEEP(a); KEEP(c);
        float s = a + c;
        KEEP(s);
        return s;
    };
    auto finish = [&](const v4f (&p)[4]) {
        float y[4];
        #pragma unroll
        for (int e = 0; e < 4; ++e) {
            const float t = act1(y[e], sum4(p, e), scale);
            y[e] = act3(y[e], act2(t));
        }
        return (v4f){y[0], y[1], y[2], y[3]};
    };
    {
        v8i x0[8];
        auto mma = [&](int s) {
            PIN();
            #pragma unroll
            for (int j = 0; j < 4; ++j)
                acc[j] = __builtin_amdgcn_mfma_scale_f32_16x16x128_f8f6f4(wf[s][j], x0[s], s ? acc[j] : zero, 0, 0, 0, 0, 0, 0);
            PIN();
        };
        STAMP(40);
        VMW(24);
        STAMP(41);
        #pragma unroll
        for (int s = 0; s < 8; ++s) x0[s] = read_xs(xr0, s);
        LGKM0();
        issue_w(6, ldsw);            VMW(24); read_w(0, wrw, 16384);
        issue_w(7, ldsw + 4096);     VMW(24); read_w(1, wrw, 20480);                          mma(0);
        issue_w(8, ldsw + 8192);     VMW(24); read_w(2, wrw, 24576);
        issue_w(9, ldsw + 12288);    VMW(24); read_w(3, wrw, 28672);                          mma(1);
        issue_w(10, ldsw + 16384);   VMW(24); read_w(4, wrr, 0);
        issue_w(11, ldsw + 20480);   VMW(20); read_w(5, wrr, 16384);  read_w(6, wrw, 0);      mma(2);
        issue_w(12, ldsw + 24576);   VMW(16); read_w(7, wrw, 4096);   read_w(8, wrw, 8192);   mma(3);
        issue_w(13, ldsw + 28672);   VMW(16); read_w(9, wrw, 12288);                          mma(4);
        issue_w(14, ldsr);           VMW(16); read_w(10, wrw, 16384);
        issue_w(15, ldsr + 16384);   VMW(16); read_w(11, wrw, 20480);                         mma(5);
        asm volatile("s_barrier" ::: "memory");
        ovh = ovoff + orow;
        unsigned orow1 = orow + 16 * NSH * 4;
        if (staged) {
            const unsigned call = call_no();
            xin = myws + WS_XST + (size_t)(call & 1u) * WS_XST_PAR + (wave * 1024 - 4096) + (size_t)t0 * (16 * KD);
            xsrd = make_srd(xin, XWIN);
            const int sl = t0 / XT;
            ocnt = XT - (t0 - sl * XT);
            orow1 = (((rank - 1 - sl) & (NR - 1)) * MSH + MSH - ocnt * 16) * (NSH * 4);
            const unsigned long long tw = __builtin_amdgcn_s_memrealtime();
            while (true) {
                unsigned f, tm;
                asm volatile("global_load_dword %0, %2, %3 sc0 sc1\n\t"
                             "global_load_dword %1, %4, %5 sc0 sc1\n\t"
                             "s_waitcnt vmcnt(0)" : "=&v"(f), "=&v"(tm)
                             : "v"((lane < 56 ? lane : 0) * 4), "s"(myws + WS_XFLAG), "v"(0u), "s"(myws + WS_TMO) : "memory");
                if (__builtin_amdgcn_ballot_w64(f < call) == 0ull) break;
                if (first(tm) == call || __builtin_amdgcn_s_memrealtime() - tw > DEADLINE_TICKS) {
                    if (lane == 0) store4_sys(0, call, myws + WS_TMO);
                    poison = __builtin_nanf("");
                    break;
                }
                __builtin_amdgcn_s_sleep(8);
            }
            STAMP(8);
        }
        orow = orow1;
        dma4(ldsw, xo, xsrd, xq[0]);          VMW(12); read_w(12, wrw, 24576); read_w(13, wrw, 28672); mma(6);
        dma4(ldsw + 4096, xo, xsrd, xq[1]);   VMW(8);  read_w(14, wrr, 0);     read_w(15, wrr, 16384); mma(7);
        STAMP(2); CSTAMP(11);
        asm volatile("s_waitcnt lgkmcnt(0)\n\tv_mov_b32 %0, %1\n\tv_mul_f32 %0, %2, %0" : "=v"(scale) : "s"(sclx), "s"(sclw));
        scale += poison;
        dma4(ldsw + 8192, xo, xsrd, xq[2]);
        dma4(ldsw + 12288, xo, xsrd, xq[3]);
        next_x();
        #pragma unroll
        for (int q = 0; q < (8 - SH) / 2; ++q) dma4(ldsw + 16384 + q * 4096, xo, xsrd, xq[q]);
        set_m0(ldsw + 16384 + (8 - SH) / 2 * 4096);
        dma<0>(xo[0], xsrd, xq[(8 - SH) / 2]);
        dma<1024>(xo[1], xsrd, xq[(8 - SH) / 2]);
        VMN(30 - 2 * SH);
        xf[0] = read_xs(xr0, 0);
        xf[1] = read_xs(xr0, 1);
        PIN();
        oh = zero;
        STAMP(13);
    }

    unsigned boff = 0;
    #pragma unroll 1
    for (int i = 1; i < nt; ++i) {
        const unsigned lb = ldsw + boff, lo = ldsw + (boff ^ 16384u), xr = xr0 + boff, xn = xr0 + (boff ^ 16384u);
        v4f n[4], p[4];
        float y[4], t[4];
        #define MM(D, S, J, C) do { PIN(); KEEP(xf[(S) & 3]); \
            D = __builtin_amdgcn_mfma_scale_f32_16x16x128_f8f6f4(wf[S][J], xf[(S) & 3], C, 0, 0, 0, 0, 0, 0); \
            asm volatile("" : "+a"(D)); PIN(); } while (0)
        #define SQ(S) (((S) + 9 - SH) & 7)
        #define RD(S) do { VMN(28 - 2 * SH); xf[((S) + 2) & 3] = read_xs((S) < 6 ? xr : xn, ((S) + 2) & 7); \
            if (SQ(S) == 0) { next_x(); if (i + 2 >= nt) xsrd[2] = 0; } \
            if ((SQ(S) & 1) == 0) set_m0(((S) + 1 >= SH ? lb : lo) + SQ(S) * 2048); } while (0)
        #define RQA(S) dma<(SQ(S) & 1) * 2048>(xo[(SQ(S) & 1) * 2], xsrd, xq[SQ(S) >> 1])
        #define RQB(S) dma<(SQ(S) & 1) * 2048 + 1024>(xo[(SQ(S) & 1) * 2 + 1], xsrd, xq[SQ(S) >> 1])
        #define OUT() do { oh = (v4f){y[0], y[1], y[2], y[3]}; store16_sys(ovh, oh, obase); } while (0)
        MM(n[0], 0, 0, zero);  RD(0);  pw(0, boff);
        MM(n[1], 0, 1, zero);  RQA(0); pw(1, boff);
        MM(n[2], 0, 2, zero);          pw(2, boff);
        MM(n[3], 0, 3, zero);  RQB(0); pw(3, boff);
        MM(n[0], 1, 0, n[0]);  RD(1);
        MM(n[1], 1, 1, n[1]);  RQA(1);
        MM(n[2], 1, 2, n[2]);
        MM(n[3], 1, 3, n[3]);  RQB(1);
        MM(n[0], 2, 0, n[0]);  RD(2);
        MM(n[1], 2, 1, n[1]);  RQA(2);
        asm volatile("s_waitcnt lgkmcnt(2)\n\ts_barrier" ::: "memory");
        p[0] = pr(0, boff); p[1] = pr(1, boff);
        MM(n[2], 2, 2, n[2]);
        p[2] = pr(2, boff); p[3] = pr(3, boff);
        MM(n[3], 2, 3, n[3]);  RQB(2);
        MM(n[0], 3, 0, n[0]);  RD(3);
        MM(n[1], 3, 1, n[1]);  RQA(3);
        MM(n[2], 3, 2, n[2]);          y[0] = sum4(p, 0);
        MM(n[3], 3, 3, n[3]);  RQB(3); y[1] = sum4(p, 1);
        MM(n[0], 4, 0, n[0]);  RD(4);  t[0] = act1(y[0], y[0], scale);
        MM(n[1], 4, 1, n[1]);  RQA(4); y[2] = sum4(p, 2);                  t[0] = act2(t[0]);
        MM(n[2], 4, 2, n[2]);          t[1] = act1(y[1], y[1], scale);     HOLD(oh); y[0] = act3(y[0], t[0]);
        MM(n[3], 4, 3, n[3]);  RQB(4); y[3] = sum4(p, 3);                  t[1] = act2(t[1]);
        MM(n[0], 5, 0, n[0]);  RD(5);                                      y[1] = act3(y[1], t[1]);
        MM(n[1], 5, 1, n[1]);  RQA(5); t[2] = act1(y[2], y[2], scale);
        MM(n[2], 5, 2, n[2]);          t[3] = act1(y[3], y[3], scale);     t[2] = act2(t[2]);
        MM(n[3], 5, 3, n[3]);  RQB(5); t[3] = act2(t[3]);                  y[2] = act3(y[2], t[2]);
        MM(n[0], 6, 0, n[0]);  RD(6);  y[3] = act3(y[3], t[3]);
        MM(n[1], 6, 1, n[1]);  RQA(6);
        MM(n[2], 6, 2, n[2]);  OUT();
        MM(n[3], 6, 3, n[3]);  RQB(6);
        MM(acc[0], 7, 0, n[0]);  RD(7);
        MM(acc[1], 7, 1, n[1]);  RQA(7);
        MM(acc[2], 7, 2, n[2]);
        boff ^= 16384u;
        MM(acc[3], 7, 3, n[3]);  RQB(7);
        ovh = ovoff + orow;
        next_o();
    }
    {
        #pragma unroll
        for (int j = 0; j < 4; ++j) pw(j, boff);
        barrier();
        v4f p[4];
        #pragma unroll
        for (int k = 0; k < 4; ++k) p[k] = pr(k, boff);
        store16_sys(ovh, finish(p), obase);
        HOLD(oh);
        STAMP(15 + nt);
    }
    STAMP(3); CSTAMP(12);

    VMW(0);
    STAMP(6);
    barrier();
    asm volatile(".Lpc_end:");
    if (wave == 2) {
        const unsigned call = call_no();
        if (lane == 0) {
            if (rel != 0) store4_sys(0, call, dws + WS_OFLAG + (slot * 32 + idx) * 4);
            store4_sys(0, call, cntp);
        }
        STAMP(4);
        if (b == 0) {
            const unsigned fo = (lane < 56 ? lane : 0) * 16;
            const unsigned long long tw = __builtin_amdgcn_s_memrealtime();
            while (true) {
                v4u f;
                unsigned tm;
                asm volatile("global_load_dwordx4 %0, %2, %3 sc0 sc1\n\t"
                             "global_load_dword %1, %4, %3 offset:-2048 sc0 sc1\n\t"
                             "s_waitcnt vmcnt(0)" : "=&v"(f), "=&v"(tm) : "v"(fo), "s"(myws + WS_OFLAG), "v"(0u) : "memory");
                const bool miss = f[0] < call || f[1] < call || f[2] < call || f[3] < call;
                if (__builtin_amdgcn_ballot_w64(miss) == 0ull) break;
                if (first(tm) == call || __builtin_amdgcn_s_memrealtime() - tw > DEADLINE_TICKS) {
                    if (lane == 0) {
                        store4_sys(0, call, myws + WS_TMO);
                        store4_sys(0, 0x7fc00000u, aout);
                    }
                    break;
                }
                __builtin_amdgcn_s_sleep(1);
            }
            STAMP(5);
        }
    }
}
static_assert(WS_OFLAG - WS_TMO == 2048, "layout");

inline unsigned mkjob(int nt, int t0, int strip, int xg, int delay50, int staged) {
    return (unsigned)nt | (unsigned)t0 << 5 | (unsigned)strip << 10 | (unsigned)xg << 13 | (unsigned)delay50 << 16 |
           (unsigned)staged << 24;
}

#define GROUP 0xb2
#define DLY1 0
#define DLYE 120
#define DLY2 140
#define DLY3 140
#define DLYL 20
#define DLYM 20

unsigned build_jobs(Args& a) {
    unsigned first4 = 0;
    const int n = 32 - XT;
    for (int idx = 0; idx < 32; ++idx) {
        {
            const int strip = (idx >> 4) * 4 + (idx & 3), sub = 3 - ((idx >> 2) & 3);
            const int lo = sub * n / 4, hi = (sub + 1) * n / 4;
            const int dly = idx < 8 ? 0 : idx < 16 ? DLY1 : idx < 20 ? DLYE : idx < 24 ? DLY2 : DLY3;
            a.jobs[0][idx] = mkjob(hi - lo, lo, strip, (idx & 3) * XGA, dly, 0);
            if (idx < 16) first4 |= (unsigned)lo << (5 * sub);
        }
        {
            const int strip = idx & 7, e = (idx >> 3) & 1, ns = 7 * XT;
            const int xg = LXG ? (idx >> 3) * 2 + (idx & 1) : idx & 7;
            if (idx < 16) a.jobs[1][idx] = mkjob(15, e * 15, strip, xg, DLYL, 0);
            else a.jobs[1][idx] = mkjob(1 + (e ? ns - ns / 2 : ns / 2), e ? ns / 2 : 0, strip, xg, DLYM, 1 + 2 * e);
        }
    }
    return first4;
}

}

extern "C" void kernel_launch(int rank, int nranks, ncclComm_t comm,
                              hipStream_t stream,
                              void* const* d_in, const int* in_sizes, int n_in,
                              void* d_out, int out_size,
                              const int* in_dtypes, int out_dtype,
                              void* const* d_out_all,
                              void* const* d_ws_all, size_t ws_bytes) {
    (void)comm; (void)in_dtypes; (void)out_dtype;
    if (nranks != NR || n_in != 4 || (unsigned)rank >= (unsigned)NR || ws_bytes < WS_NEED ||
        in_sizes[0] != MSH * KD || in_sizes[1] != KD * NR * NSH || out_size != NR * MSH * NSH) {
        fprintf(stderr, "kernel_launch: unexpected problem shape\n");
        return;
    }
    Args a;
    memset(&a, 0, sizeof(a));
    const unsigned first4 = build_jobs(a);
    int order[NR], n = 0;
    for (int pass = 0; pass < 2; ++pass)
        for (int r = 0; r < NR; ++r) {
            const int d = (rank + r) % NR;
            const bool same = ((GROUP >> d) & 1) == ((GROUP >> rank) & 1);
            if (same == (pass == 0)) order[n++] = d;
        }
    for (int r = 0; r < NR; ++r) {
        const int dest = order[r];
        Rec& c = a.rec[r];
        c.x = (const u8*)d_in[0];
        c.w = (const u8*)d_in[1] + dest * NSH;
        c.out = (u8*)(r ? d_out_all[dest] : d_out);
        c.dws = (u8*)d_ws_all[dest];
        c.myws = (u8*)d_ws_all[rank];
        c.sx = (const float*)d_in[2];
        c.sw = (const float*)d_in[3];
        c.rank = (unsigned)rank | (unsigned)((dest - rank - 1) & (NR - 1)) << 8;
        c.first4 = r ? first4 : 15u << 5;
    }
    hipLaunchKernelGGL(fused_kernel, dim3(NWG), dim3(NT), 0, stream, a);
    HIP_CHECK(hipGetLastError());
}
```

```cpp
#include <hip/hip_runtime.h>
#include <rccl/rccl.h>
#include <cstdio>
#include <cstring>

#define HIP_CHECK(expr) do { hipError_t _e = (expr); if (_e != hipSuccess) { \
    fprintf(stderr, "HIP error %s at %s:%d\n", hipGetErrorString(_e), __FILE__, __LINE__); \
    return; } } while (0)

namespace {

typedef int v2i __attribute__((ext_vector_type(2)));
typedef int v4i __attribute__((ext_vector_type(4)));
typedef int v8i __attribute__((ext_vector_type(8)));
typedef float v4f __attribute__((ext_vector_type(4)));
typedef unsigned v4u __attribute__((ext_vector_type(4)));
typedef unsigned v16u __attribute__((ext_vector_type(16)));
typedef unsigned char u8;
#define LDS_AS __attribute__((address_space(3)))

constexpr int NR = 8;
constexpr int NWG = 256;
constexpr int NT = 256;
constexpr int KD = 4096;
constexpr int MSH = 512;
constexpr int NSH = 512;
#define XPF 2
#define XPD 23
#define DLYW 100
#define PEEL 1
#define PJ 4
#define DLYE 120
#define DLY2 140
#define DLYL 20
#define HOIST 1
constexpr int XM = 64;
constexpr int XT = XM / 16;
constexpr unsigned long long DEADLINE_TICKS = 100000000ull;

constexpr int LDS_BUF = 32768;
constexpr int LDS_BYTES = LDS_BUF + 4 * 32768;

constexpr size_t WS_CNT = 0;
constexpr size_t WS_TMO = 2048;
constexpr size_t WS_OFLAG = 4096;
constexpr size_t WS_XFLAG = 8192;
constexpr size_t WS_XST = 65536;
constexpr size_t WS_XST_PAR = (size_t)7 * 64 * KD;
constexpr size_t WS_DBG = WS_XST + 2 * WS_XST_PAR;
constexpr size_t WS_NEED = WS_DBG + 131072;

struct Rec {
    const u8* x;
    const u8* w;
    u8* out;
    u8* dws;
    u8* myws;
    const float* sx;
    const float* sw;
    unsigned rank;
    unsigned slot;
};
struct alignas(64) Args {
    Rec rec[NR];
    u8* xdst[16];
    u8* xflag[8];
};
static_assert(sizeof(Rec) == 64 && offsetof(Args, xdst) == 512 && offsetof(Args, xflag) == 640, "layout");

template <int O>
__device__ __forceinline__ void dma(unsigned v, v4u srd, unsigned s) {
    asm volatile("buffer_load_dwordx4 %0, %1, %2 offen offset:%3 lds" :: "v"(v), "s"(srd), "s"(s), "n"(O) : "memory");
}
__device__ __forceinline__ void set_m0(unsigned lds) {
    asm volatile("s_mov_b32 m0, %0\n\ts_nop 0" :: "s"(lds) : "memory", "m0");
}
__device__ __forceinline__ void dma4(unsigned lds, const unsigned (&v)[4], v4u srd, unsigned s) {
    set_m0(lds);
    dma<0>(v[0], srd, s); dma<1024>(v[1], srd, s); dma<2048>(v[2], srd, s); dma<3072>(v[3], srd, s);
}
__device__ __forceinline__ void glds1(unsigned lds, unsigned v0, const u8* base) {
    asm volatile("s_mov_b32 m0, %0\n\ts_nop 0\n\tglobal_load_lds_dwordx4 %1, %2" :: "s"(lds), "v"(v0), "s"(base) : "memory", "m0");
}
__device__ __forceinline__ v4u make_srd(const u8* p, unsigned bytes) {
    const unsigned long long a = (unsigned long long)p;
    return (v4u){(unsigned)a, (unsigned)(a >> 32) & 0xffffu, bytes, 0x00020000u};
}

#define VMW(n) asm volatile("s_waitcnt vmcnt(" #n ")" ::: "memory")
#define LGKM0() asm volatile("s_waitcnt lgkmcnt(0)" ::: "memory")
#define PIN() __builtin_amdgcn_sched_barrier(0)
#define KEEP(x) asm volatile("" : "+v"(x))
#define HOLD(a) asm volatile("" :: "v"(a))
#define VMN(n) asm volatile("s_waitcnt vmcnt(%0)" :: "n"(n) : "memory")

__device__ __forceinline__ void store16_sys(unsigned voff, v4f v, const void* base) {
    asm volatile("global_store_dwordx4 %0, %1, %2 sc0 sc1\n\ts_nop 1" :: "v"(voff), "v"(v), "s"(base) : "memory");
}
__device__ __forceinline__ void store4_sys(unsigned voff, unsigned v, const void* base) {
    asm volatile("global_store_dword %0, %1, %2 sc0 sc1" :: "v"(voff), "v"(v), "s"(base) : "memory");
}
__device__ __forceinline__ void barrier() {
    asm volatile("s_waitcnt lgkmcnt(0)\n\ts_barrier" ::: "memory");
}
__device__ __forceinline__ unsigned first(unsigned v) { return (unsigned)__builtin_amdgcn_readfirstlane(v); }

__device__ __forceinline__ float act1(float& y, float acc, float s) {
    y = acc * s;
    float t = __builtin_amdgcn_fmed3f(y, -60.0f, 60.0f) * -1.44269504f;
    KEEP(y); KEEP(t);
    return t;
}
__device__ __forceinline__ float act2(float t) {
    float d = __builtin_amdgcn_exp2f(t) + 1.0f;
    KEEP(d);
    return d;
}
__device__ __forceinline__ float act3(float y, float d) {
    float o = y * __builtin_amdgcn_rcpf(d);
    KEEP(o);
    return o;
}

#define ESTAMP(i) do {} while (0)
#define STAMP(i) do {} while (0)
#define CSTAMP(i) do {} while (0)

__device__ __forceinline__ u8* ptr(unsigned lo, unsigned hi) {
    return (u8*)(((unsigned long long)hi << 32) | lo);
}

__global__ __launch_bounds__(NT) void fused_kernel(const Args args) {
    __shared__ __attribute__((aligned(1024))) u8 smem[LDS_BYTES];
    const int b = blockIdx.x;
    const int rel = b & 7, idx = (b >> 3) & 31;
    v16u rec;
    unsigned long long t_start;
    asm volatile("s_load_dwordx16 %0, %2, %3\n\t"
                 "s_memrealtime %1"
                 : "=&s"(rec), "=&s"(t_start)
                 : "s"(__builtin_amdgcn_kernarg_segment_ptr()), "s"(rel * 64) : "memory");
    const int tid = threadIdx.x;
    const int lane = tid & 63;
    const int wave = __builtin_amdgcn_readfirstlane(tid >> 6);
    const unsigned lds0 = (unsigned)(size_t)(LDS_AS u8*)smem;
    const unsigned ldsw = lds0 + LDS_BUF + wave * 32768;
    const unsigned ldsr = lds0 + wave * 4096;
    const int g = lane >> 4, i16 = lane & 15;
    if (idx == 16 && wave >= 2) {
        const u8* pc;
        unsigned len;
        asm volatile("s_getpc_b64 %0\n.Lpc_here:\n\ts_mov_b32 %1, .Lpc_end-.Lpc_here" : "=s"(pc), "=s"(len));
        unsigned o = ((wave - 2) * 64 + lane) * 128;
        if (o > len - 16) o = len - 16;
        glds1(ldsw + 31 * 1024, o & ~15u, (const u8*)((unsigned long long)pc & ~15ull));
    }

    const bool staged = rel == 0 && idx >= 16;
    const int nt = rel ? 7 : 15;
    const int sh = rel ? (idx << 1) & 24 : idx & 24;
    const int t0 = ((rel ? 0x00070e15u : 0x0e000f00u) >> sh) & 31u;
    const int tf = ((rel ? 0x00070e15u : 0x1f1e0f00u) >> sh) & 31u;
    const int strip = (idx & 3) | ((rel ? idx >> 2 : idx) & 4);
    const int xg = rel ? (idx & 3) * 2 : (idx >> 2 & 6) | (idx & 1);
    const unsigned delay = (rel == 0 ? DLYL : idx < 16 ? 0 : idx < 20 ? DLYE : DLY2) * 5u;
    static_assert(XT == 4, "job table");

    unsigned wo[4];
    {
        const int row = lane >> 2, pos = lane & 3;
        #pragma unroll
        for (int p = 0; p < 4; ++p)
            wo[p] = (16 * p + row) * KD + ((pos ^ (((row >> 2) & 1) | ((p & 1) << 1))) << 4) - p * 1024;
    }
    unsigned xo[4];
    #pragma unroll
    for (int p = 0; p < 4; ++p) {
        const int m = 8 * (p & 1) + (lane >> 3);
        const int cs = (lane & 7) ^ (m >> 1);
        xo[p] = m * KD + ((((p >> 1) ^ (xg & 1)) << 7) + (cs << 4)) - p * 1024 + 4096;
    }
    unsigned xq[4];
    #pragma unroll
    for (int q = 0; q < 4; ++q) xq[q] = (q ^ (xg >> 1)) << 8;
    int xrel = wave * 1024 - 4096 + tf * (16 * KD);
    unsigned wrel = wave * 1024 * KD + strip * 64;
    unsigned crel = WS_CNT + rel * 128 + idx * 4;
    const unsigned pn = (rel - 1) * 8 + (idx & 7);
    const unsigned pe = (rel - 1) * 5 + (idx & 7);
    const bool two = wave == 0 && (idx & 7) < 5 && pe < XM * 4 - 224;
    const unsigned xoff = (pn * 4 + wave) * 1024 + lane * 16, xoff2 = (224 + pe) * 1024 + lane * 16;
    unsigned xs1 = xoff + (MSH - XM) * KD, xs2 = (two ? xoff2 : xoff) + (MSH - XM) * KD;
    unsigned psel = rel ? idx : 0;
    #pragma unroll
    for (int p = 0; p < 4; ++p) { KEEP(wo[p]); KEEP(xo[p]); asm volatile("" : "+s"(xq[p])); }
    asm volatile("" : "+s"(xrel), "+s"(wrel), "+s"(crel), "+s"(psel));
    v4f xv[2];
    asm volatile("s_waitcnt lgkmcnt(0)" : "+s"(rec), "+s"(t_start), "+v"(xs1), "+v"(xs2) :: "memory");
    const u8* const ax = ptr(rec[0], rec[1]);
    const u8* const aw = ptr(rec[2], rec[3]);
    u8* const aout = ptr(rec[4], rec[5]);
    u8* const dws = ptr(rec[6], rec[7]);
    u8* const myws = ptr(rec[8], rec[9]);
    asm volatile("s_cmp_lt_u32 %4, 24\n\t"
                 "s_cbranch_scc1 .Lno_rows\n\t"
                 "global_load_dwordx4 %0, %2, %5\n\t"
                 "global_load_dwordx4 %1, %3, %5\n"
                 ".Lno_rows:"
                 : "=&v"(xv[0]), "=&v"(xv[1])
                 : "v"(xs1), "v"(xs2), "s"(psel), "s"(ax) : "memory", "scc");
    const int rank = rec[14] & (NR - 1);
    const unsigned slot = min(rec[15], 6u);
    unsigned cnt;
    float sclx, sclw;
    asm volatile("s_load_dword %0, %3, %6\n\t"
                 "s_load_dword %1, %4, 0x0\n\t"
                 "s_load_dword %2, %5, 0x0"
                 : "=&s"(cnt), "=&s"(sclx), "=&s"(sclw)
                 : "s"(myws), "s"(ptr(rec[10], rec[11])), "s"(ptr(rec[12], rec[13])), "s"(crel) : "memory");
    auto call_no = [&]() {
        unsigned c;
        asm volatile("s_waitcnt lgkmcnt(0)\n\ts_add_u32 %0, %1, 1" : "=s"(c) : "s"(cnt) : "memory", "scc");
        return c;
    };

    auto wait_until = [&](unsigned ticks) {
        while ((unsigned)__builtin_amdgcn_s_memrealtime() - (unsigned)t_start < ticks) __builtin_amdgcn_s_sleep(1);
    };
    if (__builtin_expect(idx >= 16, 0)) {
        const int hw = (idx - 16) * 4 + wave;
        auto help_x = [&]() {
            #pragma unroll
            for (int j = 0; j < 4; ++j) glds1(ldsw + j * 1024, lane * 16, ax + j * (7 * 16 * KD) + hw * 1024);
        };
        auto help_w = [&](int j, int hw) {
            const unsigned q = (lane >> 3) & 1;
            glds1(ldsw + 4096 + j * 1024, (((j >> 1 & 3) | (j & 1) << 2 | (j & 8)) ^ (q << 3)) * (64 * KD) + (lane >> 4) * KD + (lane & 15) * 16,
                  aw + (size_t)((hw >> 4) * 1024 + (hw & 15) * 4) * KD);
        };
        if (rel == 0) {
            if (wave == 0) __builtin_amdgcn_fence(__ATOMIC_ACQUIRE, "");
            VMW(0);
            STAMP(9);
        } else if (idx >= 24) {
            v16u xd;
            unsigned long long fa;
            asm volatile("s_load_dwordx16 %0, %2, %3\n\t"
                         "global_load_dwordx2 %1, %4, %2 offset:640"
                         : "=&s"(xd), "=&v"(fa)
                         : "s"(__builtin_amdgcn_kernarg_segment_ptr()), "s"(512 + (pn * 4 + wave) % 7 * 8), "v"((lane & 7) * 8) : "memory");
            help_x();
            #pragma unroll
            for (int j = 0; j < PJ; ++j) help_w(j, hw);
            ESTAMP(35);
            const unsigned call = call_no();
            ESTAMP(36);
            asm volatile("" : "+s"(xd));
            const unsigned par = (call & 1u) * (unsigned)WS_XST_PAR;
            VMN(4 + PJ + 2);
            ESTAMP(37);
            STAMP(9);
            KEEP(xv[0]);
            #pragma unroll
            for (int i = 0; i < 7; ++i) store16_sys(xoff + par, xv[0], ptr(xd[2 * i], xd[2 * i + 1]));
            if (two) {
                VMN(4 + PJ + 1 + 7);
                KEEP(xv[1]);
                #pragma unroll
                for (int i = 0; i < 7; ++i) store16_sys(xoff2 + par, xv[1], ptr(xd[2 * i], xd[2 * i + 1]));
            }
            ESTAMP(39);
            VMW(0);
            KEEP(fa);
            barrier();
            if (tid < 7) asm volatile("global_store_dword %0, %1, off sc0 sc1" :: "v"(fa + pn * 4), "v"(call) : "memory");
            STAMP(1);
        } else {
            ESTAMP(35);
            help_x();
            ESTAMP(36);
            #pragma unroll
            for (int j = 0; j < 16; ++j) {
                help_w(j, hw);
                if (j >= PJ) help_w(j, hw + 32);
            }
            ESTAMP(37);
            VMW(8);
            STAMP(9);
            #pragma unroll 1
            for (int j = 0; j < XPF * 8; ++j) {
                const unsigned tl = ((j >> 1) & 3) * 7 + 1 + (j >> 3);
                glds1(ldsw + (j & 31) * 1024, lane * 16, ax + (size_t)tl * (16 * KD) + (hw * 2 + (j & 1)) * 1024);
                VMN(XPD);
            }
            if (DLYW) wait_until(DLYW * 5);
            #pragma unroll 1
            for (int j = 0; j < 32; ++j) {
                glds1(ldsw + j * 1024, (lane >> 4) * KD + (lane & 15) * 16, aw + (size_t)(j * 128 + hw * 4) * KD + 256);
                VMN(XPD);
            }
            VMW(0);
            STAMP(1);
        }
    }

    if (__builtin_expect(delay != 0, 0)) {
        wait_until(delay);
        STAMP(7); CSTAMP(10);
    }

    constexpr unsigned XWIN = 17 * KD + 4096;
    const u8* xin = ax + xrel;
    v4u xsrd = make_srd(xin, XWIN);
    if (idx < 16) { ESTAMP(35); }
    #pragma unroll
    for (int q = 0; q < 4; ++q) { dma4(ldsw + q * 4096, xo, xsrd, xq[q]); PIN(); }
    if (idx < 16) { ESTAMP(36); }
    const v4u wsrd = make_srd(aw + wrel, 1024 * KD);
    unsigned orow = (rank * MSH + tf * 16) * (NSH * 4);
    int ocnt = 32;
    float poison = 0.f;
    auto next_x = [&]() { xin += 16 * KD; xsrd = make_srd(xin, XWIN); };
    auto next_o = [&]() {
        orow += 16 * NSH * 4;
        if (--ocnt == 0) { orow = (orow - (MSH + XM) * NSH * 4) & (NR * MSH * NSH * 4 - 1); ocnt = XT; }
    };
    auto issue_w = [&](int h, unsigned unit) { dma4(unit, wo, wsrd, (unsigned)(h ^ (xg << 1)) * (64 * KD)); };
    next_x();
    issue_w(0, ldsw + 16384);
    issue_w(1, ldsw + 20480);
    if (idx < 16) { ESTAMP(37); }
    asm volatile("s_barrier" ::: "memory");
    if (idx < 16) { ESTAMP(39); }
    issue_w(2, ldsw + 24576);
    issue_w(3, ldsw + 28672);
    issue_w(4, ldsr);
    issue_w(5, ldsr + 16384);
    PIN();
    STAMP(14);
    const LDS_AS u8* wrw[4];
    const LDS_AS u8* wrr[4];
    {
        const int q = i16 >> 1, p = i16 & 1;
        const int f = ((q >> 2) & 1) | ((g & 1) << 1);
        #pragma unroll
        for (int j = 0; j < 4; ++j) {
            const unsigned o = g * 1024 + q * 64 + ((j ^ f) << 4) + 8 * p;
            wrw[j] = (const LDS_AS u8*)(size_t)(ldsw + o);
            wrr[j] = (const LDS_AS u8*)(size_t)(ldsr + o);
        }
    }
    const unsigned xr0 = ldsw + i16 * 128 + ((g ^ (i16 >> 1)) << 4);
    const int om = 4 * wave + g;
    const unsigned ovoff = om * (NSH * 4) + i16 * 16;
    const unsigned prd0 = lds0 + om * 256 + ((i16 ^ om) << 4);
    unsigned pwr0[4];
    #pragma unroll
    for (int j = 0; j < 4; ++j) pwr0[j] = ldsr + i16 * 256 + (((4 * j + g) ^ i16) << 4);
    v8i wf[8][4], xf[4];
    v4f acc[4];
    float scale;
    v4f oh;
    unsigned ovh;
    auto read_xs = [&](unsigned ad, int s) {
        const v4i lo = *(const LDS_AS v4i*)((const LDS_AS u8*)(size_t)ad + s * 2048);
        const v4i hi = *(const LDS_AS v4i*)((const LDS_AS u8*)(size_t)(ad ^ 64u) + s * 2048);
        return (v8i){lo[0], lo[1], lo[2], lo[3], hi[0], hi[1], hi[2], hi[3]};
    };
    auto read_w = [&](int h, const LDS_AS u8* const* base, int off) {
        #pragma unroll
        for (int j = 0; j < 4; ++j) {
            #pragma unroll
            for (int r = 0; r < 2; ++r) {
                const v2i t = __builtin_amdgcn_ds_read_tr8_b64_v2i32((LDS_AS v2i*)(base[j] + off + r * 512));
                wf[h >> 1][j][(h & 1) * 4 + 2 * r] = t[0];
                wf[h >> 1][j][(h & 1) * 4 + 2 * r + 1] = t[1];
            }
        }
        LGKM0();
        STAMP(42 + h);
        if (h & 1) {
            #pragma unroll
            for (int j = 0; j < 4; ++j) {
                if (h < 12) asm volatile("" : "+a"(wf[h >> 1][j]));
                else asm volatile("" : "+v"(wf[h >> 1][j]));
            }
        }
    };
    const v4f zero = {0.f, 0.f, 0.f, 0.f};
    const u8* const obase = aout + strip * 256;
    auto pw = [&](int j, unsigned bo) { *(LDS_AS v4f*)(size_t)(pwr0[j] + bo) = acc[j]; };
    auto pr = [&](int k, unsigned bo) { return *(const LDS_AS v4f*)((const LDS_AS u8*)(size_t)(prd0 + bo) + k * 4096); };
    auto sum4 = [&](const v4f (&p)[4], int e) {
        float a = p[0][e] + p[1][e], c = p[2][e] + p[3][e];
        KEEP(a); KEEP(c);
        float s = a + c;
        KEEP(s);
        return s;
    };
    auto finish = [&](const v4f (&p)[4]) {
        float y[4];
        #pragma unroll
        for (int e = 0; e < 4; ++e) {
            const float t = act1(y[e], sum4(p, e), scale);
            y[e] = act3(y[e], act2(t));
        }
        return (v4f){y[0], y[1], y[2], y[3]};
    };
    {
        v8i x0[8];
        auto mma = [&](int s) {
            PIN();
            #pragma unroll
            for (int j = 0; j < 4; ++j)
                acc[j] = __builtin_amdgcn_mfma_scale_f32_16x16x128_f8f6f4(wf[s][j], x0[s], s ? acc[j] : zero, 0, 0, 0, 0, 0, 0);
            PIN();
        };
        STAMP(40);
        VMW(24);
        STAMP(41);
        #pragma unroll
        for (int s = 0; s < 8; ++s) x0[s] = read_xs(xr0, s);
        LGKM0();
        issue_w(6, ldsw);            VMW(24); read_w(0, wrw, 16384);
        issue_w(7, ldsw + 4096);     VMW(24); read_w(1, wrw, 20480);                          mma(0);
        issue_w(8, ldsw + 8192);     VMW(24); read_w(2, wrw, 24576);
        issue_w(9, ldsw + 12288);    VMW(24); read_w(3, wrw, 28672);                          mma(1);
        issue_w(10, ldsw + 16384);   VMW(24); read_w(4, wrr, 0);
        issue_w(11, ldsw + 20480);   VMW(20); read_w(5, wrr, 16384);  read_w(6, wrw, 0);      mma(2);
        issue_w(12, ldsw + 24576);   VMW(16); read_w(7, wrw, 4096);   read_w(8, wrw, 8192);   mma(3);
        issue_w(13, ldsw + 28672);   VMW(16); read_w(9, wrw, 12288);                          mma(4);
        issue_w(14, ldsr);           VMW(16); read_w(10, wrw, 16384);
        issue_w(15, ldsr + 16384);   VMW(16); read_w(11, wrw, 20480);                         mma(5);
        asm volatile("s_barrier" ::: "memory");
        ovh = ovoff + orow;
        unsigned orow1 = orow + 16 * NSH * 4;
        if (staged) {
            const unsigned call = call_no();
            xin = myws + WS_XST + (size_t)(call & 1u) * WS_XST_PAR + (wave * 1024 - 4096) + (size_t)t0 * (16 * KD);
            xsrd = make_srd(xin, XWIN);
            const int sl = t0 / XT;
            ocnt = XT - (t0 - sl * XT);
            orow1 = (((rank - 1 - sl) & (NR - 1)) * MSH + MSH - ocnt * 16) * (NSH * 4);
            const unsigned long long tw = __builtin_amdgcn_s_memrealtime();
            while (true) {
                v4u f, h;
                unsigned tm;
                asm volatile("global_load_dwordx4 %0, %3, %4 sc0 sc1\n\t"
                             "global_load_dwordx4 %1, %3, %4 offset:16 sc0 sc1\n\t"
                             "global_load_dword %2, %5, %6 sc0 sc1\n\t"
                             "s_waitcnt vmcnt(0)" : "=&v"(f), "=&v"(h), "=&v"(tm)
                             : "v"((lane < 49 ? lane : 0) * 32), "s"(myws + WS_XFLAG), "v"(0u), "s"(myws + WS_TMO) : "memory");
                const unsigned fm = min(min(min(f[0], f[1]), min(f[2], f[3])), min(min(h[0], h[1]), min(h[2], h[3])));
                if (__builtin_amdgcn_ballot_w64(fm < call) == 0ull) break;
                if (first(tm) == call || __builtin_amdgcn_s_memrealtime() - tw > DEADLINE_TICKS) {
                    if (lane == 0) store4_sys(0, call, myws + WS_TMO);
                    poison = __builtin_nanf("");
                    break;
                }
                __builtin_amdgcn_s_sleep(8);
            }
            STAMP(8);
        }
        orow = orow1;
        dma4(ldsw, xo, xsrd, xq[0]);          VMW(12); read_w(12, wrw, 24576); read_w(13, wrw, 28672); mma(6);
        dma4(ldsw + 4096, xo, xsrd, xq[1]);   VMW(8);  read_w(14, wrr, 0);     read_w(15, wrr, 16384); mma(7);
        STAMP(2); CSTAMP(11);
        asm volatile("s_waitcnt lgkmcnt(0)\n\tv_mov_b32 %0, %1\n\tv_mul_f32 %0, %2, %0" : "=v"(scale) : "s"(sclx), "s"(sclw));
        scale += poison;
        dma4(ldsw + 8192, xo, xsrd, xq[2]);
        PIN();
        #pragma unroll
        for (int j = 0; j < 4; ++j) pw(j, 0);
        PIN();
        dma4(ldsw + 12288, xo, xsrd, xq[3]);
        next_x();
        set_m0(ldsw + 16384);
        dma<0>(xo[0], xsrd, xq[0]);
        dma<1024>(xo[1], xsrd, xq[0]);
        VMW(14);
        xf[0] = read_xs(xr0, 0);
        xf[1] = read_xs(xr0, 1);
        PIN();
        oh = zero;
        STAMP(13);
    }
    #define MM(D, S, J, C) do { PIN(); KEEP(xf[(S) & 3]); \
        D = __builtin_amdgcn_mfma_scale_f32_16x16x128_f8f6f4(wf[S][J], xf[(S) & 3], C, 0, 0, 0, 0, 0, 0); \
        asm volatile("" : "+a"(D)); PIN(); } while (0)
    #define OUT() do { oh = (v4f){y[0], y[1], y[2], y[3]}; store16_sys(ovh, oh, obase); } while (0)
    {
        const unsigned lo = ldsw + 16384, xn = xr0 + 16384;
        v4f n[4], p[4];
        float y[4], t[4];
        #define RH(S, N) do { VMW(N); xf[((S) + 2) & 3] = read_xs((S) < 6 ? xr0 : xn, ((S) + 2) & 7); } while (0)
        #define QA(Q) do { if (((Q) & 1) == 0) set_m0(lo + (Q) * 2048); dma<((Q) & 1) * 2048>(xo[((Q) & 1) * 2], xsrd, xq[(Q) >> 1]); } while (0)
        #define QB(Q) dma<((Q) & 1) * 2048 + 1024>(xo[((Q) & 1) * 2 + 1], xsrd, xq[(Q) >> 1])
        asm volatile("s_waitcnt lgkmcnt(4)\n\ts_barrier" ::: "memory");
        #pragma unroll
        for (int k = 0; k < 4; ++k) p[k] = pr(k, 0);
        MM(n[0], 0, 0, zero);  RH(0, 12); QA(1);
        MM(n[1], 0, 1, zero);  QB(1);
        MM(n[2], 0, 2, zero);  QA(2);     y[0] = sum4(p, 0);
        MM(n[3], 0, 3, zero);  QB(2);     y[1] = sum4(p, 1);
        MM(n[0], 1, 0, n[0]);  RH(1, 14); QA(3); t[0] = act1(y[0], y[0], scale);
        MM(n[1], 1, 1, n[1]);  QB(3);     y[2] = sum4(p, 2);                  t[0] = act2(t[0]);
        MM(n[2], 1, 2, n[2]);  QA(4);     t[1] = act1(y[1], y[1], scale);     y[0] = act3(y[0], t[0]);
        MM(n[3], 1, 3, n[3]);  QB(4);     y[3] = sum4(p, 3);                  t[1] = act2(t[1]);
        MM(n[0], 2, 0, n[0]);  RH(2, 16); QA(5);                              y[1] = act3(y[1], t[1]);
        MM(n[1], 2, 1, n[1]);  QB(5);     t[2] = act1(y[2], y[2], scale);
        MM(n[2], 2, 2, n[2]);  QA(6);     t[3] = act1(y[3], y[3], scale);     t[2] = act2(t[2]);
        MM(n[3], 2, 3, n[3]);  QB(6);     t[3] = act2(t[3]);                  y[2] = act3(y[2], t[2]);
        MM(n[0], 3, 0, n[0]);  RH(3, 18); QA(7); y[3] = act3(y[3], t[3]);
        MM(n[1], 3, 1, n[1]);  QB(7);
        MM(n[2], 3, 2, n[2]);  OUT();
        STAMP(34);
        MM(n[3], 3, 3, n[3]);
        MM(n[0], 4, 0, n[0]);  RH(4, 19);
        MM(n[1], 4, 1, n[1]);
        MM(n[2], 4, 2, n[2]);
        MM(n[3], 4, 3, n[3]);
        MM(n[0], 5, 0, n[0]);  RH(5, 17);
        MM(n[1], 5, 1, n[1]);
        MM(n[2], 5, 2, n[2]);
        MM(n[3], 5, 3, n[3]);
        MM(n[0], 6, 0, n[0]);  RH(6, 15);
        MM(n[1], 6, 1, n[1]);
        MM(n[2], 6, 2, n[2]);
        MM(n[3], 6, 3, n[3]);
        MM(acc[0], 7, 0, n[0]);  RH(7, 13); next_x(); set_m0(ldsw);
        MM(acc[1], 7, 1, n[1]);  dma<0>(xo[0], xsrd, xq[0]);
        STAMP(16);
        MM(acc[2], 7, 2, n[2]);
        MM(acc[3], 7, 3, n[3]);  dma<1024>(xo[1], xsrd, xq[0]);
        ovh = ovoff + orow;
        next_o();
    }

    unsigned boff = PEEL ? 16384 : 0;
    #pragma unroll 1
    for (int i = PEEL ? 2 : 1; i < nt; ++i) {
        const unsigned lb = ldsw + boff, lo = ldsw + (boff ^ 16384u), xr = xr0 + boff, xn = xr0 + (boff ^ 16384u);
        v4f n[4], p[4];
        float y[4], t[4];
        #define SQ(S) (((S) + 1) & 7)
        #define RD(S) do { VMW(12); xf[((S) + 2) & 3] = read_xs((S) < 6 ? xr : xn, ((S) + 2) & 7); \
            if (SQ(S) == 0) { next_x(); if (i + 2 >= nt) xsrd[2] = 0; } \
            if ((SQ(S) & 1) == 0) set_m0(((S) == 7 ? lb : lo) + SQ(S) * 2048); } while (0)
        #define RQA(S) dma<(SQ(S) & 1) * 2048>(xo[(SQ(S) & 1) * 2], xsrd, xq[SQ(S) >> 1])
        #define RQB(S) dma<(SQ(S) & 1) * 2048 + 1024>(xo[(SQ(S) & 1) * 2 + 1], xsrd, xq[SQ(S) >> 1])
        MM(n[0], 0, 0, zero);  RD(0);  pw(0, boff);
        MM(n[1], 0, 1, zero);  RQA(0); pw(1, boff);
        MM(n[2], 0, 2, zero);          pw(2, boff);
        MM(n[3], 0, 3, zero);  RQB(0); pw(3, boff);
        MM(n[0], 1, 0, n[0]);  RD(1);
        MM(n[1], 1, 1, n[1]);  RQA(1);
        MM(n[2], 1, 2, n[2]);
        MM(n[3], 1, 3, n[3]);  RQB(1);
        MM(n[0], 2, 0, n[0]);  RD(2);
        MM(n[1], 2, 1, n[1]);  RQA(2);
        asm volatile("s_waitcnt lgkmcnt(2)\n\ts_barrier" ::: "memory");
        p[0] = pr(0, boff); p[1] = pr(1, boff);
        MM(n[2], 2, 2, n[2]);
        p[2] = pr(2, boff); p[3] = pr(3, boff);
        MM(n[3], 2, 3, n[3]);  RQB(2);
        MM(n[0], 3, 0, n[0]);  RD(3);
        MM(n[1], 3, 1, n[1]);  RQA(3);
        MM(n[2], 3, 2, n[2]);          y[0] = sum4(p, 0);
        MM(n[3], 3, 3, n[3]);  RQB(3); y[1] = sum4(p, 1);
        MM(n[0], 4, 0, n[0]);  RD(4);  t[0] = act1(y[0], y[0], scale);
        MM(n[1], 4, 1, n[1]);  RQA(4); y[2] = sum4(p, 2);                  t[0] = act2(t[0]);
        MM(n[2], 4, 2, n[2]);          t[1] = act1(y[1], y[1], scale);     HOLD(oh); y[0] = act3(y[0], t[0]);
        MM(n[3], 4, 3, n[3]);  RQB(4); y[3] = sum4(p, 3);                  t[1] = act2(t[1]);
        MM(n[0], 5, 0, n[0]);  RD(5);                                      y[1] = act3(y[1], t[1]);
        MM(n[1], 5, 1, n[1]);  RQA(5); t[2] = act1(y[2], y[2], scale);
        MM(n[2], 5, 2, n[2]);          t[3] = act1(y[3], y[3], scale);     t[2] = act2(t[2]);
        MM(n[3], 5, 3, n[3]);  RQB(5); t[3] = act2(t[3]);                  y[2] = act3(y[2], t[2]);
        MM(n[0], 6, 0, n[0]);  RD(6);  y[3] = act3(y[3], t[3]);
        MM(n[1], 6, 1, n[1]);  RQA(6);
        MM(n[2], 6, 2, n[2]);  OUT();
        MM(n[3], 6, 3, n[3]);  RQB(6);
        MM(acc[0], 7, 0, n[0]);  RD(7);
        MM(acc[1], 7, 1, n[1]);  RQA(7);
        MM(acc[2], 7, 2, n[2]);
        boff ^= 16384u;
        MM(acc[3], 7, 3, n[3]);  RQB(7);
        ovh = ovoff + orow;
        next_o();
    }
    {
        #pragma unroll
        for (int j = 0; j < 4; ++j) pw(j, boff);
        barrier();
        v4f p[4];
        #pragma unroll
        for (int k = 0; k < 4; ++k) p[k] = pr(k, boff);
        store16_sys(ovh, finish(p), obase);
        HOLD(oh);
        STAMP(15 + nt);
    }
    STAMP(3); CSTAMP(12);

    VMW(0);
    STAMP(6);
    barrier();
    asm volatile(".Lpc_end:");
    if (wave == 2) {
        const unsigned call = call_no();
        if (lane == 0) {
            if (rel != 0) store4_sys(0, call, dws + WS_OFLAG + (slot * 32 + idx) * 4);
            store4_sys(crel, call, myws);
        }
        STAMP(4);
        if (b == 0) {
            const unsigned fo = (lane < 56 ? lane : 0) * 16;
            const unsigned long long tw = __builtin_amdgcn_s_memrealtime();
            while (true) {
                v4u f;
                unsigned tm;
                asm volatile("global_load_dwordx4 %0, %2, %3 sc0 sc1\n\t"
                             "global_load_dword %1, %4, %3 offset:-2048 sc0 sc1\n\t"
                             "s_waitcnt vmcnt(0)" : "=&v"(f), "=&v"(tm) : "v"(fo), "s"(myws + WS_OFLAG), "v"(0u) : "memory");
                const bool miss = f[0] < call || f[1] < call || f[2] < call || f[3] < call;
                if (__builtin_amdgcn_ballot_w64(miss) == 0ull) break;
                if (first(tm) == call || __builtin_amdgcn_s_memrealtime() - tw > DEADLINE_TICKS) {
                    if (lane == 0) {
                        store4_sys(0, call, myws + WS_TMO);
                        store4_sys(0, 0x7fc00000u, aout);
                    }
                    break;
                }
                __builtin_amdgcn_s_sleep(1);
            }
            STAMP(5);
        }
    }
}
static_assert(WS_OFLAG - WS_TMO == 2048, "layout");

}

extern "C" void kernel_launch(int rank, int nranks, ncclComm_t comm,
                              hipStream_t stream,
                              void* const* d_in, const int* in_sizes, int n_in,
                              void* d_out, int out_size,
                              const int* in_dtypes, int out_dtype,
                              void* const* d_out_all,
                              void* const* d_ws_all, size_t ws_bytes) {
    (void)comm; (void)in_dtypes; (void)out_dtype;
    if (nranks != NR || n_in != 4 || (unsigned)rank >= (unsigned)NR || ws_bytes < WS_NEED ||
        in_sizes[0] != MSH * KD || in_sizes[1] != KD * NR * NSH || out_size != NR * MSH * NSH) {
        fprintf(stderr, "kernel_launch: unexpected problem shape\n");
        return;
    }
    Args a;
    memset(&a, 0, sizeof(a));
    constexpr int GROUP = 0xb2;
    int order[NR], n = 0;
    for (int pass = 0; pass < 2; ++pass)
        for (int r = 0; r < NR; ++r) {
            const int d = (rank + r) % NR;
            const bool same = ((GROUP >> d) & 1) == ((GROUP >> rank) & 1);
            if (same == (pass == 0)) order[n++] = d;
        }
    for (int r = 0; r < NR; ++r) {
        const int dest = order[r];
        Rec& c = a.rec[r];
        c.x = (const u8*)d_in[0];
        c.w = (const u8*)d_in[1] + dest * NSH;
        c.out = (u8*)(r ? d_out_all[dest] : d_out);
        c.dws = (u8*)d_ws_all[dest];
        c.myws = (u8*)d_ws_all[rank];
        c.sx = (const float*)d_in[2];
        c.sw = (const float*)d_in[3];
        c.rank = (unsigned)rank;
        c.slot = (unsigned)((dest - rank - 1) & (NR - 1));
        if (r) {
            a.xdst[r - 1] = a.xdst[r + 6] = c.dws + WS_XST + (size_t)c.slot * (XM * KD);
            a.xflag[r - 1] = c.dws + WS_XFLAG + c.slot * (56 * 4);
        }
    }
    a.xdst[14] = a.xdst[15] = a.xdst[0];
    a.xflag[7] = a.xflag[0];
    hipLaunchKernelGGL(fused_kernel, dim3(NWG), dim3(NT), 0, stream, a);
    HIP_CHECK(hipGetLastError());
}
```

```cpp
#include <hip/hip_runtime.h>
#include <rccl/rccl.h>
#include <cstdio>
#include <cstring>

#define HIP_CHECK(expr) do { hipError_t _e = (expr); if (_e != hipSuccess) { \
    fprintf(stderr, "HIP error %s at %s:%d\n", hipGetErrorString(_e), __FILE__, __LINE__); \
    return; } } while (0)

namespace {

typedef int v2i __attribute__((ext_vector_type(2)));
typedef int v4i __attribute__((ext_vector_type(4)));
typedef int v8i __attribute__((ext_vector_type(8)));
typedef float v4f __attribute__((ext_vector_type(4)));
typedef unsigned v4u __attribute__((ext_vector_type(4)));
typedef unsigned v16u __attribute__((ext_vector_type(16)));
typedef unsigned char u8;
#define LDS_AS __attribute__((address_space(3)))

constexpr int NR = 8;
constexpr int NWG = 256;
constexpr int NT = 256;
constexpr int KD = 4096;
constexpr int MSH = 512;
constexpr int NSH = 512;
#define XPF 2
#define XPD 23
#define DLYW 100
#define PEEL 1
#define PJ 4
#define DLYE 120
#define DLY2 140
#define DLYL 20
#define HOIST 1
constexpr int XM = 64;
constexpr int XT = XM / 16;
constexpr unsigned long long DEADLINE_TICKS = 100000000ull;

constexpr int LDS_BUF = 32768;
constexpr int LDS_BYTES = LDS_BUF + 4 * 32768;

constexpr size_t WS_CNT = 0;
constexpr size_t WS_TMO = 2048;
constexpr size_t WS_OFLAG = 4096;
constexpr size_t WS_XFLAG = 8192;
constexpr size_t WS_XST = 65536;
constexpr size_t WS_XST_PAR = (size_t)7 * 64 * KD;
constexpr size_t WS_DBG = WS_XST + 2 * WS_XST_PAR;
constexpr size_t WS_NEED = WS_DBG + 131072;

struct Rec {
    const u8* x;
    const u8* w;
    u8* out;
    u8* dws;
    u8* myws;
    const float* sx;
    const float* sw;
    unsigned rank;
    unsigned slot;
};
struct alignas(64) Args {
    Rec rec[NR];
    u8* xdst[16];
    u8* xflag[8];
};
static_assert(sizeof(Rec) == 64 && offsetof(Args, xdst) == 512 && offsetof(Args, xflag) == 640, "layout");

template <int O>
__device__ __forceinline__ void dma(unsigned v, v4u srd, unsigned s) {
    asm volatile("buffer_load_dwordx4 %0, %1, %2 offen offset:%3 lds" :: "v"(v), "s"(srd), "s"(s), "n"(O) : "memory");
}
__device__ __forceinline__ void set_m0(unsigned lds) {
    asm volatile("s_mov_b32 m0, %0\n\ts_nop 0" :: "s"(lds) : "memory", "m0");
}
__device__ __forceinline__ void dma4(unsigned lds, const unsigned (&v)[4], v4u srd, unsigned s) {
    set_m0(lds);
    dma<0>(v[0], srd, s); dma<1024>(v[1], srd, s); dma<2048>(v[2], srd, s); dma<3072>(v[3], srd, s);
}
__device__ __forceinline__ void glds1(unsigned lds, unsigned v0, const u8* base) {
    asm volatile("s_mov_b32 m0, %0\n\ts_nop 0\n\tglobal_load_lds_dwordx4 %1, %2" :: "s"(lds), "v"(v0), "s"(base) : "memory", "m0");
}
__device__ __forceinline__ v4u make_srd(const u8* p, unsigned bytes) {
    const unsigned long long a = (unsigned long long)p;
    return (v4u){(unsigned)a, (unsigned)(a >> 32) & 0xffffu, bytes, 0x00020000u};
}

#define VMW(n) asm volatile("s_waitcnt vmcnt(" #n ")" ::: "memory")
#define LGKM0() asm volatile("s_waitcnt lgkmcnt(0)" ::: "memory")
#define PIN() __builtin_amdgcn_sched_barrier(0)
#define KEEP(x) asm volatile("" : "+v"(x))
#define HOLD(a) asm volatile("" :: "v"(a))
#define VMN(n) asm volatile("s_waitcnt vmcnt(%0)" :: "n"(n) : "memory")

__device__ __forceinline__ void store16_sys(unsigned voff, v4f v, const void* base) {
    asm volatile("global_store_dwordx4 %0, %1, %2 sc0 sc1\n\ts_nop 1" :: "v"(voff), "v"(v), "s"(base) : "memory");
}
__device__ __forceinline__ void store4_sys(unsigned voff, unsigned v, const void* base) {
    asm volatile("global_store_dword %0, %1, %2 sc0 sc1" :: "v"(voff), "v"(v), "s"(base) : "memory");
}
__device__ __forceinline__ void barrier() {
    asm volatile("s_waitcnt lgkmcnt(0)\n\ts_barrier" ::: "memory");
}
__device__ __forceinline__ unsigned first(unsigned v) { return (unsigned)__builtin_amdgcn_readfirstlane(v); }

__device__ __forceinline__ float act1(float& y, float acc, float s) {
    y = acc * s;
    float t = __builtin_amdgcn_fmed3f(y, -60.0f, 60.0f) * -1.44269504f;
    KEEP(y); KEEP(t);
    return t;
}
__device__ __forceinline__ float act2(float t) {
    float d = __builtin_amdgcn_exp2f(t) + 1.0f;
    KEEP(d);
    return d;
}
__device__ __forceinline__ float act3(float y, float d) {
    float o = y * __builtin_amdgcn_rcpf(d);
    KEEP(o);
    return o;
}

#define ESTAMP(i) do {} while (0)
#define STAMP(i) do {} while (0)
#define CSTAMP(i) do {} while (0)

__device__ __forceinline__ u8* ptr(unsigned lo, unsigned hi) {
    return (u8*)(((unsigned long long)hi << 32) | lo);
}

__global__ __launch_bounds__(NT) void fused_kernel(const Args args) {
    __shared__ __attribute__((aligned(1024))) u8 smem[LDS_BYTES];
    const int b = blockIdx.x;
    const int rel = b & 7, idx = (b >> 3) & 31;
    v16u rec;
    unsigned long long t_start;
    asm volatile("s_load_dwordx16 %0, %2, %3\n\t"
                 "s_memrealtime %1"
                 : "=&s"(rec), "=&s"(t_start)
                 : "s"(__builtin_amdgcn_kernarg_segment_ptr()), "s"(rel * 64) : "memory");
    const int tid = threadIdx.x;
    const int lane = tid & 63;
    const int wave = __builtin_amdgcn_readfirstlane(tid >> 6);
    const unsigned lds0 = (unsigned)(size_t)(LDS_AS u8*)smem;
    const unsigned ldsw = lds0 + LDS_BUF + wave * 32768;
    const unsigned ldsr = lds0 + wave * 4096;
    const int g = lane >> 4, i16 = lane & 15;
    if (idx == 16 && wave >= 2) {
        const u8* pc;
        unsigned len;
        asm volatile("s_getpc_b64 %0\n.Lpc_here:\n\ts_mov_b32 %1, .Lpc_end-.Lpc_here" : "=s"(pc), "=s"(len));
        unsigned o = ((wave - 2) * 64 + lane) * 128;
        if (o > len - 16) o = len - 16;
        glds1(ldsw + 31 * 1024, o & ~15u, (const u8*)((unsigned long long)pc & ~15ull));
    }

    const bool staged = rel == 0 && idx >= 16;
    const int nt = rel ? 7 : 15;
    const int sh = rel ? (idx << 1) & 24 : idx & 24;
    const int t0 = ((rel ? 0x00070e15u : 0x0e000f00u) >> sh) & 31u;
    const int tf = ((rel ? 0x00070e15u : 0x1f1e0f00u) >> sh) & 31u;
    const int strip = (idx & 3) | ((rel ? idx >> 2 : idx) & 4);
    const int xg = rel ? (idx & 3) * 2 : (idx >> 2 & 6) | (idx & 1);
    const unsigned delay = (rel == 0 ? DLYL : idx < 16 ? 0 : idx < 20 ? DLYE : DLY2) * 5u;
    static_assert(XT == 4, "job table");

    unsigned wo[4];
    {
        const int row = lane >> 2, pos = lane & 3;
        #pragma unroll
        for (int p = 0; p < 4; ++p)
            wo[p] = (16 * p + row) * KD + ((pos ^ (((row >> 2) & 1) | ((p & 1) << 1))) << 4) - p * 1024;
    }
    unsigned xo[4];
    #pragma unroll
    for (int p = 0; p < 4; ++p) {
        const int m = 8 * (p & 1) + (lane >> 3);
        const int cs = (lane & 7) ^ (m >> 1);
        xo[p] = m * KD + ((((p >> 1) ^ (xg & 1)) << 7) + (cs << 4)) - p * 1024 + 4096;
    }
    unsigned xq[4];
    #pragma unroll
    for (int q = 0; q < 4; ++q) xq[q] = (q ^ (xg >> 1)) << 8;
    int xrel = wave * 1024 - 4096 + tf * (16 * KD);
    unsigned wrel = wave * 1024 * KD + strip * 64;
    unsigned crel = WS_CNT + rel * 128 + idx * 4;
    const unsigned pn = (rel - 1) * 8 + (idx & 7);
    const unsigned pe = (rel - 1) * 5 + (idx & 7);
    const bool two = wave == 0 && (idx & 7) < 5 && pe < XM * 4 - 224;
    const unsigned xoff = (pn * 4 + wave) * 1024 + lane * 16, xoff2 = (224 + pe) * 1024 + lane * 16;
    unsigned xs1 = xoff + (MSH - XM) * KD, xs2 = (two ? xoff2 : xoff) + (MSH - XM) * KD;
    unsigned psel = rel ? idx : 0;
    #pragma unroll
    for (int p = 0; p < 4; ++p) { KEEP(wo[p]); KEEP(xo[p]); asm volatile("" : "+s"(xq[p])); }
    asm volatile("" : "+s"(xrel), "+s"(wrel), "+s"(crel), "+s"(psel));
    v4f xv[2];
    asm volatile("s_waitcnt lgkmcnt(0)" : "+s"(rec), "+s"(t_start), "+v"(xs1), "+v"(xs2) :: "memory");
    const u8* const ax = ptr(rec[0], rec[1]);
    const u8* const aw = ptr(rec[2], rec[3]);
    u8* const aout = ptr(rec[4], rec[5]);
    u8* const dws = ptr(rec[6], rec[7]);
    u8* const myws = ptr(rec[8], rec[9]);
    asm volatile("s_cmp_lt_u32 %4, 24\n\t"
                 "s_cbranch_scc1 .Lno_rows\n\t"
                 "global_load_dwordx4 %0, %2, %5\n\t"
                 "global_load_dwordx4 %1, %3, %5\n"
                 ".Lno_rows:"
                 : "=&v"(xv[0]), "=&v"(xv[1])
                 : "v"(xs1), "v"(xs2), "s"(psel), "s"(ax) : "memory", "scc");
    const int rank = rec[14] & (NR - 1);
    const unsigned slot = min(rec[15], 6u);
    unsigned cnt;
    float sclx, sclw;
    asm volatile("s_load_dword %0, %3, %6\n\t"
                 "s_load_dword %1, %4, 0x0\n\t"
                 "s_load_dword %2, %5, 0x0"
                 : "=&s"(cnt), "=&s"(sclx), "=&s"(sclw)
                 : "s"(myws), "s"(ptr(rec[10], rec[11])), "s"(ptr(rec[12], rec[13])), "s"(crel) : "memory");
    auto call_no = [&]() {
        unsigned c;
        asm volatile("s_waitcnt lgkmcnt(0)\n\ts_add_u32 %0, %1, 1" : "=s"(c) : "s"(cnt) : "memory", "scc");
        return c;
    };

    auto wait_until = [&](unsigned ticks) {
        while ((unsigned)__builtin_amdgcn_s_memrealtime() - (unsigned)t_start < ticks) __builtin_amdgcn_s_sleep(1);
    };
    if (__builtin_expect(idx >= 16, 0)) {
        const int hw = (idx - 16) * 4 + wave;
        auto help_x = [&]() {
            #pragma unroll
            for (int j = 0; j < 4; ++j) glds1(ldsw + j * 1024, lane * 16, ax + j * (7 * 16 * KD) + hw * 1024);
        };
        auto help_w = [&](int j, int hw) {
            const unsigned q = (lane >> 3) & 1;
            glds1(ldsw + 4096 + j * 1024, (((j >> 1 & 3) | (j & 1) << 2 | (j & 8)) ^ (q << 3)) * (64 * KD) + (lane >> 4) * KD + (lane & 15) * 16,
                  aw + (size_t)((hw >> 4) * 1024 + (hw & 15) * 4) * KD);
        };
        if (rel == 0) {
            if (wave == 0) __builtin_amdgcn_fence(__ATOMIC_ACQUIRE, "");
            VMW(0);
            STAMP(9);
        } else if (idx >= 24) {
            v16u xd;
            unsigned long long fa;
            asm volatile("s_load_dwordx16 %0, %2, %3\n\t"
                         "global_load_dwordx2 %1, %4, %2 offset:640"
                         : "=&s"(xd), "=&v"(fa)
                         : "s"(__builtin_amdgcn_kernarg_segment_ptr()), "s"(512 + (pn * 4 + wave) % 7 * 8), "v"((lane & 7) * 8) : "memory");
            help_x();
            #pragma unroll
            for (int j = 0; j < PJ; ++j) help_w(j, hw);
            ESTAMP(35);
            const unsigned call = call_no();
            ESTAMP(36);
            asm volatile("" : "+s"(xd));
            const unsigned par = (call & 1u) * (unsigned)WS_XST_PAR;
            VMN(4 + PJ + 2);
            ESTAMP(37);
            STAMP(9);
            KEEP(xv[0]);
            #pragma unroll
            for (int i = 0; i < 7; ++i) store16_sys(xoff + par, xv[0], ptr(xd[2 * i], xd[2 * i + 1]));
            if (two) {
                VMN(4 + PJ + 1 + 7);
                KEEP(xv[1]);
                #pragma unroll
                for (int i = 0; i < 7; ++i) store16_sys(xoff2 + par, xv[1], ptr(xd[2 * i], xd[2 * i + 1]));
            }
            ESTAMP(39);
            VMW(0);
            KEEP(fa); KEEP(xv[0]); KEEP(xv[1]);
            barrier();
            if (tid < 7) asm volatile("global_store_dword %0, %1, off sc0 sc1" :: "v"(fa + pn * 4), "v"(call) : "memory");
            STAMP(1);
        } else {
            ESTAMP(35);
            help_x();
            ESTAMP(36);
            #pragma unroll
            for (int j = 0; j < 16; ++j) {
                help_w(j, hw);
                if (j >= PJ) help_w(j, hw + 32);
            }
            ESTAMP(37);
            VMW(8);
            STAMP(9);
            #pragma unroll 1
            for (int j = 0; j < XPF * 8; ++j) {
                const unsigned tl = ((j >> 1) & 3) * 7 + 1 + (j >> 3);
                glds1(ldsw + (j & 31) * 1024, lane * 16, ax + (size_t)tl * (16 * KD) + (hw * 2 + (j & 1)) * 1024);
                VMN(XPD);
            }
            if (DLYW) wait_until(DLYW * 5);
            #pragma unroll 1
            for (int j = 0; j < 32; ++j) {
                glds1(ldsw + j * 1024, (lane >> 4) * KD + (lane & 15) * 16, aw + (size_t)(j * 128 + hw * 4) * KD + 256);
                VMN(XPD);
            }
            VMW(0);
            STAMP(1);
        }
    }

    if (__builtin_expect(delay != 0, 0)) {
        wait_until(delay);
        STAMP(7); CSTAMP(10);
    }

    constexpr unsigned XWIN = 17 * KD + 4096;
    const u8* xin = ax + xrel;
    v4u xsrd = make_srd(xin, XWIN);
    if (idx < 16) { ESTAMP(35); }
    #pragma unroll
    for (int q = 0; q < 4; ++q) { dma4(ldsw + q * 4096, xo, xsrd, xq[q]); PIN(); }
    if (idx < 16) { ESTAMP(36); }
    const v4u wsrd = make_srd(aw + wrel, 1024 * KD);
    unsigned orow = (rank * MSH + tf * 16) * (NSH * 4);
    int ocnt = 32;
    float poison = 0.f;
    auto next_x = [&]() { xin += 16 * KD; xsrd = make_srd(xin, XWIN); };
    auto next_o = [&]() {
        orow += 16 * NSH * 4;
        if (--ocnt == 0) { orow = (orow - (MSH + XM) * NSH * 4) & (NR * MSH * NSH * 4 - 1); ocnt = XT; }
    };
    auto issue_w = [&](int h, unsigned unit) { dma4(unit, wo, wsrd, (unsigned)(h ^ (xg << 1)) * (64 * KD)); };
    next_x();
    issue_w(0, ldsw + 16384);
    issue_w(1, ldsw + 20480);
    if (idx < 16) { ESTAMP(37); }
    asm volatile("s_barrier" ::: "memory");
    if (idx < 16) { ESTAMP(39); }
    issue_w(2, ldsw + 24576);
    issue_w(3, ldsw + 28672);
    issue_w(4, ldsr);
    issue_w(5, ldsr + 16384);
    PIN();
    STAMP(14);
    const LDS_AS u8* wrw[4];
    const LDS_AS u8* wrr[4];
    {
        const int q = i16 >> 1, p = i16 & 1;
        const int f = ((q >> 2) & 1) | ((g & 1) << 1);
        #pragma unroll
        for (int j = 0; j < 4; ++j) {
            const unsigned o = g * 1024 + q * 64 + ((j ^ f) << 4) + 8 * p;
            wrw[j] = (const LDS_AS u8*)(size_t)(ldsw + o);
            wrr[j] = (const LDS_AS u8*)(size_t)(ldsr + o);
        }
    }
    const unsigned xr0 = ldsw + i16 * 128 + ((g ^ (i16 >> 1)) << 4);
    const int om = 4 * wave + g;
    const unsigned ovoff = om * (NSH * 4) + i16 * 16;
    const unsigned prd0 = lds0 + om * 256 + ((i16 ^ om) << 4);
    unsigned pwr0[4];
    #pragma unroll
    for (int j = 0; j < 4; ++j) pwr0[j] = ldsr + i16 * 256 + (((4 * j + g) ^ i16) << 4);
    v8i wf[8][4], xf[4];
    v4f acc[4];
    float scale;
    v4f oh;
    unsigned ovh;
    auto read_xs = [&](unsigned ad, int s) {
        const v4i lo = *(const LDS_AS v4i*)((const LDS_AS u8*)(size_t)ad + s * 2048);
        const v4i hi = *(const LDS_AS v4i*)((const LDS_AS u8*)(size_t)(ad ^ 64u) + s * 2048);
        return (v8i){lo[0], lo[1], lo[2], lo[3], hi[0], hi[1], hi[2], hi[3]};
    };
    auto read_w = [&](int h, const LDS_AS u8* const* base, int off) {
        #pragma unroll
        for (int j = 0; j < 4; ++j) {
            #pragma unroll
            for (int r = 0; r < 2; ++r) {
                const v2i t = __builtin_amdgcn_ds_read_tr8_b64_v2i32((LDS_AS v2i*)(base[j] + off + r * 512));
                wf[h >> 1][j][(h & 1) * 4 + 2 * r] = t[0];
                wf[h >> 1][j][(h & 1) * 4 + 2 * r + 1] = t[1];
            }
        }
        LGKM0();
        STAMP(42 + h);
        if (h & 1) {
            #pragma unroll
            for (int j = 0; j < 4; ++j) {
                if (h < 12) asm volatile("" : "+a"(wf[h >> 1][j]));
                else asm volatile("" : "+v"(wf[h >> 1][j]));
            }
        }
    };
    const v4f zero = {0.f, 0.f, 0.f, 0.f};
    const u8* const obase = aout + strip * 256;
    auto pw = [&](int j, unsigned bo) { *(LDS_AS v4f*)(size_t)(pwr0[j] + bo) = acc[j]; };
    auto pr = [&](int k, unsigned bo) { return *(const LDS_AS v4f*)((const LDS_AS u8*)(size_t)(prd0 + bo) + k * 4096); };
    auto sum4 = [&](const v4f (&p)[4], int e) {
        float a = p[0][e] + p[1][e], c = p[2][e] + p[3][e];
        KEEP(a); KEEP(c);
        float s = a + c;
        KEEP(s);
        return s;
    };
    auto finish = [&](const v4f (&p)[4]) {
        float y[4];
        #pragma unroll
        for (int e = 0; e < 4; ++e) {
            const float t = act1(y[e], sum4(p, e), scale);
            y[e] = act3(y[e], act2(t));
        }
        return (v4f){y[0], y[1], y[2], y[3]};
    };
    {
        v8i x0[8];
        auto mma = [&](int s) {
            PIN();
            #pragma unroll
            for (int j = 0; j < 4; ++j)
                acc[j] = __builtin_amdgcn_mfma_scale_f32_16x16x128_f8f6f4(wf[s][j], x0[s], s ? acc[j] : zero, 0, 0, 0, 0, 0, 0);
            PIN();
        };
        STAMP(40);
        VMW(24);
        STAMP(41);
        #pragma unroll
        for (int s = 0; s < 8; ++s) x0[s] = read_xs(xr0, s);
        LGKM0();
        issue_w(6, ldsw);            VMW(24); read_w(0, wrw, 16384);
        issue_w(7, ldsw + 4096);     VMW(24); read_w(1, wrw, 20480);                          mma(0);
        issue_w(8, ldsw + 8192);     VMW(24); read_w(2, wrw, 24576);
        issue_w(9, ldsw + 12288);    VMW(24); read_w(3, wrw, 28672);                          mma(1);
        issue_w(10, ldsw + 16384);   VMW(24); read_w(4, wrr, 0);
        issue_w(11, ldsw + 20480);   VMW(20); read_w(5, wrr, 16384);  read_w(6, wrw, 0);      mma(2);
        issue_w(12, ldsw + 24576);   VMW(16); read_w(7, wrw, 4096);   read_w(8, wrw, 8192);   mma(3);
        issue_w(13, ldsw + 28672);   VMW(16); read_w(9, wrw, 12288);                          mma(4);
        issue_w(14, ldsr);           VMW(16); read_w(10, wrw, 16384);
        issue_w(15, ldsr + 16384);   VMW(16); read_w(11, wrw, 20480);                         mma(5);
        asm volatile("s_barrier" ::: "memory");
        ovh = ovoff + orow;
        unsigned orow1 = orow + 16 * NSH * 4;
        if (staged) {
            const unsigned call = call_no();
            xin = myws + WS_XST + (size_t)(call & 1u) * WS_XST_PAR + (wave * 1024 - 4096) + (size_t)t0 * (16 * KD);
            xsrd = make_srd(xin, XWIN);
            const int sl = t0 / XT;
            ocnt = XT - (t0 - sl * XT);
            orow1 = (((rank - 1 - sl) & (NR - 1)) * MSH + MSH - ocnt * 16) * (NSH * 4);
            const unsigned long long tw = __builtin_amdgcn_s_memrealtime();
            while (true) {
                v4u f, h;
                unsigned tm;
                asm volatile("global_load_dwordx4 %0, %3, %4 sc0 sc1\n\t"
                             "global_load_dwordx4 %1, %3, %4 offset:16 sc0 sc1\n\t"
                             "global_load_dword %2, %5, %6 sc0 sc1\n\t"
                             "s_waitcnt vmcnt(0)" : "=&v"(f), "=&v"(h), "=&v"(tm)
                             : "v"((lane < 49 ? lane : 0) * 32), "s"(myws + WS_XFLAG), "v"(0u), "s"(myws + WS_TMO) : "memory");
                const unsigned fm = min(min(min(f[0], f[1]), min(f[2], f[3])), min(min(h[0], h[1]), min(h[2], h[3])));
                if (__builtin_amdgcn_ballot_w64(fm < call) == 0ull) break;
                if (first(tm) == call || __builtin_amdgcn_s_memrealtime() - tw > DEADLINE_TICKS) {
                    if (lane == 0) store4_sys(0, call, myws + WS_TMO);
                    poison = __builtin_nanf("");
                    break;
                }
                __builtin_amdgcn_s_sleep(8);
            }
            STAMP(8);
        }
        orow = orow1;
        dma4(ldsw, xo, xsrd, xq[0]);          VMW(12); read_w(12, wrw, 24576); read_w(13, wrw, 28672); mma(6);
        dma4(ldsw + 4096, xo, xsrd, xq[1]);   VMW(8);  read_w(14, wrr, 0);     read_w(15, wrr, 16384); mma(7);
        STAMP(2); CSTAMP(11);
        asm volatile("s_waitcnt lgkmcnt(0)\n\tv_mov_b32 %0, %1\n\tv_mul_f32 %0, %2, %0" : "=v"(scale) : "s"(sclx), "s"(sclw));
        scale += poison;
        dma4(ldsw + 8192, xo, xsrd, xq[2]);
        PIN();
        #pragma unroll
        for (int j = 0; j < 4; ++j) pw(j, 0);
        PIN();
        dma4(ldsw + 12288, xo, xsrd, xq[3]);
        next_x();
        set_m0(ldsw + 16384);
        dma<0>(xo[0], xsrd, xq[0]);
        dma<1024>(xo[1], xsrd, xq[0]);
        VMW(14);
        xf[0] = read_xs(xr0, 0);
        xf[1] = read_xs(xr0, 1);
        PIN();
        oh = zero;
        STAMP(13);
    }
    #define MM(D, S, J, C) do { PIN(); KEEP(xf[(S) & 3]); \
        D = __builtin_amdgcn_mfma_scale_f32_16x16x128_f8f6f4(wf[S][J], xf[(S) & 3], C, 0, 0, 0, 0, 0, 0); \
        asm volatile("" : "+a"(D)); PIN(); } while (0)
    #define OUT() do { oh = (v4f){y[0], y[1], y[2], y[3]}; store16_sys(ovh, oh, obase); } while (0)
    {
        const unsigned lo = ldsw + 16384, xn = xr0 + 16384;
        v4f n[4], p[4];
        float y[4], t[4];
        #define RH(S, N) do { VMW(N); xf[((S) + 2) & 3] = read_xs((S) < 6 ? xr0 : xn, ((S) + 2) & 7); } while (0)
        #define QA(Q) do { if (((Q) & 1) == 0) set_m0(lo + (Q) * 2048); dma<((Q) & 1) * 2048>(xo[((Q) & 1) * 2], xsrd, xq[(Q) >> 1]); } while (0)
        #define QB(Q) dma<((Q) & 1) * 2048 + 1024>(xo[((Q) & 1) * 2 + 1], xsrd, xq[(Q) >> 1])
        asm volatile("s_waitcnt lgkmcnt(4)\n\ts_barrier" ::: "memory");
        #pragma unroll
        for (int k = 0; k < 4; ++k) p[k] = pr(k, 0);
        MM(n[0], 0, 0, zero);  RH(0, 12); QA(1);
        MM(n[1], 0, 1, zero);  QB(1);
        MM(n[2], 0, 2, zero);  QA(2);     y[0] = sum4(p, 0);
        MM(n[3], 0, 3, zero);  QB(2);     y[1] = sum4(p, 1);
        MM(n[0], 1, 0, n[0]);  RH(1, 14); QA(3); t[0] = act1(y[0], y[0], scale);
        MM(n[1], 1, 1, n[1]);  QB(3);     y[2] = sum4(p, 2);                  t[0] = act2(t[0]);
        MM(n[2], 1, 2, n[2]);  QA(4);     t[1] = act1(y[1], y[1], scale);     y[0] = act3(y[0], t[0]);
        MM(n[3], 1, 3, n[3]);  QB(4);     y[3] = sum4(p, 3);                  t[1] = act2(t[1]);
        MM(n[0], 2, 0, n[0]);  RH(2, 16); QA(5);                              y[1] = act3(y[1], t[1]);
        MM(n[1], 2, 1, n[1]);  QB(5);     t[2] = act1(y[2], y[2], scale);
        MM(n[2], 2, 2, n[2]);  QA(6);     t[3] = act1(y[3], y[3], scale);     t[2] = act2(t[2]);
        MM(n[3], 2, 3, n[3]);  QB(6);     t[3] = act2(t[3]);                  y[2] = act3(y[2], t[2]);
        MM(n[0], 3, 0, n[0]);  RH(3, 18); QA(7); y[3] = act3(y[3], t[3]);
        MM(n[1], 3, 1, n[1]);  QB(7);
        MM(n[2], 3, 2, n[2]);  OUT();
        STAMP(34);
        MM(n[3], 3, 3, n[3]);
        MM(n[0], 4, 0, n[0]);  RH(4, 19);
        MM(n[1], 4, 1, n[1]);
        MM(n[2], 4, 2, n[2]);
        MM(n[3], 4, 3, n[3]);
        MM(n[0], 5, 0, n[0]);  RH(5, 17);
        MM(n[1], 5, 1, n[1]);
        MM(n[2], 5, 2, n[2]);
        MM(n[3], 5, 3, n[3]);
        MM(n[0], 6, 0, n[0]);  RH(6, 15);
        MM(n[1], 6, 1, n[1]);
        MM(n[2], 6, 2, n[2]);
        MM(n[3], 6, 3, n[3]);
        MM(acc[0], 7, 0, n[0]);  RH(7, 13); next_x(); set_m0(ldsw);
        MM(acc[1], 7, 1, n[1]);  dma<0>(xo[0], xsrd, xq[0]);
        STAMP(16);
        MM(acc[2], 7, 2, n[2]);
        MM(acc[3], 7, 3, n[3]);  dma<1024>(xo[1], xsrd, xq[0]);
        ovh = ovoff + orow;
        next_o();
    }

    unsigned boff = PEEL ? 16384 : 0;
    #pragma unroll 1
    for (int i = PEEL ? 2 : 1; i < nt; ++i) {
        const unsigned lb = ldsw + boff, lo = ldsw + (boff ^ 16384u), xr = xr0 + boff, xn = xr0 + (boff ^ 16384u);
        v4f n[4], p[4];
        float y[4], t[4];
        #define SQ(S) (((S) + 1) & 7)
        #define RD(S) do { VMW(12); xf[((S) + 2) & 3] = read_xs((S) < 6 ? xr : xn, ((S) + 2) & 7); \
            if (SQ(S) == 0) { next_x(); if (i + 2 >= nt) xsrd[2] = 0; } \
            if ((SQ(S) & 1) == 0) set_m0(((S) == 7 ? lb : lo) + SQ(S) * 2048); } while (0)
        #define RQA(S) dma<(SQ(S) & 1) * 2048>(xo[(SQ(S) & 1) * 2], xsrd, xq[SQ(S) >> 1])
        #define RQB(S) dma<(SQ(S) & 1) * 2048 + 1024>(xo[(SQ(S) & 1) * 2 + 1], xsrd, xq[SQ(S) >> 1])
        MM(n[0], 0, 0, zero);  RD(0);  pw(0, boff);
        MM(n[1], 0, 1, zero);  RQA(0); pw(1, boff);
        MM(n[2], 0, 2, zero);          pw(2, boff);
        MM(n[3], 0, 3, zero);  RQB(0); pw(3, boff);
        MM(n[0], 1, 0, n[0]);  RD(1);
        MM(n[1], 1, 1, n[1]);  RQA(1);
        MM(n[2], 1, 2, n[2]);
        MM(n[3], 1, 3, n[3]);  RQB(1);
        MM(n[0], 2, 0, n[0]);  RD(2);
        MM(n[1], 2, 1, n[1]);  RQA(2);
        asm volatile("s_waitcnt lgkmcnt(2)\n\ts_barrier" ::: "memory");
        p[0] = pr(0, boff); p[1] = pr(1, boff);
        MM(n[2], 2, 2, n[2]);
        p[2] = pr(2, boff); p[3] = pr(3, boff);
        MM(n[3], 2, 3, n[3]);  RQB(2);
        MM(n[0], 3, 0, n[0]);  RD(3);
        MM(n[1], 3, 1, n[1]);  RQA(3);
        MM(n[2], 3, 2, n[2]);          y[0] = sum4(p, 0);
        MM(n[3], 3, 3, n[3]);  RQB(3); y[1] = sum4(p, 1);
        MM(n[0], 4, 0, n[0]);  RD(4);  t[0] = act1(y[0], y[0], scale);
        MM(n[1], 4, 1, n[1]);  RQA(4); y[2] = sum4(p, 2);                  t[0] = act2(t[0]);
        MM(n[2], 4, 2, n[2]);          t[1] = act1(y[1], y[1], scale);     HOLD(oh); y[0] = act3(y[0], t[0]);
        MM(n[3], 4, 3, n[3]);  RQB(4); y[3] = sum4(p, 3);                  t[1] = act2(t[1]);
        MM(n[0], 5, 0, n[0]);  RD(5);                                      y[1] = act3(y[1], t[1]);
        MM(n[1], 5, 1, n[1]);  RQA(5); t[2] = act1(y[2], y[2], scale);
        MM(n[2], 5, 2, n[2]);          t[3] = act1(y[3], y[3], scale);     t[2] = act2(t[2]);
        MM(n[3], 5, 3, n[3]);  RQB(5); t[3] = act2(t[3]);                  y[2] = act3(y[2], t[2]);
        MM(n[0], 6, 0, n[0]);  RD(6);  y[3] = act3(y[3], t[3]);
        MM(n[1], 6, 1, n[1]);  RQA(6);
        MM(n[2], 6, 2, n[2]);  OUT();
        MM(n[3], 6, 3, n[3]);  RQB(6);
        MM(acc[0], 7, 0, n[0]);  RD(7);
        MM(acc[1], 7, 1, n[1]);  RQA(7);
        MM(acc[2], 7, 2, n[2]);
        boff ^= 16384u;
        MM(acc[3], 7, 3, n[3]);  RQB(7);
        ovh = ovoff + orow;
        next_o();
    }
    {
        #pragma unroll
        for (int j = 0; j < 4; ++j) pw(j, boff);
        barrier();
        v4f p[4];
        #pragma unroll
        for (int k = 0; k < 4; ++k) p[k] = pr(k, boff);
        store16_sys(ovh, finish(p), obase);
        HOLD(oh);
        STAMP(15 + nt);
    }
    STAMP(3); CSTAMP(12);

    const unsigned call = call_no();
    if (tid == 128) store4_sys(crel, call, myws);
    if (rel != 0) {
        asm volatile("s_cmp_eq_u32 %0, 2\n\t"
                     "s_cselect_b64 exec, 1, 0\n\t"
                     ".p2align 4\n\t"
                     "s_waitcnt vmcnt(0)\n\t"
                     "s_barrier\n\t"
                     "global_store_dword %1, %2, %3 sc0 sc1\n\t"
                     "s_mov_b64 exec, -1"
                     :: "s"(wave), "v"(0u), "v"(call), "s"(dws + WS_OFLAG + (slot * 32 + idx) * 4) : "memory", "scc");
        STAMP(4);
    } else {
        STAMP(4);
    }
    if (b == 0 && wave == 2) {
        const unsigned fo = (lane < 56 ? lane : 0) * 16;
        const unsigned tw = (unsigned)__builtin_amdgcn_s_memrealtime();
        while (true) {
            v4u f;
            unsigned tm;
            unsigned long long now;
            asm volatile("global_load_dwordx4 %0, %3, %4 sc0 sc1\n\t"
                         "global_load_dword %1, %5, %4 offset:-2048 sc0 sc1\n\t"
                         "s_memrealtime %2\n\t"
                         "s_waitcnt vmcnt(0) lgkmcnt(0)" : "=&v"(f), "=&v"(tm), "=&s"(now) : "v"(fo), "s"(myws + WS_OFLAG), "v"(0u) : "memory");
            const bool miss = f[0] < call || f[1] < call || f[2] < call || f[3] < call;
            if (__builtin_amdgcn_ballot_w64(miss) == 0ull) break;
            if (first(tm) == call || (unsigned)now - tw > (unsigned)DEADLINE_TICKS) {
                if (lane == 0) {
                    store4_sys(0, call, myws + WS_TMO);
                    store4_sys(0, 0x7fc00000u, aout);
                }
                break;
            }
        }
        STAMP(5);
    }
    asm volatile(".Lpc_end:");
}
static_assert(WS_OFLAG - WS_TMO == 2048, "layout");

}

extern "C" void kernel_launch(int rank, int nranks, ncclComm_t comm,
                              hipStream_t stream,
                              void* const* d_in, const int* in_sizes, int n_in,
                              void* d_out, int out_size,
                              const int* in_dtypes, int out_dtype,
                              void* const* d_out_all,
                              void* const* d_ws_all, size_t ws_bytes) {
    (void)comm; (void)in_dtypes; (void)out_dtype;
    if (nranks != NR || n_in != 4 || (unsigned)rank >= (unsigned)NR || ws_bytes < WS_NEED ||
        in_sizes[0] != MSH * KD || in_sizes[1] != KD * NR * NSH || out_size != NR * MSH * NSH) {
        fprintf(stderr, "kernel_launch: unexpected problem shape\n");
        return;
    }
    Args a;
    memset(&a, 0, sizeof(a));
    constexpr int GROUP = 0xb2;
    int order[NR], n = 0;
    for (int pass = 0; pass < 2; ++pass)
        for (int r = 0; r < NR; ++r) {
            const int d = (rank + r) % NR;
            const bool same = ((GROUP >> d) & 1) == ((GROUP >> rank) & 1);
            if (same == (pass == 0)) order[n++] = d;
        }
    for (int r = 0; r < NR; ++r) {
        const int dest = order[r];
        Rec& c = a.rec[r];
        c.x = (const u8*)d_in[0];
        c.w = (const u8*)d_in[1] + dest * NSH;
        c.out = (u8*)(r ? d_out_all[dest] : d_out);
        c.dws = (u8*)d_ws_all[dest];
        c.myws = (u8*)d_ws_all[rank];
        c.sx = (const float*)d_in[2];
        c.sw = (const float*)d_in[3];
        c.rank = (unsigned)rank;
        c.slot = (unsigned)((dest - rank - 1) & (NR - 1));
        if (r) {
            a.xdst[r - 1] = a.xdst[r + 6] = c.dws + WS_XST + (size_t)c.slot * (XM * KD);
            a.xflag[r - 1] = c.dws + WS_XFLAG + c.slot * (56 * 4);
        }
    }
    a.xdst[14] = a.xdst[15] = a.xdst[0];
    a.xflag[7] = a.xflag[0];
    hipLaunchKernelGGL(fused_kernel, dim3(NWG), dim3(NT), 0, stream, a);
    HIP_CHECK(hipGetLastError());
}
```
